```python
import numpy as np

kernel_name = "gpu_gemm_ar_m4096_k4096_n4096_e4m3e5m2_dqrelu"


ABSMAX_RTOL = 0.05

COMPUTE_DTYPE = "float32"
EPILOGUE = "dequant_relu"
IN_QUANT = None
OUT_QUANT = None
IN_FP8 = "e4m3e5m2"
IN_DTYPES = {"x": "e4m3", "w_mat": "e5m2",
             "scale_x": "float32", "scale_w": "float32"}
WORLD_SIZE = 8

def _snap_fp8(x, mant_bits, max_abs, min_normal):
    a = np.abs(x.astype(np.float64))
    _m, _e = np.frexp(a)
    q = np.where(a >= min_normal, np.ldexp(1.0, _e - 1 - mant_bits),
                 min_normal * 2.0 ** -mant_bits)
    snapped = np.minimum(np.round(a / q) * q, max_abs)
    return (np.sign(x) * snapped).astype(np.float32)

def setup_inputs():

    x = _snap_fp8(np.random.randn(4096, 4096).astype(np.float32), 3, 448.0, 2.0 ** -6)
    w_mat = _snap_fp8(np.random.randn(4096, 4096).astype(np.float32), 2, 57344.0, 2.0 ** -14)
    scale_x = (0.02 * (1.0 + np.random.rand(1))).astype(np.float32)
    scale_w = (0.02 * (1.0 + np.random.rand(1))).astype(np.float32)
    return {"x": x, "w_mat": w_mat, "scale_x": scale_x, "scale_w": scale_w}

def reference(x, w_mat, scale_x, scale_w):
    acc = (x @ w_mat).astype(np.float32)
    y = (acc * (scale_x * scale_w)).astype(np.float32)
    return np.maximum(y, 0.0).astype(np.float32)
```

```cpp
#include <hip/hip_runtime.h>
#include <rccl/rccl.h>
#include <cstdio>
#include <cstdint>

#define HIP_CHECK(expr) do { hipError_t _e = (expr); if (_e != hipSuccess) { \
    fprintf(stderr, "HIP error %s at %s:%d\n", hipGetErrorString(_e), __FILE__, __LINE__); \
    return; } } while (0)

#define NPUSH 16
#define PUSH_D 4

namespace {

typedef unsigned u32x4 __attribute__((ext_vector_type(4)));
typedef float f32x16 __attribute__((ext_vector_type(16)));
#define LAS __attribute__((address_space(3)))

constexpr int NR = 8;
constexpr int DIM = 4096;
constexpr int KLOC = DIM / NR;
constexpr int NCH = 8;
constexpr int NBLK = 32;
constexpr unsigned ITEM = 16384;
constexpr size_t STAGE_PAR = (size_t)NR * NCH * NBLK * ITEM;
constexpr size_t FLAG_OFF = 2 * STAGE_PAR;
constexpr size_t CTR_OFF = FLAG_OFF + 32768;
constexpr size_t TMO_OFF = CTR_OFF + 32768;
constexpr size_t DBG_OFF = TMO_OFF + 4096;
constexpr size_t WS_NEED = DBG_OFF + 256 * 256;
constexpr unsigned long long DEADLINE_TICKS = 50000000ull;

constexpr int L_FULL = 0;
constexpr int L_FREE = 512;
constexpr int L_SCHED = 1024;
constexpr int L_MISC = 1280;
constexpr int L_CTRL_BYTES = 2048;
constexpr int L_RING = 16384;
constexpr int L_TOTAL = L_RING + 4 * 32768;

struct Params {
    const unsigned char* x;
    const unsigned char* w;
    const float* sx;
    const float* sw;
    float* out;
    char* myws;
    char* dws[NR];
    int rank;
};

__device__ __forceinline__ __amdgpu_buffer_rsrc_t make_rsrc(const void* p, unsigned bytes) {
    return __builtin_amdgcn_make_buffer_rsrc((void*)p, 0, (int)bytes, 0x00020000);
}

__device__ __forceinline__ void store16_wt(u32x4 v, __amdgpu_buffer_rsrc_t r, int voff, int soff) {
    __builtin_amdgcn_raw_buffer_store_b128(v, r, voff, soff, 17);
    __builtin_amdgcn_sched_barrier(0);
    asm volatile("s_nop 1" ::: "memory");
    __builtin_amdgcn_sched_barrier(0);
}

template <int N> __device__ __forceinline__ void wait_vm() {
    static_assert(N >= 0, "");
    asm volatile("s_waitcnt vmcnt(%0)" :: "i"(N > 63 ? 63 : N) : "memory");
}

#define TSTAMP(idx) do {} while (0)

constexpr int younger_ops(int j, int D, int L) {
    int back = D;
    int cnt = D - 1;
    int groups = 0;
    int pos = j;
    while (back > 0) {
        if (pos == 0) { groups++; pos = 4; }
        pos--; back--;
    }
    return cnt + groups * L;
}

template <bool IS_W, int D>
__device__ __forceinline__ void push_loop(const unsigned char* srcp, char* dws, int rank, int blk0, int stride, int nblk_log2,
                                          unsigned* ctr, int lane) {
    constexpr int L = IS_W ? 16 : 4;
    const __amdgpu_buffer_rsrc_t src = make_rsrc(srcp, DIM * KLOC);
    const int voff_ld = IS_W ? lane * 4 : (lane & 15) * KLOC + (lane >> 4) * 16;
    const int nitems = NCH << nblk_log2;
    unsigned raw[4][16];
    auto item_blk = [&](int it) { return blk0 + (it & ((1 << nblk_log2) - 1)) * stride; };
    auto item_ch = [&](int it) { return it >> nblk_log2; };
    auto load_part = [&](int it, int part, unsigned (&r)[16]) {
        it = it < nitems ? it : nitems - 1;
        const int b = item_blk(it), c = item_ch(it);
        if constexpr (IS_W) {
            const int so = (c * 64 + part * 16) * DIM + b * 256;
            #pragma unroll
            for (int k = 0; k < 16; ++k) r[k] = __builtin_amdgcn_raw_buffer_load_b32(src, voff_ld, so + k * DIM, 0);
        } else {
            const int so = (b * 256 + part * 64) * KLOC + c * 64;
            #pragma unroll
            for (int t = 0; t < 4; ++t) {
                u32x4 v = __builtin_amdgcn_raw_buffer_load_b128(src, voff_ld, so + t * 16 * KLOC, 0);
                r[4 * t] = v.x; r[4 * t + 1] = v.y; r[4 * t + 2] = v.z; r[4 * t + 3] = v.w;
            }
        }
    };
    const unsigned prev = __hip_atomic_load(ctr, __ATOMIC_RELAXED, __HIP_MEMORY_SCOPE_AGENT);
    load_part(0, 0, raw[0]);
    load_part(0, 1, raw[1]);
    __builtin_amdgcn_s_barrier();
    const unsigned call = __builtin_amdgcn_readfirstlane(prev) + 1;
    __hip_atomic_store(ctr, call, __ATOMIC_RELAXED, __HIP_MEMORY_SCOPE_AGENT);
    const __amdgpu_buffer_rsrc_t dstr = make_rsrc(dws + (size_t)(call & 1) * STAGE_PAR + (size_t)rank * (NCH * NBLK * ITEM), NCH * NBLK * ITEM);
    const __amdgpu_buffer_rsrc_t flr = make_rsrc(dws + FLAG_OFF, 32768);
    const int voff_st = lane * 16;
    const int voff_fl = lane < 8 ? lane * 4 : 0x7ffffff0;
    const unsigned fbase = call * 8;
    static_assert(D >= 2 && D <= 12, "");
    auto do_item = [&](int it, bool first) __attribute__((always_inline)) {
        const int b = item_blk(it) + (IS_W ? 16 : 0), c = item_ch(it);
        const int so_item = (c * NBLK + b) * ITEM;
        #pragma unroll
        for (int part = 0; part < 4; ++part) {
            __builtin_amdgcn_sched_barrier(0);
            load_part(it + (part + 2) / 4, (part + 2) & 3, raw[(part + 2) & 3]);
            __builtin_amdgcn_sched_barrier(0);
            unsigned (&r)[16] = raw[part];
            u32x4 o[4];
            if constexpr (IS_W) {
                #pragma unroll
                for (int g = 0; g < 4; ++g) {
                    const unsigned t0 = __builtin_amdgcn_perm(r[4 * g + 1], r[4 * g], 0x05010400u);
                    const unsigned t1 = __builtin_amdgcn_perm(r[4 * g + 1], r[4 * g], 0x07030602u);
                    const unsigned t2 = __builtin_amdgcn_perm(r[4 * g + 3], r[4 * g + 2], 0x05010400u);
                    const unsigned t3 = __builtin_amdgcn_perm(r[4 * g + 3], r[4 * g + 2], 0x07030602u);
                    o[0][g] = __builtin_amdgcn_perm(t2, t0, 0x05040100u);
                    o[1][g] = __builtin_amdgcn_perm(t2, t0, 0x07060302u);
                    o[2][g] = __builtin_amdgcn_perm(t3, t1, 0x05040100u);
                    o[3][g] = __builtin_amdgcn_perm(t3, t1, 0x07060302u);
                }
            } else {
                #pragma unroll
                for (int t = 0; t < 4; ++t) o[t] = u32x4{r[4 * t], r[4 * t + 1], r[4 * t + 2], r[4 * t + 3]};
            }
            #pragma unroll
            for (int j = 0; j < 4; ++j) {
                switch (j) {
                    case 0: wait_vm<younger_ops(0, D, L)>(); break;
                    case 1: wait_vm<younger_ops(1, D, L)>(); break;
                    case 2: wait_vm<younger_ops(2, D, L)>(); break;
                    default: wait_vm<younger_ops(3, D, L)>(); break;
                }
                if (part * 4 + j == D - 1 && !first) {
                    const int pb = item_blk(it - 1) + (IS_W ? 16 : 0), pc = item_ch(it - 1) + 1;
                    __builtin_amdgcn_raw_buffer_store_b32(fbase + pc, flr, voff_fl, (pb * NR + rank) * 128, 17);
                }
                const int unit = IS_W ? (j * 4 + part) : (part * 4 + j);
                store16_wt(o[j], dstr, voff_st, so_item + unit * 1024);
            }
        }
    };
    do_item(0, true);
    #pragma nounroll
    for (int it = 1; it < nitems; ++it) do_item(it, false);
    wait_vm<0>();
    {
        const int pb = item_blk(nitems - 1) + (IS_W ? 16 : 0);
        __builtin_amdgcn_raw_buffer_store_b32(fbase + NCH, flr, voff_fl, (pb * NR + rank) * 128, 17);
    }
    wait_vm<0>();
}

#define MFMA(c, a, b) "v_mfma_f32_32x32x64_f8f6f4 %[" #c "], " a ", " b ", %[" #c "] cbsz:0 blgp:1\n\t"
#define X0 "v[136:143]"
#define X1 "v[144:151]"
#define Y0 "v[152:159]"
#define Y1 "v[160:167]"
#define STR2(x) #x
#define STR(x) STR2(x)
#define STEP(SO, FS, NS) \
    "1:\n\t" \
    "s_waitcnt lgkmcnt(0)\n\t" \
    "v_readlane_b32 %[st], %[vf], %[sw]\n\t" \
    "s_nop 0\n\t" \
    "s_cmp_ge_u32 %[st], %[need]\n\t" \
    "s_cbranch_scc1 2f\n\t" \
    "s_sleep 1\n\t" \
    "ds_read_addtid_b32 %[vf] offset:" STR(FS) "*32\n\t" \
    "s_branch 1b\n\t" \
    "2:\n\t" \
    "ds_read_b128 v[136:139], %[va] offset:" STR(SO) "\n\t" \
    "ds_read_b128 v[140:143], %[va] offset:" STR(SO) "+256\n\t" \
    "ds_read_b128 v[152:155], %[vb] offset:" STR(SO) "\n\t" \
    "ds_read_b128 v[156:159], %[vb] offset:" STR(SO) "+1024\n\t" \
    "ds_read_b128 v[144:147], %[va] offset:" STR(SO) "+2048\n\t" \
    "ds_read_b128 v[148:151], %[va] offset:" STR(SO) "+2304\n\t" \
    "ds_read_b128 v[160:163], %[vb] offset:" STR(SO) "+4096\n\t" \
    "ds_read_b128 v[164:167], %[vb] offset:" STR(SO) "+5120\n\t" \
    "s_waitcnt lgkmcnt(4)\n\t" \
    MFMA(c00, X0, Y0) \
    "s_waitcnt lgkmcnt(2)\n\t" \
    MFMA(c10, X1, Y0) \
    "ds_read_b128 v[152:155], %[vb] offset:" STR(SO) "+8192\n\t" \
    "ds_read_b128 v[156:159], %[vb] offset:" STR(SO) "+9216\n\t" \
    "s_waitcnt lgkmcnt(2)\n\t" \
    MFMA(c01, X0, Y1) \
    MFMA(c11, X1, Y1) \
    "ds_read_b128 v[160:163], %[vb] offset:" STR(SO) "+12288\n\t" \
    "ds_read_b128 v[164:167], %[vb] offset:" STR(SO) "+13312\n\t" \
    "s_waitcnt lgkmcnt(2)\n\t" \
    MFMA(c02, X0, Y0) \
    MFMA(c12, X1, Y0) \
    "s_waitcnt lgkmcnt(0)\n\t" \
    "s_mov_b64 exec, %[mask]\n\t" \
    "ds_write_addtid_b32 %[vf] offset:512+" STR(FS) "*32\n\t" \
    "s_mov_b64 exec, -1\n\t" \
    "ds_read_addtid_b32 %[vf] offset:" STR(NS) "*32\n\t" \
    MFMA(c03, X0, Y1) \
    MFMA(c13, X1, Y1)

__global__ void __launch_bounds__(768, 3) fused_ag_gemm(Params P) {
    __shared__ __attribute__((aligned(1024))) unsigned char lds[L_TOTAL];
    LAS unsigned char* L8 = (LAS unsigned char*)lds;
    const int wave = __builtin_amdgcn_readfirstlane(threadIdx.x >> 6);
    const int lane = threadIdx.x & 63;
    const int wg = blockIdx.x;
    const int e = wg & 7, idx = wg >> 3;
    const int ti = (e >> 1) + 4 * (idx >> 3);
    const int tj = (e & 1) + 2 * (idx & 7);
    char* const myws = P.myws;
    const int rank = P.rank;

    if (wave < 8) {
        if (wave == 0) {
            #pragma unroll
            for (int i = 0; i < L_CTRL_BYTES / 256; ++i) ((LAS unsigned*)L8)[i * 64 + lane] = 0u;
            asm volatile("s_waitcnt lgkmcnt(0)" ::: "memory");
            TSTAMP(0);
        }
        __builtin_amdgcn_s_barrier();
        if (wave >= 4) __builtin_amdgcn_s_setprio(1);
        const int wm = wave >> 1, wn = wave & 1;
        const int r = lane & 31, h = lane >> 5;
        unsigned va = L_RING + wm * 4096 + (r >> 4) * 1024 + h * 512 + (r & 15) * 16;
        unsigned vb = L_RING + 16384 + h * 2048 + wn * 512 + r * 16;
        f32x16 c00 = {}, c01 = {}, c02 = {}, c03 = {}, c10 = {}, c11 = {}, c12 = {}, c13 = {};
        unsigned vf, st, need, keep;
        const unsigned long long mask = 1ull << wave;
        asm volatile(
            "s_mov_b32 %[keep], m0\n\t"
            "s_mov_b32 m0, 0\n\t"
            "s_mov_b32 %[need], 2\n\t"
            "ds_read_addtid_b32 %[vf] offset:0\n\t"
            "3:\n\t"
            STEP(0, 0, 1)
            STEP(32768, 1, 2)
            "v_xor_b32 %[va], 0x10000, %[va]\n\t"
            "v_xor_b32 %[vb], 0x10000, %[vb]\n\t"
            STEP(0, 2, 3)
            STEP(32768, 3, 0)
            "v_xor_b32 %[va], 0x10000, %[va]\n\t"
            "v_xor_b32 %[vb], 0x10000, %[vb]\n\t"
            "s_add_u32 %[need], %[need], 2\n\t"
            "s_cmp_le_u32 %[need], 32\n\t"
            "s_cbranch_scc1 3b\n\t"
            "s_waitcnt lgkmcnt(0)\n\t"
            "s_mov_b32 m0, %[keep]\n\t"
            "s_nop 15\n\t"
            "s_nop 7\n\t"
            : [c00] "+v"(c00), [c01] "+v"(c01), [c02] "+v"(c02), [c03] "+v"(c03),
              [c10] "+v"(c10), [c11] "+v"(c11), [c12] "+v"(c12), [c13] "+v"(c13),
              [va] "+v"(va), [vb] "+v"(vb), [vf] "=&v"(vf), [st] "=&s"(st), [need] "=&s"(need), [keep] "=&s"(keep)
            : [sw] "s"(wave), [mask] "s"(mask)
            : "memory", "scc",
              "v136", "v137", "v138", "v139", "v140", "v141", "v142", "v143", "v144", "v145", "v146", "v147",
              "v148", "v149", "v150", "v151", "v152", "v153", "v154", "v155", "v156", "v157", "v158", "v159",
              "v160", "v161", "v162", "v163", "v164", "v165", "v166", "v167");
        if (wave == 0) TSTAMP(4);
        unsigned verdict;
        while ((verdict = __builtin_amdgcn_readfirstlane(*(volatile LAS unsigned*)(L8 + L_MISC))) == 0u) __builtin_amdgcn_s_sleep(1);
        float s = P.sx[0] * P.sw[0];
        if (verdict != 1u) s = __builtin_nanf("");
        int lz = __builtin_amdgcn_mbcnt_hi(~0u, __builtin_amdgcn_mbcnt_lo(~0u, 0u));
        const __amdgpu_buffer_rsrc_t outr = make_rsrc(P.out + (size_t)(ti * 256 + wm * 64) * DIM + tj * 256 + wn * 128, 64u * DIM * 4u);
        const int voff = ((lz >> 5) * 4 * DIM + (lz & 31) * 4) * 4;
        auto act = [&](float a) { float y = a * s; return __builtin_bit_cast(unsigned, y < 0.f ? 0.f : y); };
        #pragma unroll
        for (int g = 0; g < 16; ++g) {
            const int row = (g & 3) + 8 * (g >> 2);
            u32x4 v0 = {act(c00[g]), act(c01[g]), act(c02[g]), act(c03[g])};
            store16_wt(v0, outr, voff, row * DIM * 4);
            u32x4 v1 = {act(c10[g]), act(c11[g]), act(c12[g]), act(c13[g])};
            store16_wt(v1, outr, voff, (32 + row) * DIM * 4);
        }
    } else if (wave == 8) {
        __builtin_amdgcn_s_setprio(3);
        unsigned* ctr = (unsigned*)(myws + CTR_OFF + wg * 128);
        char* const dws = P.dws[e];
        constexpr int SP = 32 / NPUSH;
        constexpr int HALF = NPUSH / 2;
        constexpr int NB_LOG2 = NPUSH == 8 ? 2 : NPUSH == 16 ? 1 : 0;
        if (e == 0 || idx % SP == 0) {
            const int j = e == 0 ? idx : idx / SP;
            const int half = e == 0 ? 16 : HALF;
            const int nbl = e == 0 ? 0 : NB_LOG2;
            if (j < half) push_loop<false, PUSH_D>(P.x, dws, rank, j, half, nbl, ctr, lane);
            else push_loop<true, PUSH_D>(P.w, dws, rank, j - half, half, nbl, ctr, lane);
            TSTAMP(1);
        } else {
            __builtin_amdgcn_s_barrier();
        }
    } else if (wave == 9) {
        __builtin_amdgcn_s_setprio(3);
        unsigned* ctr = (unsigned*)(myws + CTR_OFF + wg * 128 + 64);
        unsigned* tmo = (unsigned*)(myws + TMO_OFF);
        const unsigned prev = __hip_atomic_load(ctr, __ATOMIC_RELAXED, __HIP_MEMORY_SCOPE_AGENT);
        __builtin_amdgcn_s_barrier();
        const unsigned call = __builtin_amdgcn_readfirstlane(prev) + 1;
        __hip_atomic_store(ctr, call, __ATOMIC_RELAXED, __HIP_MEMORY_SCOPE_AGENT);
        const int p = (rank + (lane >> 1)) & 7, o = lane & 1;
        const unsigned* fl = (const unsigned*)(myws + FLAG_OFF + (size_t)(((o ? 16 + tj : ti) * NR + p) * 128));
        volatile LAS unsigned* sched = (volatile LAS unsigned*)(L8 + L_SCHED);
        const unsigned base = call * 8;
        const unsigned code0 = ((call & 1) * 64 + p * 8) + 1;
        int cur = 0, nsched = 0;
        bool dead = false;
        const unsigned long long t0 = __builtin_amdgcn_s_memrealtime();
        int spins = 0;
        while (nsched < 64) {
            int av = 8;
            if (!dead) {
                if (lane < 16) {
                    av = (int)(__hip_atomic_load(fl, __ATOMIC_RELAXED, __HIP_MEMORY_SCOPE_SYSTEM) - base);
                    av = av < 0 ? 0 : (av > 8 ? 8 : av);
                }
                const int other = __builtin_amdgcn_update_dpp(0, av, 0xB1, 0xf, 0xf, false);
                av = av < other ? av : other;
            }
            for (;;) {
                const bool rdy = lane < 16 && av > cur;
                const unsigned long long m = __ballot(rdy) & 0x5555ull;
                if (m == 0) break;
                if (rdy && o == 0) sched[nsched + __builtin_popcountll(m & ((1ull << lane) - 1))] = code0 + cur;
                cur += rdy ? 1 : 0;
                nsched += __builtin_popcountll(m);
            }
            if (nsched >= 64) break;
            if ((++spins & 63) == 0) {
                if (__hip_atomic_load(tmo, __ATOMIC_RELAXED, __HIP_MEMORY_SCOPE_AGENT) == call ||
                    __builtin_amdgcn_s_memrealtime() - t0 > DEADLINE_TICKS) {
                    if (lane == 0) __hip_atomic_store(tmo, call, __ATOMIC_RELAXED, __HIP_MEMORY_SCOPE_AGENT);
                    dead = true;
                }
            }
            __builtin_amdgcn_s_sleep(2);
        }
        TSTAMP(2);
        if (__hip_atomic_load(tmo, __ATOMIC_RELAXED, __HIP_MEMORY_SCOPE_AGENT) == call) dead = true;
        *(volatile LAS unsigned*)(L8 + L_MISC) = dead ? 2u : 1u;
    } else {
        __builtin_amdgcn_s_setprio(3);
        __builtin_amdgcn_s_barrier();
        const int o = wave - 10;
        const int blk = o ? 16 + tj : ti;
        volatile LAS unsigned* sched = (volatile LAS unsigned*)(L8 + L_SCHED);
        volatile LAS unsigned* freew = (volatile LAS unsigned*)(L8 + L_FREE);
        const char* stage = myws + (size_t)blk * ITEM;
        const unsigned v0 = lane * 16, v1 = v0 + 4096, v2 = v0 + 8192, v3 = v0 + 12288;
        const unsigned vfull = L_FULL + (lane & 7) * 4, vone = lane < 8 ? 1u : 0u;
        int issued = 0, pub = 0;
        while (pub < 64) {
            bool did = false;
            if (issued < 64 && issued - pub < 3) {
                const unsigned code = __builtin_amdgcn_readfirstlane(sched[issued]);
                const int slot = issued & 3;
                const unsigned fr = freew[slot * 8 + (lane & 7)];
                const bool free_ok = __all(fr >= (unsigned)(issued >> 2) * 2u);
                if (code != 0u && free_ok) {
                    const char* src = stage + (size_t)((code - 1) & 127u) * (NBLK * ITEM);
                    const unsigned ldst = L_RING + slot * 32768 + o * 16384;
                    unsigned keep;
#define DMA4(V) "global_load_lds_dwordx4 " V ", %[src] sc0 sc1\n\t" \
                "global_load_lds_dwordx4 " V ", %[src] offset:1024 sc0 sc1\n\t" \
                "global_load_lds_dwordx4 " V ", %[src] offset:2048 sc0 sc1\n\t" \
                "global_load_lds_dwordx4 " V ", %[src] offset:3072 sc0 sc1\n\t"
                    asm volatile(
                        "s_mov_b32 %[keep], m0\n\t"
                        "s_mov_b32 m0, %[ld]\n\t"
                        "s_nop 4\n\t"
                        DMA4("%[v0]")
                        "s_add_u32 m0, m0, 4096\n\t"
                        "s_nop 0\n\t"
                        DMA4("%[v1]")
                        "s_add_u32 m0, m0, 4096\n\t"
                        "s_nop 0\n\t"
                        DMA4("%[v2]")
                        "s_add_u32 m0, m0, 4096\n\t"
                        "s_nop 0\n\t"
                        DMA4("%[v3]")
                        "s_mov_b32 m0, %[keep]\n\t"
                        : [keep] "=&s"(keep)
                        : [src] "s"(src), [ld] "s"(ldst), [v0] "v"(v0), [v1] "v"(v1), [v2] "v"(v2), [v3] "v"(v3)
                        : "memory", "scc");
                    ++issued;
                    did = true;
                }
            }
            if (!did) {
                if (issued > pub) {
                    const int fl = issued - pub;
                    if (fl == 1) wait_vm<0>(); else if (fl == 2) wait_vm<16>(); else wait_vm<32>();
                    if (lane < 8) asm volatile("ds_add_u32 %0, %1" :: "v"(vfull + (pub & 3) * 32), "v"(vone) : "memory");
                    ++pub;
                } else {
                    __builtin_amdgcn_s_sleep(1);
                }
            }
        }
        if (o == 0) TSTAMP(3);
    }
}

}

extern "C" void kernel_launch(int rank, int nranks, ncclComm_t comm,
                              hipStream_t stream,
                              void* const* d_in, const int* in_sizes, int n_in,
                              void* d_out, int out_size,
                              const int* in_dtypes, int out_dtype,
                              void* const* d_out_all,
                              void* const* d_ws_all, size_t ws_bytes) {
    if (nranks != NR || n_in < 4 || ws_bytes < WS_NEED || (unsigned)rank >= (unsigned)NR ||
        in_sizes[0] != DIM * KLOC || in_sizes[1] != DIM * KLOC || out_size != DIM * DIM) {
        fprintf(stderr, "kernel_launch: unexpected problem shape\n");
        return;
    }
    Params P;
    P.x = (const unsigned char*)d_in[0];
    P.w = (const unsigned char*)d_in[1];
    P.sx = (const float*)d_in[2];
    P.sw = (const float*)d_in[3];
    P.out = (float*)d_out;
    P.myws = (char*)d_ws_all[rank];
    for (int e = 0; e < NR; ++e) P.dws[e] = (char*)d_ws_all[(rank + e) % NR];
    P.rank = rank;
    hipLaunchKernelGGL(fused_ag_gemm, dim3(256), dim3(768), 0, stream, P);
    HIP_CHECK(hipGetLastError());
}
```

```cpp
#include <hip/hip_runtime.h>
#include <rccl/rccl.h>
#include <cstdio>
#include <cstdint>
#include <type_traits>

#define HIP_CHECK(expr) do { hipError_t _e = (expr); if (_e != hipSuccess) { \
    fprintf(stderr, "HIP error %s at %s:%d\n", hipGetErrorString(_e), __FILE__, __LINE__); \
    return; } } while (0)

#define TAU16 444
#define TOFF 100
#define NSPLIT 0
#define S_FULL 31
#define S_LAST 7
#define S_LOW 23
#define S_LOX 31
#define S_HI 15
#define DCAP 2
#define DCAP_LAST 8
#define ROT 0
#define EPS 1

namespace {

typedef unsigned u32x4 __attribute__((ext_vector_type(4)));
typedef float f32x16 __attribute__((ext_vector_type(16)));
#define LAS __attribute__((address_space(3)))

constexpr int NR = 8;
constexpr int DIM = 4096;
constexpr int KLOC = DIM / NR;
constexpr int NCH = 8;
constexpr int NBLK = 32;
constexpr int NS = NSPLIT, NF = NCH - NS;
constexpr int NSTEP = NR * (NCH + NS);
constexpr unsigned ITEM = 16384;
constexpr size_t STAGE_PAR = (size_t)NR * NCH * NBLK * ITEM;
constexpr size_t FLAG_OFF = 2 * STAGE_PAR;
constexpr size_t CTR_OFF = FLAG_OFF + 32768;
constexpr size_t TMO_OFF = CTR_OFF + 32768;
constexpr size_t DBG_OFF = TMO_OFF + 4096;
constexpr size_t WS_NEED = DBG_OFF + 256 * 1024;
constexpr unsigned long long DEADLINE_TICKS = 50000000ull;
static_assert(NS >= 0 && NS <= 4, "");

constexpr int L_FULL = 0;
constexpr int L_FREE = 512;
constexpr int L_SCHED = 1024;
constexpr int L_SCALE = 1536;
constexpr int L_CTRL_BYTES = 2048;
constexpr int L_RING = 8192;
constexpr int L_TOTAL = L_RING + 6 * 24576;

struct Params {
    const unsigned char* x;
    const unsigned char* w;
    const float* sx;
    const float* sw;
    float* out;
    char* myws;
    char* dws[NR];
    int rank;
    int rot[NR];
};

__host__ __device__ constexpr int grp_chunk(bool is_w, int g) { return (is_w || g < 2 * NF) ? g >> 1 : (g < 2 * NF + NS ? g - NF : g - NF - NS); }
__host__ __device__ constexpr int grp_half(bool is_w, int g) { return (is_w || g < 2 * NF) ? g & 1 : (g < 2 * NF + NS ? 0 : 1); }
__host__ __device__ constexpr int grp_prog(bool is_w, int g) { return (is_w || g < 2 * NF) ? ((g & 1) ? (g >> 1) + 1 : 0) : g - NF + 1; }

struct Sched { unsigned t[NBLK][16]; };
constexpr int store_key(int b, int g, int u) {
    const bool is_w = b >= 16;
    const int v = b & 15, c = grp_chunk(is_w, g), h = grp_half(is_w, g);
    int end = 0, stride = 0, back = 0;
    if (c < NF) { end = (c * 32 + 2 * v + (is_w ? 2 : 1)) * 16; stride = c == 7 ? S_LAST : S_FULL; back = h ? 0 : 8; }
    else if (is_w) { end = NF * 512 + (c - NF) * 384 + v * 24 + 16; stride = S_LOW; back = h ? 0 : 8; }
    else if (h == 0) { end = NF * 512 + (c - NF) * 384 + v * 24 + 24; stride = S_LOX; back = 0; }
    else { end = NF * 512 + NS * 384 + (c - NF) * 128 + v * 8 + 8; stride = S_HI; back = 0; }
    return end - (7 - u + back) * stride + 1024;
}
constexpr Sched make_sched() {
    Sched r = {};
    unsigned short cnt[5184] = {};
    for (int b = 0; b < NBLK; ++b) for (int g = 0; g < 16; ++g) for (int u = 0; u < 8; ++u) ++cnt[store_key(b, g, u) + 1];
    for (int k = 1; k < 5184; ++k) cnt[k] += cnt[k - 1];
    for (int b = 0; b < NBLK; ++b)
        for (int g = 0; g < 16; ++g) {
            unsigned first = 0, last = 0;
            for (int u = 0; u < 8; ++u) { last = cnt[store_key(b, g, u)]++; if (u == 0) first = last; }
            r.t[b][g] = (TOFF * 4 + first * TAU16 / 64) | ((TOFF * 4 + last * TAU16 / 64) << 16);
        }
    return r;
}
__device__ constexpr Sched SEND_SCHED = make_sched();

__device__ __forceinline__ __amdgpu_buffer_rsrc_t make_rsrc(const void* p, unsigned bytes) {
    return __builtin_amdgcn_make_buffer_rsrc((void*)p, 0, (int)bytes, 0x00020000);
}

__device__ __forceinline__ void store16_wt(u32x4 v, __amdgpu_buffer_rsrc_t r, int voff, int soff) {
    __builtin_amdgcn_raw_buffer_store_b128(v, r, voff, soff, 17);
    __builtin_amdgcn_sched_barrier(0);
    asm volatile("s_nop 1" ::: "memory");
    __builtin_amdgcn_sched_barrier(0);
}

template <int I, int N, typename F> __device__ __forceinline__ void static_for(F&& f) {
    if constexpr (I < N) { f(std::integral_constant<int, I>{}); static_for<I + 1, N>(f); }
}

template <int N> __device__ __forceinline__ void wait_vm() {
    static_assert(N >= 0 && N <= 63, "");
    asm volatile("s_waitcnt vmcnt(%0)" :: "i"(N) : "memory");
}

#define CSTAMP(idx) do {} while (0)
#define TSTAMP(idx) do {} while (0)

template <bool IS_W>
__device__ __forceinline__ void push_loop(const unsigned char* srcp, char* dws, int rank, int blk, bool paced, const unsigned* tsched,
                                          unsigned* ctr, int lane, unsigned* dbg) {
    const unsigned long long t0 = __builtin_amdgcn_s_memrealtime();
    const __amdgpu_buffer_rsrc_t src = make_rsrc(srcp, DIM * KLOC);
    const int voff_ld = IS_W ? lane * 4 : (lane & 15) * KLOC + (lane >> 4) * 16;
    unsigned rawA[32], rawB[32];
    auto load_grp = [&](int g, unsigned (&raw)[32]) __attribute__((always_inline)) {
        const int c = grp_chunk(IS_W, g), h = grp_half(IS_W, g);
        if constexpr (IS_W) {
            const int so = (c * 64 + h * 32) * DIM + blk * 256;
            #pragma unroll
            for (int k = 0; k < 32; ++k) raw[k] = __builtin_amdgcn_raw_buffer_load_b32(src, voff_ld, so + k * DIM, 0);
        } else {
            const int so = (blk * 256 + h * 128) * KLOC + c * 64;
            #pragma unroll
            for (int t = 0; t < 8; ++t) {
                u32x4 v = __builtin_amdgcn_raw_buffer_load_b128(src, voff_ld, so + t * 16 * KLOC, 0);
                raw[4 * t] = v.x; raw[4 * t + 1] = v.y; raw[4 * t + 2] = v.z; raw[4 * t + 3] = v.w;
            }
        }
    };
    const unsigned prev = __hip_atomic_load(ctr, __ATOMIC_RELAXED, __HIP_MEMORY_SCOPE_AGENT);
    load_grp(0, rawA);
    __builtin_amdgcn_s_barrier();
    const unsigned call = __builtin_amdgcn_readfirstlane(prev) + 1;
    __hip_atomic_store(ctr, call, __ATOMIC_RELAXED, __HIP_MEMORY_SCOPE_AGENT);
    const __amdgpu_buffer_rsrc_t dstr = make_rsrc(dws + (size_t)(call & 1) * STAGE_PAR + (size_t)rank * (NCH * NBLK * ITEM), NCH * NBLK * ITEM);
    const __amdgpu_buffer_rsrc_t flr = make_rsrc(dws + FLAG_OFF, 32768);
    const int voff_st = lane * 16;
    const int voff_fl = lane < 8 ? lane * 4 : 0x7ffffff0;
    const int b = blk + (IS_W ? 16 : 0);
    auto send_grp = [&](int g, unsigned (&raw)[32], unsigned (&nxt)[32]) __attribute__((always_inline)) {
        u32x4 o[8];
        if constexpr (IS_W) {
            #pragma unroll
            for (int r = 0; r < 8; ++r) {
                const unsigned t0_ = __builtin_amdgcn_perm(raw[4 * r + 1], raw[4 * r], 0x05010400u);
                const unsigned t1_ = __builtin_amdgcn_perm(raw[4 * r + 1], raw[4 * r], 0x07030602u);
                const unsigned t2_ = __builtin_amdgcn_perm(raw[4 * r + 3], raw[4 * r + 2], 0x05010400u);
                const unsigned t3_ = __builtin_amdgcn_perm(raw[4 * r + 3], raw[4 * r + 2], 0x07030602u);
                o[0 + (r >> 2)][r & 3] = __builtin_amdgcn_perm(t2_, t0_, 0x05040100u);
                o[2 + (r >> 2)][r & 3] = __builtin_amdgcn_perm(t2_, t0_, 0x07060302u);
                o[4 + (r >> 2)][r & 3] = __builtin_amdgcn_perm(t3_, t1_, 0x05040100u);
                o[6 + (r >> 2)][r & 3] = __builtin_amdgcn_perm(t3_, t1_, 0x07060302u);
            }
        } else {
            #pragma unroll
            for (int t = 0; t < 8; ++t) o[t] = u32x4{raw[4 * t], raw[4 * t + 1], raw[4 * t + 2], raw[4 * t + 3]};
        }
        #pragma unroll
        for (int u = 0; u < 8; ++u) asm volatile("" : "+v"(o[u]));
        const int c = grp_chunk(IS_W, g), h = grp_half(IS_W, g);
        const int so = (c * NBLK + b) * ITEM + h * (IS_W ? 2048 : 8192);
        const unsigned ts = tsched[g];
        const unsigned ta = ts & 0xffffu, tz = ts >> 16;
        #pragma unroll
        for (int u = 0; u < 8; ++u) {
            if (paced) {
                const unsigned long long tgt = t0 + ((ta * 7 + (tz - ta) * u) / 28);
                for (;;) {
                    const long long left = (long long)(tgt - __builtin_amdgcn_s_memrealtime());
                    if (left <= 0) break;
                    if (left > 40) __builtin_amdgcn_s_sleep(8); else __builtin_amdgcn_s_sleep(1);
                }
            }
            if (paced) {
                static_for<0, 8>([&](auto uc) __attribute__((always_inline)) {
                    constexpr int uu = decltype(uc)::value, L = IS_W ? 32 : 8;
                    if (uu == u) {
                        if (g < 14) wait_vm<DCAP - 1 + ((uu >= 1 && uu <= DCAP) ? L : 0)>();
                        else if (DCAP_LAST < 8) wait_vm<DCAP_LAST - 1 + ((uu >= 1 && uu <= DCAP_LAST) ? L : 0)>();
                    }
                });
            }
            store16_wt(o[u], dstr, voff_st, so + (IS_W ? (u >> 1) * 4096 + (u & 1) * 1024 : u * 1024));
            if (u == 0) load_grp(g < 15 ? g + 1 : g, nxt);
        }
        const int prog = grp_prog(IS_W, g);
        if (prog) {
            wait_vm<0>();
            __builtin_amdgcn_raw_buffer_store_b32(call * 16 + prog, flr, voff_fl, (b * NR + rank) * 128, 17);
        }
    };
    #pragma nounroll
    for (int g = 0; g < 16; g += 2) {
        send_grp(g, rawA, rawB);
        send_grp(g + 1, rawB, rawA);
    }
    wait_vm<0>();
}

#define MFMA(c, a, b) "v_mfma_f32_32x32x64_f8f6f4 %[" #c "], " a ", " b ", %[" #c "] cbsz:0 blgp:1\n\t"
#define X0 "v[136:143]"
#define X1 "v[144:151]"
#define Y0 "v[152:159]"
#define Y1 "v[160:167]"
#define STR2(x) #x
#define STR(x) STR2(x)
#define RD(lo, hi, addr, off1, off2) \
    "ds_read_b128 " lo ", %[" #addr "] offset:" off1 "\n\t" \
    "ds_read_b128 " hi ", %[" #addr "] offset:" off2 "\n\t"
#define RD_X0(addr, o1, o2) RD("v[136:139]", "v[140:143]", addr, o1, o2)
#define RD_X1(addr, o1, o2) RD("v[144:147]", "v[148:151]", addr, o1, o2)
#define RD_Y0(addr, o1, o2) RD("v[152:155]", "v[156:159]", addr, o1, o2)
#define RD_Y1(addr, o1, o2) RD("v[160:163]", "v[164:167]", addr, o1, o2)
#define WAITL(n) "s_waitcnt lgkmcnt(" #n ")\n\t"
#define CHECK(FS) \
    "1:\n\t" \
    WAITL(0) \
    "v_readlane_b32 %[st], %[vf], %[sw]\n\t" \
    "s_nop 0\n\t" \
    "s_cmp_ge_u32 %[st], %[need]\n\t" \
    "s_cbranch_scc1 2f\n\t" \
    "s_sleep 1\n\t" \
    "ds_read_addtid_b32 %[vf] offset:" STR(FS) "*32\n\t" \
    "s_branch 1b\n\t" \
    "2:\n\t"
#define RELEASE(FS, NX) \
    WAITL(0) \
    "s_mov_b64 exec, %[mask]\n\t" \
    "ds_write_addtid_b32 %[vf] offset:512\n\t" \
    "s_mov_b64 exec, -1\n\t" \
    "ds_read_addtid_b32 %[vf] offset:" STR(NX) "*32\n\t"
#define STEP_FULL(SO, FS, NX) \
    CHECK(FS) \
    RD_X0(va, STR(SO), STR(SO) "+256") \
    RD_Y0(vb, STR(SO), STR(SO) "+1024") \
    RD_X1(va, STR(SO) "+8192", STR(SO) "+8448") \
    RD_Y1(vb, STR(SO) "+4096", STR(SO) "+5120") \
    WAITL(4) \
    MFMA(c00, X0, Y0) \
    WAITL(2) \
    MFMA(c10, X1, Y0) \
    RD_Y0(vb, STR(SO) "+8192", STR(SO) "+9216") \
    WAITL(2) \
    MFMA(c01, X0, Y1) \
    MFMA(c11, X1, Y1) \
    RD_Y1(vb, STR(SO) "+12288", STR(SO) "+13312") \
    WAITL(2) \
    MFMA(c02, X0, Y0) \
    MFMA(c12, X1, Y0) \
    RELEASE(FS, NX) \
    MFMA(c03, X0, Y1) \
    "s_add_u32 %[need], %[need], 1\n\t" \
    MFMA(c13, X1, Y1)
#define STEP_HALF \
    "s_mov_b32 %[keep], m0\n\t" \
    "s_mov_b32 m0, 0\n\t" \
    CHECK(%[fs]) \
    RD_X0(va, "%[so]", "%[so]+256") \
    RD_Y0(vb, "%[so]", "%[so]+1024") \
    RD_Y1(vb, "%[so]+4096", "%[so]+5120") \
    RD_X1(vb, "%[so]+8192", "%[so]+9216") \
    WAITL(4) \
    MFMA(d0, X0, Y0) \
    RD_Y0(vb, "%[so]+12288", "%[so]+13312") \
    WAITL(4) \
    MFMA(d1, X0, Y1) \
    WAITL(2) \
    MFMA(d2, X0, X1) \
    RELEASE(%[fs], %[nx]) \
    MFMA(d3, X0, Y0) \
    WAITL(0) \
    "s_mov_b32 m0, %[keep]\n\t"
#define FLIP \
    "v_xor_b32 %[va], 0x10000, %[va]\n\t" \
    "v_xor_b32 %[vb], 0x10000, %[vb]\n\t"
#define FRAG_CLOBBERS \
    "v136", "v137", "v138", "v139", "v140", "v141", "v142", "v143", "v144", "v145", "v146", "v147", \
    "v148", "v149", "v150", "v151", "v152", "v153", "v154", "v155", "v156", "v157", "v158", "v159", \
    "v160", "v161", "v162", "v163", "v164", "v165", "v166", "v167"

__global__ void __launch_bounds__(768, 3) fused_ag_gemm(Params P) {
    __shared__ __attribute__((aligned(1024))) unsigned char lds[L_TOTAL];
    LAS unsigned char* L8 = (LAS unsigned char*)lds;
    const int wave = __builtin_amdgcn_readfirstlane(threadIdx.x >> 6);
    const int lane = threadIdx.x & 63;
    const int wg = blockIdx.x;
    const int e = wg & 7, idx = wg >> 3;
    const int ti = (e >> 1) + 4 * (idx >> 3);
    const int tj = (e & 1) + 2 * (idx & 7);
    char* const myws = P.myws;
    const int rank = P.rank;

    if (wave < 8) {
        if (wave == 0) {
            #pragma unroll
            for (int i = 0; i < L_CTRL_BYTES / 256; ++i) ((LAS unsigned*)L8)[i * 64 + lane] = 0u;
            asm volatile("s_waitcnt lgkmcnt(0)" ::: "memory");
            TSTAMP(0);
            CSTAMP(9);
        }
        __builtin_amdgcn_s_barrier();
        if (wave >= 4) __builtin_amdgcn_s_setprio(1);
        const int wm = wave >> 1, wn = wave & 1;
        const int r = lane & 31, h = lane >> 5;
        unsigned va = L_RING + wm * 2048 + (r >> 4) * 1024 + h * 512 + (r & 15) * 16;
        unsigned vb = L_RING + 16384 + h * 2048 + wn * 512 + r * 16;
        f32x16 c00 = {}, c01 = {}, c02 = {}, c03 = {}, c10 = {}, c11 = {}, c12 = {}, c13 = {};
        unsigned vf, st, need, keep;
        const unsigned long long mask = 1ull << wave;
        float* outp = P.out + (size_t)(ti * 256 + wm * 32) * DIM + tj * 256 + wn * 128;
        asm volatile("" : "+s"(outp));
        asm volatile(
            "s_mov_b32 %[keep], m0\n\t"
            "s_mov_b32 m0, 0\n\t"
            "s_mov_b32 %[need], 1\n\t"
            "ds_read_addtid_b32 %[vf] offset:0\n\t"
            "3:\n\t"
            STEP_FULL(0, 0, 1)
            STEP_FULL(32768, 1, 2)
            FLIP
            STEP_FULL(0, 2, 3)
            STEP_FULL(32768, 3, 0)
            FLIP
            "s_cmp_le_u32 %[need], %[nfull]\n\t"
            "s_cbranch_scc1 3b\n\t"
            WAITL(0)
            "ds_read_addtid_b32 %[vf] offset:4*32\n\t"
            WAITL(0)
            "s_mov_b32 m0, %[keep]\n\t"
            : [c00] "+v"(c00), [c01] "+v"(c01), [c02] "+v"(c02), [c03] "+v"(c03),
              [c10] "+v"(c10), [c11] "+v"(c11), [c12] "+v"(c12), [c13] "+v"(c13),
              [va] "+v"(va), [vb] "+v"(vb), [vf] "=&v"(vf), [st] "=&s"(st), [need] "=&s"(need), [keep] "=&s"(keep)
            : [sw] "s"(wave), [mask] "s"(mask), [nfull] "i"(NR * NF)
            : "memory", "scc", FRAG_CLOBBERS);
        if (wave == 0) TSTAMP(8);
        vb -= 8192;
        auto half_step = [&](auto jc, f32x16& d0, f32x16& d1, f32x16& d2, f32x16& d3) __attribute__((always_inline)) {
            constexpr int j = decltype(jc)::value, k = j % 6;
            asm volatile(STEP_HALF
                : [d0] "+v"(d0), [d1] "+v"(d1), [d2] "+v"(d2), [d3] "+v"(d3), [vf] "+v"(vf), [st] "=&s"(st), [keep] "=&s"(keep)
                : [va] "v"(va), [vb] "v"(vb), [sw] "s"(wave), [mask] "s"(mask), [need] "i"(NR * NF + j + 1),
                  [so] "i"((k % 3) * 24576), [fs] "i"(4 + k), [nx] "i"(4 + (k + 1) % 6)
                : "memory", "scc", FRAG_CLOBBERS);
            if (k == 2) { va += 73728; vb += 73728; asm volatile("" : "+v"(va), "+v"(vb)); }
            if (k == 5) { va -= 73728; vb -= 73728; asm volatile("" : "+v"(va), "+v"(vb)); }
        };
        static_for<0, NR * NS>([&](auto jc) __attribute__((always_inline)) { half_step(jc, c00, c01, c02, c03); });
        asm volatile("s_nop 15\n\ts_nop 7" : "+v"(c00), "+v"(c01), "+v"(c02), "+v"(c03));
        if (wave == 0) { TSTAMP(7); CSTAMP(11); }
        const int lz = __builtin_amdgcn_mbcnt_hi(~0u, __builtin_amdgcn_mbcnt_lo(~0u, 0u));
        const __amdgpu_buffer_rsrc_t outr = make_rsrc(outp, 160u * DIM * 4u);
        const int voff = ((lz >> 5) * 4 * DIM + (lz & 31) * 4) * 4;
        const float s_lo = __builtin_bit_cast(float, __builtin_amdgcn_readfirstlane(*(volatile LAS unsigned*)(L8 + L_SCALE)));
        auto act = [](float a, float s) { return __builtin_bit_cast(unsigned, __builtin_elementwise_maximum(a * s, 0.f)); };
        static_for<0, NR * NS>([&](auto ic) __attribute__((always_inline)) {
            constexpr int i = decltype(ic)::value;
            #pragma unroll
            for (int g = i * EPS; g < (i + 1) * EPS; ++g)
                if (g < 16) {
                    u32x4 v = {act(c00[g], s_lo), act(c01[g], s_lo), act(c02[g], s_lo), act(c03[g], s_lo)};
                    store16_wt(v, outr, voff, ((g & 3) + 8 * (g >> 2)) * DIM * 4);
                }
            half_step(std::integral_constant<int, NR * NS + i>{}, c10, c11, c12, c13);
        });
        asm volatile("s_nop 15\n\ts_nop 7" : "+v"(c10), "+v"(c11), "+v"(c12), "+v"(c13));
        if (wave == 0) { TSTAMP(4); CSTAMP(10); }
        const float s_hi = __builtin_bit_cast(float, __builtin_amdgcn_readfirstlane(*(volatile LAS unsigned*)(L8 + L_SCALE)));
        #pragma unroll
        for (int g = NR * NS * EPS; g < 16; ++g) {
            u32x4 v = {act(c00[g], s_hi), act(c01[g], s_hi), act(c02[g], s_hi), act(c03[g], s_hi)};
            store16_wt(v, outr, voff, ((g & 3) + 8 * (g >> 2)) * DIM * 4);
        }
        #pragma unroll
        for (int g = 0; g < 16; ++g) {
            u32x4 v = {act(c10[g], s_hi), act(c11[g], s_hi), act(c12[g], s_hi), act(c13[g], s_hi)};
            store16_wt(v, outr, voff, (128 + (g & 3) + 8 * (g >> 2)) * DIM * 4);
        }
    } else if (wave == 8) {
        __builtin_amdgcn_s_setprio(3);
        unsigned* ctr = (unsigned*)(myws + CTR_OFF + wg * 128);
        char* const dws = P.dws[e];
        unsigned* dbg = (unsigned*)(myws + DBG_OFF + wg * 1024 + 128);
        const unsigned* tsched = SEND_SCHED.t[(idx & 16) | ((idx + P.rot[e]) & 15)];
        if (idx < 16) push_loop<false>(P.x, dws, rank, idx, e != 0, tsched, ctr, lane, dbg);
        else push_loop<true>(P.w, dws, rank, idx - 16, e != 0, tsched, ctr, lane, dbg);
        TSTAMP(1);
    } else if (wave == 9) {
        __builtin_amdgcn_s_setprio(3);
        unsigned* ctr = (unsigned*)(myws + CTR_OFF + wg * 128 + 64);
        unsigned* tmo = (unsigned*)(myws + TMO_OFF);
        const unsigned prev = __hip_atomic_load(ctr, __ATOMIC_RELAXED, __HIP_MEMORY_SCOPE_AGENT);
        const float scale = P.sx[0] * P.sw[0];
        __builtin_amdgcn_s_barrier();
        const unsigned call = __builtin_amdgcn_readfirstlane(prev) + 1;
        __hip_atomic_store(ctr, call, __ATOMIC_RELAXED, __HIP_MEMORY_SCOPE_AGENT);
        const int p = (rank + (lane >> 1)) & 7, o = lane & 1;
        const unsigned* fl = lane < 16 ? (const unsigned*)(myws + FLAG_OFF + (size_t)(((o ? 16 + tj : ti) * NR + p) * 128)) : tmo;
        volatile LAS unsigned* sched = (volatile LAS unsigned*)(L8 + L_SCHED);
        volatile LAS float* lscale = (volatile LAS float*)(L8 + L_SCALE);
        *lscale = scale;
        const unsigned base = call * 16;
        const unsigned code0 = ((call & 1) * 64 + p * 8) + 1;
        int cur = 0, nsched = 0, limit = NF;
        bool dead = false;
        const unsigned long long t0 = __builtin_amdgcn_s_memrealtime();
        for (;;) {
            int av = 15;
            if (!dead) {
                unsigned val = 0;
                if (lane <= 16) val = __hip_atomic_load(fl, __ATOMIC_RELAXED, __HIP_MEMORY_SCOPE_SYSTEM);
                if (__builtin_amdgcn_readlane(val, 16) == call || __builtin_amdgcn_s_memrealtime() - t0 > DEADLINE_TICKS) {
                    __hip_atomic_store(tmo, call, __ATOMIC_RELAXED, __HIP_MEMORY_SCOPE_AGENT);
                    *lscale = __builtin_nanf("");
                    dead = true;
                } else {
                    av = (int)(val - base);
                    if (o && av >= NCH) av = 15;
                    const int other = __builtin_amdgcn_update_dpp(0, av, 0xB1, 0xf, 0xf, false);
                    av = av < other ? av : other;
                }
            }
            for (;;) {
                const bool rdy = lane < 16 && av > cur && cur < limit;
                const unsigned long long m = __ballot(rdy) & 0x5555ull;
                if (m == 0) break;
                if (rdy && o == 0) sched[nsched + __builtin_popcountll(m & ((1ull << lane) - 1))] = code0 + (cur < NCH ? cur : cur - NS);
                cur += rdy ? 1 : 0;
                nsched += __builtin_popcountll(m);
                if (nsched == NR * limit) limit = limit == NF ? NCH : NCH + NS;
            }
            if (nsched >= NSTEP) break;
            __builtin_amdgcn_s_sleep(1);
        }
        TSTAMP(2);
    } else {
        __builtin_amdgcn_s_setprio(3);
        __builtin_amdgcn_s_barrier();
        const int o = wave - 10;
        volatile LAS unsigned* sched = (volatile LAS unsigned*)(L8 + L_SCHED);
        volatile LAS unsigned* freew = (volatile LAS unsigned*)(L8 + L_FREE);
        volatile LAS unsigned* fullw = (volatile LAS unsigned*)(L8 + L_FULL);
        const char* stage_x = myws + (size_t)ti * ITEM;
        const char* stage_w = myws + (size_t)(16 + tj) * ITEM;
        const unsigned v0 = lane * 16, v1 = v0 + 4096;
        auto dma8 = [&](const char* src, unsigned ldst) __attribute__((always_inline)) {
            unsigned keep;
#define DMA4(V) "global_load_lds_dwordx4 " V ", %[src] sc0 sc1\n\t" \
                "global_load_lds_dwordx4 " V ", %[src] offset:1024 sc0 sc1\n\t" \
                "global_load_lds_dwordx4 " V ", %[src] offset:2048 sc0 sc1\n\t" \
                "global_load_lds_dwordx4 " V ", %[src] offset:3072 sc0 sc1\n\t"
            asm volatile(
                "s_mov_b32 %[keep], m0\n\t"
                "s_mov_b32 m0, %[ld]\n\t"
                "s_nop 0\n\t"
                DMA4("%[v0]")
                "s_add_u32 m0, m0, 4096\n\t"
                "s_nop 0\n\t"
                DMA4("%[v1]")
                "s_mov_b32 m0, %[keep]\n\t"
                : [keep] "=&s"(keep) : [src] "s"(src), [ld] "s"(ldst), [v0] "v"(v0), [v1] "v"(v1) : "memory", "scc");
        };
        auto slot_id = [](int s_) { return s_ < NR * NF ? (s_ & 3) : 4 + (s_ - NR * NF) % 6; };
        int issued = o, pub = o;
        while (pub < NSTEP) {
            bool did = false;
            if (issued < NSTEP && issued - pub < (pub >= NR * NF ? 6 : 4)) {
                const unsigned codev = sched[issued];
                const int fr = (int)freew[lane & 7];
                const unsigned code = __builtin_amdgcn_readfirstlane(codev);
                const int thr = issued < NR * NF ? issued - 3 : (issued - 5 > NR * NF ? issued - 5 : NR * NF);
                if (code != 0u && __all(fr >= thr)) {
                    const size_t off = (size_t)((code - 1) & 127u) * (NBLK * ITEM);
                    const char* sx_ = stage_x + off;
                    const char* sw_ = stage_w + off;
                    asm volatile("s_nop 4" : "+s"(sx_), "+s"(sw_));
                    if (issued < NR * NF) {
                        const unsigned ldst = L_RING + (issued & 3) * 32768;
                        dma8(sx_, ldst); dma8(sw_, ldst + 16384); dma8(sx_ + 8192, ldst + 8192); dma8(sw_ + 8192, ldst + 24576);
                    } else {
                        const unsigned ldst = L_RING + (slot_id(issued) - 4) * 24576;
                        dma8(sx_ + (issued >= NR * NCH ? 8192 : 0), ldst); dma8(sw_, ldst + 8192); dma8(sw_ + 8192, ldst + 16384);
                    }
                    issued += 2;
                    did = true;
                }
            }
            if (!did || issued - pub >= (pub >= NR * NF ? 6 : 4)) {
                if (issued > pub) {
                    if (issued - pub == 2) wait_vm<0>();
                    else if (pub + 2 < NR * NF) wait_vm<32>();
                    else if (issued - pub == 4) wait_vm<24>();
                    else wait_vm<48>();
                    fullw[slot_id(pub) * 8 + (lane & 7)] = pub + 1;
                    pub += 2;
                } else {
                    __builtin_amdgcn_s_sleep(1);
                }
            }
        }
        if (o == 0) TSTAMP(3);
    }
}

}

extern "C" void kernel_launch(int rank, int nranks, ncclComm_t comm,
                              hipStream_t stream,
                              void* const* d_in, const int* in_sizes, int n_in,
                              void* d_out, int out_size,
                              const int* in_dtypes, int out_dtype,
                              void* const* d_out_all,
                              void* const* d_ws_all, size_t ws_bytes) {
    if (nranks != NR || n_in < 4 || ws_bytes < WS_NEED || (unsigned)rank >= (unsigned)NR ||
        in_sizes[0] != DIM * KLOC || in_sizes[1] != DIM * KLOC || out_size != DIM * DIM) {
        fprintf(stderr, "kernel_launch: unexpected problem shape\n");
        return;
    }
    Params P;
    P.x = (const unsigned char*)d_in[0];
    P.w = (const unsigned char*)d_in[1];
    P.sx = (const float*)d_in[2];
    P.sw = (const float*)d_in[3];
    P.out = (float*)d_out;
    P.myws = (char*)d_ws_all[rank];
    {
        const int quad[NR] = {0, 1, 0, 0, 1, 1, 0, 1};
        int n = 0;
        for (int pass = 0; pass < 2; ++pass)
            for (int q = 0; q < NR; ++q) {
                const int p = (rank + q) % NR;
                if ((quad[p] == quad[rank]) == (pass == 0)) { P.dws[n] = (char*)d_ws_all[p]; P.rot[n] = q * ROT / 7; ++n; }
            }
    }
    P.rank = rank;
    hipLaunchKernelGGL(fused_ag_gemm, dim3(256), dim3(768), 0, stream, P);
    HIP_CHECK(hipGetLastError());
}
```

```cpp
#include <hip/hip_runtime.h>
#include <rccl/rccl.h>
#include <cstdio>
#include <cstdint>
#include <type_traits>

#define HIP_CHECK(expr) do { hipError_t _e = (expr); if (_e != hipSuccess) { \
    fprintf(stderr, "HIP error %s at %s:%d\n", hipGetErrorString(_e), __FILE__, __LINE__); \
    return; } } while (0)

#define TAU16 444
#define TOFF 100
#define NSPLIT 0
#define S_FULL 31
#define S_LAST 7
#define S_LOW 23
#define S_LOX 31
#define S_HI 15
#define DCAP 2
#define DCAP_LAST 8
#define ROT 0
#define EPS 1

namespace {

typedef unsigned u32x4 __attribute__((ext_vector_type(4)));
typedef float f32x16 __attribute__((ext_vector_type(16)));
#define LAS __attribute__((address_space(3)))

constexpr int NR = 8;
constexpr int DIM = 4096;
constexpr int KLOC = DIM / NR;
constexpr int NCH = 8;
constexpr int NBLK = 32;
constexpr int NS = NSPLIT, NF = NCH - NS;
constexpr int NSTEP = NR * (NCH + NS);
constexpr unsigned ITEM = 16384;
constexpr size_t STAGE_PAR = (size_t)NR * NCH * NBLK * ITEM;
constexpr size_t FLAG_OFF = 2 * STAGE_PAR;
constexpr size_t CTR_OFF = FLAG_OFF + 32768;
constexpr size_t TMO_OFF = CTR_OFF + 32768;
constexpr size_t DBG_OFF = TMO_OFF + 4096;
constexpr size_t WS_NEED = DBG_OFF + 256 * 1024;
constexpr unsigned long long DEADLINE_TICKS = 50000000ull;
static_assert(NS >= 0 && NS <= 4, "");

constexpr int L_FULL = 0;
constexpr int L_FREE = 512;
constexpr int L_SCHED = 1024;
constexpr int L_SCALE = 1536;
constexpr int L_CTRL_BYTES = 2048;
constexpr int L_RING = 8192;
constexpr int L_TOTAL = L_RING + 6 * 24576;

struct Params {
    const unsigned char* x;
    const unsigned char* w;
    const float* sx;
    const float* sw;
    float* out;
    char* myws;
    char* dws[NR];
    int rank;
    int rot[NR];
};

__host__ __device__ constexpr int grp_chunk(bool is_w, int g) { return (is_w || g < 2 * NF) ? g >> 1 : (g < 2 * NF + NS ? g - NF : g - NF - NS); }
__host__ __device__ constexpr int grp_half(bool is_w, int g) { return (is_w || g < 2 * NF) ? g & 1 : (g < 2 * NF + NS ? 0 : 1); }
__host__ __device__ constexpr int grp_prog(bool is_w, int g) { return (is_w || g < 2 * NF) ? ((g & 1) ? (g >> 1) + 1 : 0) : g - NF + 1; }

struct Sched { unsigned t[NBLK][16]; };
constexpr int store_key(int b, int g, int u) {
    const bool is_w = b >= 16;
    const int v = b & 15, c = grp_chunk(is_w, g), h = grp_half(is_w, g);
    int end = 0, stride = 0, back = 0;
    if (c < NF) { end = (c * 32 + 2 * v + (is_w ? 2 : 1)) * 16; stride = c == 7 ? S_LAST : S_FULL; back = h ? 0 : 8; }
    else if (is_w) { end = NF * 512 + (c - NF) * 384 + v * 24 + 16; stride = S_LOW; back = h ? 0 : 8; }
    else if (h == 0) { end = NF * 512 + (c - NF) * 384 + v * 24 + 24; stride = S_LOX; back = 0; }
    else { end = NF * 512 + NS * 384 + (c - NF) * 128 + v * 8 + 8; stride = S_HI; back = 0; }
    return end - (7 - u + back) * stride + 1024;
}
constexpr Sched make_sched() {
    Sched r = {};
    unsigned short cnt[5184] = {};
    for (int b = 0; b < NBLK; ++b) for (int g = 0; g < 16; ++g) for (int u = 0; u < 8; ++u) ++cnt[store_key(b, g, u) + 1];
    for (int k = 1; k < 5184; ++k) cnt[k] += cnt[k - 1];
    for (int b = 0; b < NBLK; ++b)
        for (int g = 0; g < 16; ++g) {
            unsigned first = 0, last = 0;
            for (int u = 0; u < 8; ++u) { last = cnt[store_key(b, g, u)]++; if (u == 0) first = last; }
            r.t[b][g] = (TOFF * 4 + first * TAU16 / 64) | ((TOFF * 4 + last * TAU16 / 64) << 16);
        }
    return r;
}
__device__ constexpr Sched SEND_SCHED = make_sched();

__device__ __forceinline__ __amdgpu_buffer_rsrc_t make_rsrc(const void* p, unsigned bytes) {
    return __builtin_amdgcn_make_buffer_rsrc((void*)p, 0, (int)bytes, 0x00020000);
}

__device__ __forceinline__ void store16_wt(u32x4 v, __amdgpu_buffer_rsrc_t r, int voff, int soff) {
    __builtin_amdgcn_raw_buffer_store_b128(v, r, voff, soff, 17);
    __builtin_amdgcn_sched_barrier(0);
    asm volatile("s_nop 1" ::: "memory");
    __builtin_amdgcn_sched_barrier(0);
}

template <int I, int N, typename F> __device__ __forceinline__ void static_for(F&& f) {
    if constexpr (I < N) { f(std::integral_constant<int, I>{}); static_for<I + 1, N>(f); }
}

template <int N> __device__ __forceinline__ void wait_vm() {
    static_assert(N >= 0 && N <= 63, "");
    asm volatile("s_waitcnt vmcnt(%0)" :: "i"(N) : "memory");
}

#define CSTAMP(idx) do {} while (0)
#define TSTAMP(idx) do {} while (0)

template <bool IS_W>
__device__ __forceinline__ void push_loop(const Params& P, const unsigned char* srcp, int rank, int blk,
                                          unsigned* ctr, unsigned* claims, int lane, unsigned* dbg) {
    const unsigned long long t0 = __builtin_amdgcn_s_memrealtime();
    const __amdgpu_buffer_rsrc_t src = make_rsrc(srcp, DIM * KLOC);
    const int voff_ld = IS_W ? lane * 4 : (lane & 15) * KLOC + (lane >> 4) * 16;
    unsigned rawA[32], rawB[32];
    auto load_grp = [&](int g, unsigned (&raw)[32]) __attribute__((always_inline)) {
        const int c = grp_chunk(IS_W, g), h = grp_half(IS_W, g);
        if constexpr (IS_W) {
            const int so = (c * 64 + h * 32) * DIM + blk * 256;
            #pragma unroll
            for (int k = 0; k < 32; ++k) raw[k] = __builtin_amdgcn_raw_buffer_load_b32(src, voff_ld, so + k * DIM, 0);
        } else {
            const int so = (blk * 256 + h * 128) * KLOC + c * 64;
            #pragma unroll
            for (int t = 0; t < 8; ++t) {
                u32x4 v = __builtin_amdgcn_raw_buffer_load_b128(src, voff_ld, so + t * 16 * KLOC, 0);
                raw[4 * t] = v.x; raw[4 * t + 1] = v.y; raw[4 * t + 2] = v.z; raw[4 * t + 3] = v.w;
            }
        }
    };
    auto claim_add = [&](int sl, unsigned d) __attribute__((always_inline)) {
        unsigned old;
        asm volatile("s_mov_b64 exec, 1\n\ts_nop 4\n\tglobal_atomic_add %0, %1, %2, %3 sc0\n\ts_mov_b64 exec, -1"
                     : "=&v"(old) : "v"(sl * 128), "v"(d), "s"(claims) : "memory");
        return old;
    };
    auto claim_wait = [&](unsigned& v) __attribute__((always_inline)) {
        asm volatile("s_waitcnt vmcnt(0)" : "+v"(v) :: "memory");
        return (unsigned)__builtin_amdgcn_readfirstlane(v);
    };
    unsigned xcc;
    asm volatile("s_getreg_b32 %0, hwreg(HW_REG_XCC_ID)" : "=s"(xcc));
    int e = (int)(xcc & 7u);
    const unsigned prev = __hip_atomic_load(ctr, __ATOMIC_RELAXED, __HIP_MEMORY_SCOPE_AGENT);
    unsigned got = claim_add(e, 1u);
    load_grp(0, rawA);
    char* dws = P.dws[e];
    const unsigned* tsched = SEND_SCHED.t[(IS_W ? 16 : 0) | ((blk + P.rot[e]) & 15)];
    { unsigned warm = tsched[0]; asm volatile("" :: "s"(warm), "s"(dws)); }
    __builtin_amdgcn_s_barrier();
    unsigned gs = claim_wait(got);
    const unsigned call = __builtin_amdgcn_readfirstlane(prev) + 1;
    #pragma nounroll
    for (int tries = 0; tries < 7 && gs != call - 1; ++tries) {
        got = claim_add(e, ~0u);
        claim_wait(got);
        e = (e + 1) & 7;
        got = claim_add(e, 1u);
        gs = claim_wait(got);
        dws = P.dws[e];
    }
    __hip_atomic_store(ctr, call, __ATOMIC_RELAXED, __HIP_MEMORY_SCOPE_AGENT);
    const bool paced = e != 0;
    const __amdgpu_buffer_rsrc_t dstr = make_rsrc(dws + (size_t)(call & 1) * STAGE_PAR + (size_t)rank * (NCH * NBLK * ITEM), NCH * NBLK * ITEM);
    const __amdgpu_buffer_rsrc_t flr = make_rsrc(dws + FLAG_OFF, 32768);
    const int voff_st = lane * 16;
    const int voff_fl = lane < 8 ? lane * 4 : 0x7ffffff0;
    const int b = blk + (IS_W ? 16 : 0);
    auto send_grp = [&](int g, unsigned (&raw)[32], unsigned (&nxt)[32]) __attribute__((always_inline)) {
        u32x4 o[8];
        if constexpr (IS_W) {
            #pragma unroll
            for (int r = 0; r < 8; ++r) {
                const unsigned t0_ = __builtin_amdgcn_perm(raw[4 * r + 1], raw[4 * r], 0x05010400u);
                const unsigned t1_ = __builtin_amdgcn_perm(raw[4 * r + 1], raw[4 * r], 0x07030602u);
                const unsigned t2_ = __builtin_amdgcn_perm(raw[4 * r + 3], raw[4 * r + 2], 0x05010400u);
                const unsigned t3_ = __builtin_amdgcn_perm(raw[4 * r + 3], raw[4 * r + 2], 0x07030602u);
                o[0 + (r >> 2)][r & 3] = __builtin_amdgcn_perm(t2_, t0_, 0x05040100u);
                o[2 + (r >> 2)][r & 3] = __builtin_amdgcn_perm(t2_, t0_, 0x07060302u);
                o[4 + (r >> 2)][r & 3] = __builtin_amdgcn_perm(t3_, t1_, 0x05040100u);
                o[6 + (r >> 2)][r & 3] = __builtin_amdgcn_perm(t3_, t1_, 0x07060302u);
            }
        } else {
            #pragma unroll
            for (int t = 0; t < 8; ++t) o[t] = u32x4{raw[4 * t], raw[4 * t + 1], raw[4 * t + 2], raw[4 * t + 3]};
        }
        #pragma unroll
        for (int u = 0; u < 8; ++u) asm volatile("" : "+v"(o[u]));
        const int c = grp_chunk(IS_W, g), h = grp_half(IS_W, g);
        const int so = (c * NBLK + b) * ITEM + h * (IS_W ? 2048 : 8192);
        const unsigned ts = tsched[g];
        const unsigned ta = ts & 0xffffu, tz = ts >> 16;
        #pragma unroll
        for (int u = 0; u < 8; ++u) {
            if (paced) {
                const unsigned long long tgt = t0 + ((ta * 7 + (tz - ta) * u) / 28);
                for (;;) {
                    const long long left = (long long)(tgt - __builtin_amdgcn_s_memrealtime());
                    if (left <= 0) break;
                    if (left > 40) __builtin_amdgcn_s_sleep(8); else __builtin_amdgcn_s_sleep(1);
                }
            }
            if (paced) {
                static_for<0, 8>([&](auto uc) __attribute__((always_inline)) {
                    constexpr int uu = decltype(uc)::value, L = IS_W ? 32 : 8;
                    if (uu == u) {
                        if (g < 14) wait_vm<DCAP - 1 + ((uu >= 1 && uu <= DCAP) ? L : 0)>();
                        else if (DCAP_LAST < 8) wait_vm<DCAP_LAST - 1 + ((uu >= 1 && uu <= DCAP_LAST) ? L : 0)>();
                    }
                });
            }
            store16_wt(o[u], dstr, voff_st, so + (IS_W ? (u >> 1) * 4096 + (u & 1) * 1024 : u * 1024));
            if (u == 0) load_grp(g < 15 ? g + 1 : g, nxt);
        }
        const int prog = grp_prog(IS_W, g);
        if (prog) {
            wait_vm<0>();
            __builtin_amdgcn_raw_buffer_store_b32(call * 16 + prog, flr, voff_fl, (b * NR + rank) * 128, 17);
        }
    };
    #pragma nounroll
    for (int g = 0; g < 16; g += 2) {
        send_grp(g, rawA, rawB);
        send_grp(g + 1, rawB, rawA);
    }
    wait_vm<0>();
}

#define MFMA(c, a, b) "v_mfma_f32_32x32x64_f8f6f4 %[" #c "], " a ", " b ", %[" #c "] cbsz:0 blgp:1\n\t"
#define X0 "v[136:143]"
#define X1 "v[144:151]"
#define Y0 "v[152:159]"
#define Y1 "v[160:167]"
#define STR2(x) #x
#define STR(x) STR2(x)
#define RD(lo, hi, addr, off1, off2) \
    "ds_read_b128 " lo ", %[" #addr "] offset:" off1 "\n\t" \
    "ds_read_b128 " hi ", %[" #addr "] offset:" off2 "\n\t"
#define RD_X0(addr, o1, o2) RD("v[136:139]", "v[140:143]", addr, o1, o2)
#define RD_X1(addr, o1, o2) RD("v[144:147]", "v[148:151]", addr, o1, o2)
#define RD_Y0(addr, o1, o2) RD("v[152:155]", "v[156:159]", addr, o1, o2)
#define RD_Y1(addr, o1, o2) RD("v[160:163]", "v[164:167]", addr, o1, o2)
#define WAITL(n) "s_waitcnt lgkmcnt(" #n ")\n\t"
#define CHECK(FS) \
    "1:\n\t" \
    WAITL(0) \
    "v_readlane_b32 %[st], %[vf], %[sw]\n\t" \
    "s_nop 0\n\t" \
    "s_cmp_ge_u32 %[st], %[need]\n\t" \
    "s_cbranch_scc1 2f\n\t" \
    "s_sleep 1\n\t" \
    "ds_read_addtid_b32 %[vf] offset:" STR(FS) "*32\n\t" \
    "s_branch 1b\n\t" \
    "2:\n\t"
#define RELEASE(FS, NX) \
    WAITL(0) \
    "s_mov_b64 exec, %[mask]\n\t" \
    "ds_write_addtid_b32 %[vf] offset:512\n\t" \
    "s_mov_b64 exec, -1\n\t" \
    "ds_read_addtid_b32 %[vf] offset:" STR(NX) "*32\n\t"
#define STEP_FULL(SO, FS, NX) \
    CHECK(FS) \
    RD_X0(va, STR(SO), STR(SO) "+256") \
    RD_Y0(vb, STR(SO), STR(SO) "+1024") \
    RD_X1(va, STR(SO) "+8192", STR(SO) "+8448") \
    RD_Y1(vb, STR(SO) "+4096", STR(SO) "+5120") \
    WAITL(4) \
    MFMA(c00, X0, Y0) \
    WAITL(2) \
    MFMA(c10, X1, Y0) \
    RD_Y0(vb, STR(SO) "+8192", STR(SO) "+9216") \
    WAITL(2) \
    MFMA(c01, X0, Y1) \
    MFMA(c11, X1, Y1) \
    RD_Y1(vb, STR(SO) "+12288", STR(SO) "+13312") \
    WAITL(2) \
    MFMA(c02, X0, Y0) \
    MFMA(c12, X1, Y0) \
    RELEASE(FS, NX) \
    MFMA(c03, X0, Y1) \
    "s_add_u32 %[need], %[need], 1\n\t" \
    MFMA(c13, X1, Y1)
#define STEP_HALF \
    "s_mov_b32 %[keep], m0\n\t" \
    "s_mov_b32 m0, 0\n\t" \
    CHECK(%[fs]) \
    RD_X0(va, "%[so]", "%[so]+256") \
    RD_Y0(vb, "%[so]", "%[so]+1024") \
    RD_Y1(vb, "%[so]+4096", "%[so]+5120") \
    RD_X1(vb, "%[so]+8192", "%[so]+9216") \
    WAITL(4) \
    MFMA(d0, X0, Y0) \
    RD_Y0(vb, "%[so]+12288", "%[so]+13312") \
    WAITL(4) \
    MFMA(d1, X0, Y1) \
    WAITL(2) \
    MFMA(d2, X0, X1) \
    RELEASE(%[fs], %[nx]) \
    MFMA(d3, X0, Y0) \
    WAITL(0) \
    "s_mov_b32 m0, %[keep]\n\t"
#define FLIP \
    "v_xor_b32 %[va], 0x10000, %[va]\n\t" \
    "v_xor_b32 %[vb], 0x10000, %[vb]\n\t"
#define FRAG_CLOBBERS \
    "v136", "v137", "v138", "v139", "v140", "v141", "v142", "v143", "v144", "v145", "v146", "v147", \
    "v148", "v149", "v150", "v151", "v152", "v153", "v154", "v155", "v156", "v157", "v158", "v159", \
    "v160", "v161", "v162", "v163", "v164", "v165", "v166", "v167"

__global__ void __launch_bounds__(768, 3) fused_ag_gemm(Params P) {
    __shared__ __attribute__((aligned(1024))) unsigned char lds[L_TOTAL];
    LAS unsigned char* L8 = (LAS unsigned char*)lds;
    const int wave = __builtin_amdgcn_readfirstlane(threadIdx.x >> 6);
    const int lane = threadIdx.x & 63;
    const int wg = blockIdx.x;
    const int e = wg & 7, idx = wg >> 3;
    const int ti = (e >> 1) + 4 * (idx >> 3);
    const int tj = (e & 1) + 2 * (idx & 7);
    char* const myws = P.myws;
    const int rank = P.rank;

    if (wave < 8) {
        if (wave == 0) {
            #pragma unroll
            for (int i = 0; i < L_CTRL_BYTES / 256; ++i) ((LAS unsigned*)L8)[i * 64 + lane] = 0u;
            asm volatile("s_waitcnt lgkmcnt(0)" ::: "memory");
            TSTAMP(0);
            CSTAMP(9);
        }
        __builtin_amdgcn_s_barrier();
        if (wave >= 4) __builtin_amdgcn_s_setprio(1);
        const int wm = wave >> 1, wn = wave & 1;
        const int r = lane & 31, h = lane >> 5;
        unsigned va = L_RING + wm * 2048 + (r >> 4) * 1024 + h * 512 + (r & 15) * 16;
        unsigned vb = L_RING + 16384 + h * 2048 + wn * 512 + r * 16;
        f32x16 c00 = {}, c01 = {}, c02 = {}, c03 = {}, c10 = {}, c11 = {}, c12 = {}, c13 = {};
        unsigned vf, st, need, keep;
        const unsigned long long mask = 1ull << wave;
        float* outp = P.out + (size_t)(ti * 256 + wm * 32) * DIM + tj * 256 + wn * 128;
        asm volatile("" : "+s"(outp));
        asm volatile(
            "s_mov_b32 %[keep], m0\n\t"
            "s_mov_b32 m0, 0\n\t"
            "s_mov_b32 %[need], 1\n\t"
            "ds_read_addtid_b32 %[vf] offset:0\n\t"
            "3:\n\t"
            STEP_FULL(0, 0, 1)
            STEP_FULL(32768, 1, 2)
            FLIP
            STEP_FULL(0, 2, 3)
            STEP_FULL(32768, 3, 0)
            FLIP
            "s_cmp_le_u32 %[need], %[nfull]\n\t"
            "s_cbranch_scc1 3b\n\t"
            WAITL(0)
            "ds_read_addtid_b32 %[vf] offset:4*32\n\t"
            WAITL(0)
            "s_mov_b32 m0, %[keep]\n\t"
            : [c00] "+v"(c00), [c01] "+v"(c01), [c02] "+v"(c02), [c03] "+v"(c03),
              [c10] "+v"(c10), [c11] "+v"(c11), [c12] "+v"(c12), [c13] "+v"(c13),
              [va] "+v"(va), [vb] "+v"(vb), [vf] "=&v"(vf), [st] "=&s"(st), [need] "=&s"(need), [keep] "=&s"(keep)
            : [sw] "s"(wave), [mask] "s"(mask), [nfull] "i"(NR * NF)
            : "memory", "scc", FRAG_CLOBBERS);
        if (wave == 0) TSTAMP(8);
        vb -= 8192;
        auto half_step = [&](auto jc, f32x16& d0, f32x16& d1, f32x16& d2, f32x16& d3) __attribute__((always_inline)) {
            constexpr int j = decltype(jc)::value, k = j % 6;
            asm volatile(STEP_HALF
                : [d0] "+v"(d0), [d1] "+v"(d1), [d2] "+v"(d2), [d3] "+v"(d3), [vf] "+v"(vf), [st] "=&s"(st), [keep] "=&s"(keep)
                : [va] "v"(va), [vb] "v"(vb), [sw] "s"(wave), [mask] "s"(mask), [need] "i"(NR * NF + j + 1),
                  [so] "i"((k % 3) * 24576), [fs] "i"(4 + k), [nx] "i"(4 + (k + 1) % 6)
                : "memory", "scc", FRAG_CLOBBERS);
            if (k == 2) { va += 73728; vb += 73728; asm volatile("" : "+v"(va), "+v"(vb)); }
            if (k == 5) { va -= 73728; vb -= 73728; asm volatile("" : "+v"(va), "+v"(vb)); }
        };
        static_for<0, NR * NS>([&](auto jc) __attribute__((always_inline)) { half_step(jc, c00, c01, c02, c03); });
        asm volatile("s_nop 15\n\ts_nop 7" : "+v"(c00), "+v"(c01), "+v"(c02), "+v"(c03));
        if (wave == 0) { TSTAMP(7); CSTAMP(11); }
        const int lz = __builtin_amdgcn_mbcnt_hi(~0u, __builtin_amdgcn_mbcnt_lo(~0u, 0u));
        const __amdgpu_buffer_rsrc_t outr = make_rsrc(outp, 160u * DIM * 4u);
        const int voff = ((lz >> 5) * 4 * DIM + (lz & 31) * 4) * 4;
        const float s_lo = __builtin_bit_cast(float, __builtin_amdgcn_readfirstlane(*(volatile LAS unsigned*)(L8 + L_SCALE)));
        auto act = [](float a, float s) { return __builtin_bit_cast(unsigned, __builtin_elementwise_maximum(a * s, 0.f)); };
        static_for<0, NR * NS>([&](auto ic) __attribute__((always_inline)) {
            constexpr int i = decltype(ic)::value;
            #pragma unroll
            for (int g = i * EPS; g < (i + 1) * EPS; ++g)
                if (g < 16) {
                    u32x4 v = {act(c00[g], s_lo), act(c01[g], s_lo), act(c02[g], s_lo), act(c03[g], s_lo)};
                    store16_wt(v, outr, voff, ((g & 3) + 8 * (g >> 2)) * DIM * 4);
                }
            half_step(std::integral_constant<int, NR * NS + i>{}, c10, c11, c12, c13);
        });
        asm volatile("s_nop 15\n\ts_nop 7" : "+v"(c10), "+v"(c11), "+v"(c12), "+v"(c13));
        if (wave == 0) { TSTAMP(4); CSTAMP(10); }
        const float s_hi = __builtin_bit_cast(float, __builtin_amdgcn_readfirstlane(*(volatile LAS unsigned*)(L8 + L_SCALE)));
        #pragma unroll
        for (int g = NR * NS * EPS; g < 16; ++g) {
            u32x4 v = {act(c00[g], s_hi), act(c01[g], s_hi), act(c02[g], s_hi), act(c03[g], s_hi)};
            store16_wt(v, outr, voff, ((g & 3) + 8 * (g >> 2)) * DIM * 4);
        }
        #pragma unroll
        for (int g = 0; g < 16; ++g) {
            u32x4 v = {act(c10[g], s_hi), act(c11[g], s_hi), act(c12[g], s_hi), act(c13[g], s_hi)};
            store16_wt(v, outr, voff, (128 + (g & 3) + 8 * (g >> 2)) * DIM * 4);
        }
    } else if (wave == 8) {
        __builtin_amdgcn_s_setprio(3);
        unsigned* ctr = (unsigned*)(myws + CTR_OFF + wg * 128);
        unsigned* claims = (unsigned*)(myws + CTR_OFF + idx * 8 * 128 + 32);
        unsigned* dbg = (unsigned*)(myws + DBG_OFF + wg * 1024 + 128);
        if (idx < 16) push_loop<false>(P, P.x, rank, idx, ctr, claims, lane, dbg);
        else push_loop<true>(P, P.w, rank, idx - 16, ctr, claims, lane, dbg);
        TSTAMP(1);
    } else if (wave == 9) {
        __builtin_amdgcn_s_setprio(3);
        unsigned* ctr = (unsigned*)(myws + CTR_OFF + wg * 128 + 64);
        unsigned* tmo = (unsigned*)(myws + TMO_OFF);
        const unsigned prev = __hip_atomic_load(ctr, __ATOMIC_RELAXED, __HIP_MEMORY_SCOPE_AGENT);
        const float scale = P.sx[0] * P.sw[0];
        __builtin_amdgcn_s_barrier();
        const unsigned call = __builtin_amdgcn_readfirstlane(prev) + 1;
        __hip_atomic_store(ctr, call, __ATOMIC_RELAXED, __HIP_MEMORY_SCOPE_AGENT);
        const int p = (rank + (lane >> 1)) & 7, o = lane & 1;
        const unsigned* fl = lane < 16 ? (const unsigned*)(myws + FLAG_OFF + (size_t)(((o ? 16 + tj : ti) * NR + p) * 128)) : tmo;
        volatile LAS unsigned* sched = (volatile LAS unsigned*)(L8 + L_SCHED);
        volatile LAS float* lscale = (volatile LAS float*)(L8 + L_SCALE);
        *lscale = scale;
        const unsigned base = call * 16;
        const unsigned code0 = ((call & 1) * 64 + p * 8) + 1;
        int cur = 0, nsched = 0, limit = NF;
        bool dead = false;
        const unsigned long long t0 = __builtin_amdgcn_s_memrealtime();
        for (;;) {
            int av = 15;
            if (!dead) {
                unsigned val = 0;
                if (lane <= 16) val = __hip_atomic_load(fl, __ATOMIC_RELAXED, __HIP_MEMORY_SCOPE_SYSTEM);
                if (__builtin_amdgcn_readlane(val, 16) == call || __builtin_amdgcn_s_memrealtime() - t0 > DEADLINE_TICKS) {
                    __hip_atomic_store(tmo, call, __ATOMIC_RELAXED, __HIP_MEMORY_SCOPE_AGENT);
                    *lscale = __builtin_nanf("");
                    dead = true;
                } else {
                    av = (int)(val - base);
                    if (o && av >= NCH) av = 15;
                    const int other = __builtin_amdgcn_update_dpp(0, av, 0xB1, 0xf, 0xf, false);
                    av = av < other ? av : other;
                }
            }
            for (;;) {
                const bool rdy = lane < 16 && av > cur && cur < limit;
                const unsigned long long m = __ballot(rdy) & 0x5555ull;
                if (m == 0) break;
                if (rdy && o == 0) sched[nsched + __builtin_popcountll(m & ((1ull << lane) - 1))] = code0 + (cur < NCH ? cur : cur - NS);
                cur += rdy ? 1 : 0;
                nsched += __builtin_popcountll(m);
                if (nsched == NR * limit) limit = limit == NF ? NCH : NCH + NS;
            }
            if (nsched >= NSTEP) break;
            __builtin_amdgcn_s_sleep(1);
        }
        TSTAMP(2);
    } else {
        __builtin_amdgcn_s_setprio(3);
        __builtin_amdgcn_s_barrier();
        const int o = wave - 10;
        volatile LAS unsigned* sched = (volatile LAS unsigned*)(L8 + L_SCHED);
        volatile LAS unsigned* freew = (volatile LAS unsigned*)(L8 + L_FREE);
        volatile LAS unsigned* fullw = (volatile LAS unsigned*)(L8 + L_FULL);
        const char* stage_x = myws + (size_t)ti * ITEM;
        const char* stage_w = myws + (size_t)(16 + tj) * ITEM;
        const unsigned v0 = lane * 16, v1 = v0 + 4096;
        auto dma8 = [&](const char* src, unsigned ldst) __attribute__((always_inline)) {
            unsigned keep;
#define DMA4(V) "global_load_lds_dwordx4 " V ", %[src] sc0 sc1\n\t" \
                "global_load_lds_dwordx4 " V ", %[src] offset:1024 sc0 sc1\n\t" \
                "global_load_lds_dwordx4 " V ", %[src] offset:2048 sc0 sc1\n\t" \
                "global_load_lds_dwordx4 " V ", %[src] offset:3072 sc0 sc1\n\t"
            asm volatile(
                "s_mov_b32 %[keep], m0\n\t"
                "s_mov_b32 m0, %[ld]\n\t"
                "s_nop 0\n\t"
                DMA4("%[v0]")
                "s_add_u32 m0, m0, 4096\n\t"
                "s_nop 0\n\t"
                DMA4("%[v1]")
                "s_mov_b32 m0, %[keep]\n\t"
                : [keep] "=&s"(keep) : [src] "s"(src), [ld] "s"(ldst), [v0] "v"(v0), [v1] "v"(v1) : "memory", "scc");
        };
        auto slot_id = [](int s_) { return s_ < NR * NF ? (s_ & 3) : 4 + (s_ - NR * NF) % 6; };
        int issued = o, pub = o;
        while (pub < NSTEP) {
            bool did = false;
            if (issued < NSTEP && issued - pub < (pub >= NR * NF ? 6 : 4)) {
                const unsigned codev = sched[issued];
                const int fr = (int)freew[lane & 7];
                const unsigned code = __builtin_amdgcn_readfirstlane(codev);
                const int thr = issued < NR * NF ? issued - 3 : (issued - 5 > NR * NF ? issued - 5 : NR * NF);
                if (code != 0u && __all(fr >= thr)) {
                    const size_t off = (size_t)((code - 1) & 127u) * (NBLK * ITEM);
                    const char* sx_ = stage_x + off;
                    const char* sw_ = stage_w + off;
                    asm volatile("s_nop 4" : "+s"(sx_), "+s"(sw_));
                    if (issued < NR * NF) {
                        const unsigned ldst = L_RING + (issued & 3) * 32768;
                        dma8(sx_, ldst); dma8(sw_, ldst + 16384); dma8(sx_ + 8192, ldst + 8192); dma8(sw_ + 8192, ldst + 24576);
                    } else {
                        const unsigned ldst = L_RING + (slot_id(issued) - 4) * 24576;
                        dma8(sx_ + (issued >= NR * NCH ? 8192 : 0), ldst); dma8(sw_, ldst + 8192); dma8(sw_ + 8192, ldst + 16384);
                    }
                    issued += 2;
                    did = true;
                }
            }
            if (!did || issued - pub >= (pub >= NR * NF ? 6 : 4)) {
                if (issued > pub) {
                    if (issued - pub == 2) wait_vm<0>();
                    else if (pub + 2 < NR * NF) wait_vm<32>();
                    else if (issued - pub == 4) wait_vm<24>();
                    else wait_vm<48>();
                    fullw[slot_id(pub) * 8 + (lane & 7)] = pub + 1;
                    pub += 2;
                } else {
                    __builtin_amdgcn_s_sleep(1);
                }
            }
        }
        if (o == 0) TSTAMP(3);
    }
}

}

extern "C" void kernel_launch(int rank, int nranks, ncclComm_t comm,
                              hipStream_t stream,
                              void* const* d_in, const int* in_sizes, int n_in,
                              void* d_out, int out_size,
                              const int* in_dtypes, int out_dtype,
                              void* const* d_out_all,
                              void* const* d_ws_all, size_t ws_bytes) {
    if (nranks != NR || n_in < 4 || ws_bytes < WS_NEED || (unsigned)rank >= (unsigned)NR ||
        in_sizes[0] != DIM * KLOC || in_sizes[1] != DIM * KLOC || out_size != DIM * DIM) {
        fprintf(stderr, "kernel_launch: unexpected problem shape\n");
        return;
    }
    Params P;
    P.x = (const unsigned char*)d_in[0];
    P.w = (const unsigned char*)d_in[1];
    P.sx = (const float*)d_in[2];
    P.sw = (const float*)d_in[3];
    P.out = (float*)d_out;
    P.myws = (char*)d_ws_all[rank];
    {
        const int quad[NR] = {0, 1, 0, 0, 1, 1, 0, 1};
        int n = 0;
        for (int pass = 0; pass < 2; ++pass)
            for (int q = 0; q < NR; ++q) {
                const int p = (rank + q) % NR;
                if ((quad[p] == quad[rank]) == (pass == 0)) { P.dws[n] = (char*)d_ws_all[p]; P.rot[n] = q * ROT / 7; ++n; }
            }
    }
    P.rank = rank;
    hipLaunchKernelGGL(fused_ag_gemm, dim3(256), dim3(768), 0, stream, P);
    HIP_CHECK(hipGetLastError());
}
```

```cpp
#include <hip/hip_runtime.h>
#include <rccl/rccl.h>
#include <cstdio>
#include <cstdint>
#include <type_traits>

#define HIP_CHECK(expr) do { hipError_t _e = (expr); if (_e != hipSuccess) { \
    fprintf(stderr, "HIP error %s at %s:%d\n", hipGetErrorString(_e), __FILE__, __LINE__); \
    return; } } while (0)

#define TAU16 440
#define TOFF 80
#define S_FULL 31
#define S_LAST 7
#define S_PREV 13
#define SKEW_L 16
#define SKEW_G 3
#define DCAP 2
#define CAP_SPAN 1200
#define OUT_AUX 17
#define PUSH_AUX 17
#define SRC_AUX 0
#define CONS_SLEEP 1
#define POLL_SLOW 200
#define POLL_FAST 2
#define POLL_LEAD 90

namespace {

typedef unsigned u32x4 __attribute__((ext_vector_type(4)));
typedef float f32x16 __attribute__((ext_vector_type(16)));
#define LAS __attribute__((address_space(3)))

constexpr int NR = 8;
constexpr int DIM = 4096;
constexpr int KLOC = DIM / NR;
constexpr int NCH = 8;
constexpr int NBLK = 32;
constexpr int NSTEP = NR * NCH;
constexpr unsigned ITEM = 16384;
constexpr size_t STAGE_PAR = (size_t)NR * NCH * NBLK * ITEM;
constexpr size_t FLAG_OFF = 2 * STAGE_PAR;
constexpr size_t CTR_OFF = FLAG_OFF + 32768;
constexpr size_t TMO_OFF = CTR_OFF + 32768;
constexpr size_t DBG_OFF = TMO_OFF + 4096;
constexpr size_t WS_NEED = DBG_OFF + 256 * 1024;
constexpr unsigned long long DEADLINE_TICKS = 50000000ull;

constexpr int L_FULL = 0;
constexpr int L_FREE = 512;
constexpr int L_SCHED = 1024;
constexpr int L_SCALE = 1536;
constexpr int L_CTRL_BYTES = 2048;
constexpr int L_RING = 8192;
constexpr int L_TOTAL = L_RING + 4 * 32768;

struct Params {
    const unsigned char* x;
    const unsigned char* w;
    const float* sx;
    const float* sw;
    float* out;
    char* myws;
    char* dws[NR];
    int rank;
};

struct Sched { unsigned t[NBLK][16]; };
constexpr int item_end(int b, int c) {
    const int u = 2 * (b & 15) + (b >> 4) + 1, g = c + SKEW_G - 7 > 0 ? c + SKEW_G - 7 : 0;
    return (c * 32 + u) * 16 - g * SKEW_L * (32 - u) / SKEW_G;
}
constexpr int store_key(int b, int g, int u) {
    const int c = g >> 1, gap = (item_end(b, c) - item_end(b, c - 1)) / 16 - 1;
    int stride = c == NCH - 1 ? S_LAST : c == NCH - 2 ? S_PREV : S_FULL;
    if (stride > gap) stride = gap;
    return item_end(b, c) - (15 - (g & 1) * 8 - u) * stride + 1024;
}
constexpr Sched make_sched() {
    Sched r = {};
    unsigned short cnt[5184] = {};
    for (int b = 0; b < NBLK; ++b) for (int g = 0; g < 16; ++g) for (int u = 0; u < 8; ++u) ++cnt[store_key(b, g, u) + 1];
    for (int k = 1; k < 5184; ++k) cnt[k] += cnt[k - 1];
    for (int b = 0; b < NBLK; ++b)
        for (int g = 0; g < 16; ++g) {
            unsigned first = 0, last = 0;
            for (int u = 0; u < 8; ++u) { last = cnt[store_key(b, g, u)]++; if (u == 0) first = last; }
            r.t[b][g] = (TOFF * 4 + first * TAU16 / 64) | ((TOFF * 4 + last * TAU16 / 64) << 16);
        }
    return r;
}
__device__ constexpr Sched SEND_SCHED = make_sched();

__device__ __forceinline__ __amdgpu_buffer_rsrc_t make_rsrc(const void* p, unsigned bytes) {
    return __builtin_amdgcn_make_buffer_rsrc((void*)p, 0, (int)bytes, 0x00020000);
}

template <int AUX = 17>
__device__ __forceinline__ void store16_wt(u32x4 v, __amdgpu_buffer_rsrc_t r, int voff, int soff) {
    __builtin_amdgcn_raw_buffer_store_b128(v, r, voff, soff, AUX);
    __builtin_amdgcn_sched_barrier(0);
    asm volatile("s_nop 1" ::: "memory");
    __builtin_amdgcn_sched_barrier(0);
}

template <int I, int N, typename F> __device__ __forceinline__ void static_for(F&& f) {
    if constexpr (I < N) { f(std::integral_constant<int, I>{}); static_for<I + 1, N>(f); }
}

template <int N> __device__ __forceinline__ void wait_vm() {
    static_assert(N >= 0 && N <= 63, "");
    asm volatile("s_waitcnt vmcnt(%0)" :: "i"(N) : "memory");
}

#define CSTAMP(idx) do {} while (0)
#define TSTAMP(idx) do {} while (0)

template <bool IS_W>
__device__ __forceinline__ void push_loop(const Params& P, const unsigned char* srcp, int rank, int blk,
                                          unsigned* ctr, unsigned* claims, int lane, unsigned* dbg) {
    const unsigned t0 = (unsigned)__builtin_amdgcn_s_memrealtime();
    const __amdgpu_buffer_rsrc_t src = make_rsrc(srcp, DIM * KLOC);
    const int voff_ld = IS_W ? lane * 4 : (lane & 15) * KLOC + (lane >> 4) * 16;
    unsigned rawA[32], rawB[32];
    auto load_grp = [&](int g, unsigned (&raw)[32]) __attribute__((always_inline)) {
        const int c = g >> 1, h = g & 1;
        if constexpr (IS_W) {
            const int so = (c * 64 + h * 32) * DIM + blk * 256;
            #pragma unroll
            for (int k = 0; k < 32; ++k) raw[k] = __builtin_amdgcn_raw_buffer_load_b32(src, voff_ld, so + k * DIM, SRC_AUX);
        } else {
            const int so = (blk * 256 + h * 128) * KLOC + c * 64;
            #pragma unroll
            for (int t = 0; t < 8; ++t) {
                u32x4 v = __builtin_amdgcn_raw_buffer_load_b128(src, voff_ld, so + t * 16 * KLOC, SRC_AUX);
                raw[4 * t] = v.x; raw[4 * t + 1] = v.y; raw[4 * t + 2] = v.z; raw[4 * t + 3] = v.w;
            }
        }
    };
    auto claim_add = [&](int sl, unsigned d) __attribute__((always_inline)) {
        unsigned old;
        asm volatile("s_mov_b64 exec, 1\n\ts_nop 4\n\tglobal_atomic_add %0, %1, %2, %3 sc0\n\ts_mov_b64 exec, -1"
                     : "=&v"(old) : "v"(sl * 128), "v"(d), "s"(claims) : "memory");
        return old;
    };
    auto claim_wait = [&](unsigned& v) __attribute__((always_inline)) {
        asm volatile("s_waitcnt vmcnt(0)" : "+v"(v) :: "memory");
        return (unsigned)__builtin_amdgcn_readfirstlane(v);
    };
    unsigned xcc;
    asm volatile("s_getreg_b32 %0, hwreg(HW_REG_XCC_ID)" : "=s"(xcc));
    int e = (int)(xcc & 7u);
    const unsigned prev = __hip_atomic_load(ctr, __ATOMIC_RELAXED, __HIP_MEMORY_SCOPE_AGENT);
    unsigned got = claim_add(e, 1u);
    load_grp(0, rawA);
    char* dws = P.dws[e];
    const unsigned* tsched = SEND_SCHED.t[(IS_W ? 16 : 0) | blk];
    { unsigned warm = tsched[0]; asm volatile("" :: "s"(warm), "s"(dws)); }
    __builtin_amdgcn_s_barrier();
    unsigned gs = claim_wait(got);
    const unsigned call = __builtin_amdgcn_readfirstlane(prev) + 1;
    #pragma nounroll
    for (int tries = 0; tries < 7 && gs != call - 1; ++tries) {
        got = claim_add(e, ~0u);
        claim_wait(got);
        e = (e + 1) & 7;
        got = claim_add(e, 1u);
        gs = claim_wait(got);
        dws = P.dws[e];
    }
    __hip_atomic_store(ctr, call, __ATOMIC_RELAXED, __HIP_MEMORY_SCOPE_AGENT);
    const bool paced = e != 0;
    const __amdgpu_buffer_rsrc_t dstr = make_rsrc(dws + (size_t)(call & 1) * STAGE_PAR + (size_t)rank * (NCH * NBLK * ITEM), NCH * NBLK * ITEM);
    const __amdgpu_buffer_rsrc_t flr = make_rsrc(dws + FLAG_OFF, 32768);
    const int voff_st = lane * 16;
    const int voff_fl = lane < 8 ? lane * 4 : 0x7ffffff0;
    const int b = blk + (IS_W ? 16 : 0);
    auto send_grp = [&](int g, unsigned (&raw)[32], auto first) __attribute__((always_inline)) {
        u32x4 o[8];
        if constexpr (IS_W) {
            #pragma unroll
            for (int r = 0; r < 8; ++r) {
                const unsigned t0_ = __builtin_amdgcn_perm(raw[4 * r + 1], raw[4 * r], 0x05010400u);
                const unsigned t1_ = __builtin_amdgcn_perm(raw[4 * r + 1], raw[4 * r], 0x07030602u);
                const unsigned t2_ = __builtin_amdgcn_perm(raw[4 * r + 3], raw[4 * r + 2], 0x05010400u);
                const unsigned t3_ = __builtin_amdgcn_perm(raw[4 * r + 3], raw[4 * r + 2], 0x07030602u);
                o[0 + (r >> 2)][r & 3] = __builtin_amdgcn_perm(t2_, t0_, 0x05040100u);
                o[2 + (r >> 2)][r & 3] = __builtin_amdgcn_perm(t2_, t0_, 0x07060302u);
                o[4 + (r >> 2)][r & 3] = __builtin_amdgcn_perm(t3_, t1_, 0x05040100u);
                o[6 + (r >> 2)][r & 3] = __builtin_amdgcn_perm(t3_, t1_, 0x07060302u);
            }
        } else {
            #pragma unroll
            for (int t = 0; t < 8; ++t) o[t] = u32x4{raw[4 * t], raw[4 * t + 1], raw[4 * t + 2], raw[4 * t + 3]};
        }
        #pragma unroll
        for (int u = 0; u < 8; ++u) asm volatile("" : "+v"(o[u]));
        const int so = ((g >> 1) * NBLK + b) * ITEM + (g & 1) * (IS_W ? 2048 : 8192);
        const unsigned ts = tsched[g];
        const int ta = (ts & 0xffffu) * 7, span = (ts >> 16) - (ts & 0xffffu);
        const bool capped = paced && span >= CAP_SPAN;
        static_for<0, 8>([&](auto uc) __attribute__((always_inline)) {
            constexpr int u = decltype(uc)::value, L = IS_W ? 32 : 8, LU = decltype(first)::value ? 4 : 0;
            if (paced) {
                const int tgt = ta + span * u;
                for (;;) {
                    const int left = tgt - (int)((unsigned)__builtin_amdgcn_s_memrealtime() - t0) * 28;
                    if (left <= 0) break;
                    if (left > 40 * 28) __builtin_amdgcn_s_sleep(8); else if (left > 6 * 28) __builtin_amdgcn_s_sleep(1);
                }
            }
            if (capped) wait_vm<DCAP - 1 + ((u > LU && u <= LU + DCAP) ? L : 0)>();
            store16_wt<PUSH_AUX>(o[u], dstr, voff_st, so + (IS_W ? (u >> 1) * 4096 + (u & 1) * 1024 : u * 1024));
            if (decltype(first)::value && u == 0) load_grp(1, rawB);
            if (u == LU && g < 14) load_grp(g + 2, raw);
        });
        if (g & 1) {
            wait_vm<0>();
            __builtin_amdgcn_raw_buffer_store_b32(call * 16 + (g >> 1) + 1, flr, voff_fl, (b * NR + rank) * 128, 17);
        }
    };
    send_grp(0, rawA, std::true_type{});
    send_grp(1, rawB, std::false_type{});
    #pragma nounroll
    for (int g = 2; g < 16; g += 2) {
        send_grp(g, rawA, std::false_type{});
        send_grp(g + 1, rawB, std::false_type{});
    }
    wait_vm<0>();
}

#define MFMA(c, a, b) "v_mfma_f32_32x32x64_f8f6f4 %[" #c "], " a ", " b ", %[" #c "] cbsz:0 blgp:1\n\t"
#define X0 "v[136:143]"
#define X1 "v[144:151]"
#define Y0 "v[152:159]"
#define Y1 "v[160:167]"
#define STR2(x) #x
#define STR(x) STR2(x)
#define RD(lo, hi, addr, off1, off2) \
    "ds_read_b128 " lo ", %[" #addr "] offset:" off1 "\n\t" \
    "ds_read_b128 " hi ", %[" #addr "] offset:" off2 "\n\t"
#define RD_X0(addr, o1, o2) RD("v[136:139]", "v[140:143]", addr, o1, o2)
#define RD_X1(addr, o1, o2) RD("v[144:147]", "v[148:151]", addr, o1, o2)
#define RD_Y0(addr, o1, o2) RD("v[152:155]", "v[156:159]", addr, o1, o2)
#define RD_Y1(addr, o1, o2) RD("v[160:163]", "v[164:167]", addr, o1, o2)
#define WAITL(n) "s_waitcnt lgkmcnt(" #n ")\n\t"
#define CHECK(FS) \
    "1:\n\t" \
    WAITL(0) \
    "v_readlane_b32 %[st], %[vf], %[sw]\n\t" \
    "s_nop 0\n\t" \
    "s_cmp_ge_u32 %[st], %[need]\n\t" \
    "s_cbranch_scc1 2f\n\t" \
    "s_sleep " STR(CONS_SLEEP) "\n\t" \
    "ds_read_addtid_b32 %[vf] offset:" STR(FS) "*32\n\t" \
    "s_branch 1b\n\t" \
    "2:\n\t"
#define RELEASE(FS, NX) \
    WAITL(0) \
    "s_mov_b64 exec, %[mask]\n\t" \
    "ds_write_addtid_b32 %[vf] offset:512\n\t" \
    "s_mov_b64 exec, -1\n\t" \
    "ds_read_addtid_b32 %[vf] offset:" STR(NX) "*32\n\t"
#define STEP_FULL(SO, FS, NX) \
    CHECK(FS) \
    RD_X0(va, STR(SO), STR(SO) "+256") \
    RD_Y0(vb, STR(SO), STR(SO) "+1024") \
    RD_X1(va, STR(SO) "+8192", STR(SO) "+8448") \
    RD_Y1(vb, STR(SO) "+4096", STR(SO) "+5120") \
    WAITL(4) \
    MFMA(c00, X0, Y0) \
    WAITL(2) \
    MFMA(c10, X1, Y0) \
    RD_Y0(vb, STR(SO) "+8192", STR(SO) "+9216") \
    WAITL(2) \
    MFMA(c01, X0, Y1) \
    MFMA(c11, X1, Y1) \
    RD_Y1(vb, STR(SO) "+12288", STR(SO) "+13312") \
    WAITL(2) \
    MFMA(c02, X0, Y0) \
    MFMA(c12, X1, Y0) \
    RELEASE(FS, NX) \
    MFMA(c03, X0, Y1) \
    "s_add_u32 %[need], %[need], 1\n\t" \
    MFMA(c13, X1, Y1)
#define FLIP \
    "v_xor_b32 %[va], 0x10000, %[va]\n\t" \
    "v_xor_b32 %[vb], 0x10000, %[vb]\n\t"
#define FRAG_CLOBBERS \
    "v136", "v137", "v138", "v139", "v140", "v141", "v142", "v143", "v144", "v145", "v146", "v147", \
    "v148", "v149", "v150", "v151", "v152", "v153", "v154", "v155", "v156", "v157", "v158", "v159", \
    "v160", "v161", "v162", "v163", "v164", "v165", "v166", "v167"

__global__ void __launch_bounds__(768, 3) fused_ag_gemm(Params P) {
    __shared__ __attribute__((aligned(1024))) unsigned char lds[L_TOTAL];
    LAS unsigned char* L8 = (LAS unsigned char*)lds;
    const int wave = __builtin_amdgcn_readfirstlane(threadIdx.x >> 6);
    const int lane = threadIdx.x & 63;
    const int wg = blockIdx.x;
    const int e = wg & 7, idx = wg >> 3;
    const int ti = (e >> 1) + 4 * (idx >> 3);
    const int tj = (e & 1) + 2 * (idx & 7);
    char* const myws = P.myws;
    const int rank = P.rank;

    if (wave < 8) {
        if (wave == 0) {
            #pragma unroll
            for (int i = 0; i < L_CTRL_BYTES / 256; ++i) ((LAS unsigned*)L8)[i * 64 + lane] = 0u;
            asm volatile("s_waitcnt lgkmcnt(0)" ::: "memory");
            TSTAMP(0);
            CSTAMP(9);
        }
        __builtin_amdgcn_s_barrier();
        if (wave >= 4) __builtin_amdgcn_s_setprio(1);
        const int wm = wave >> 1, wn = wave & 1;
        const int r = lane & 31, h = lane >> 5;
        unsigned va = L_RING + wm * 2048 + (r >> 4) * 1024 + h * 512 + (r & 15) * 16;
        unsigned vb = L_RING + 16384 + h * 2048 + wn * 512 + r * 16;
        f32x16 c00 = {}, c01 = {}, c02 = {}, c03 = {}, c10 = {}, c11 = {}, c12 = {}, c13 = {};
        unsigned vf, st, need, keep;
        const unsigned long long mask = 1ull << wave;
        float* outp = P.out + (size_t)(ti * 256 + wm * 32) * DIM + tj * 256 + wn * 128;
        asm volatile("" : "+s"(outp));
        asm volatile(
            "s_mov_b32 %[keep], m0\n\t"
            "s_mov_b32 m0, 0\n\t"
            "s_mov_b32 %[need], 1\n\t"
            "ds_read_addtid_b32 %[vf] offset:0\n\t"
            "3:\n\t"
            STEP_FULL(0, 0, 1)
            STEP_FULL(32768, 1, 2)
            FLIP
            STEP_FULL(0, 2, 3)
            STEP_FULL(32768, 3, 0)
            FLIP
            "s_cmp_le_u32 %[need], %[nfull]\n\t"
            "s_cbranch_scc1 3b\n\t"
            WAITL(0)
            "s_mov_b32 m0, %[keep]\n\t"
            "s_nop 15\n\t"
            "s_nop 7\n\t"
            : [c00] "+v"(c00), [c01] "+v"(c01), [c02] "+v"(c02), [c03] "+v"(c03),
              [c10] "+v"(c10), [c11] "+v"(c11), [c12] "+v"(c12), [c13] "+v"(c13),
              [va] "+v"(va), [vb] "+v"(vb), [vf] "=&v"(vf), [st] "=&s"(st), [need] "=&s"(need), [keep] "=&s"(keep)
            : [sw] "s"(wave), [mask] "s"(mask), [nfull] "i"(NSTEP)
            : "memory", "scc", FRAG_CLOBBERS);
        if (wave == 0) { TSTAMP(8); CSTAMP(10); }
        const int lz = __builtin_amdgcn_mbcnt_hi(~0u, __builtin_amdgcn_mbcnt_lo(~0u, 0u));
        const __amdgpu_buffer_rsrc_t outr = make_rsrc(outp, 160u * DIM * 4u);
        const int voff = ((lz >> 5) * 4 * DIM + (lz & 31) * 4) * 4;
        const float sc = __builtin_bit_cast(float, __builtin_amdgcn_readfirstlane(*(volatile LAS unsigned*)(L8 + L_SCALE)));
        auto act = [&](float a) { return __builtin_bit_cast(unsigned, __builtin_elementwise_maximum(a * sc, 0.f)); };
        #pragma unroll
        for (int g = 0; g < 16; ++g) {
            u32x4 v0 = {act(c00[g]), act(c01[g]), act(c02[g]), act(c03[g])};
            store16_wt<OUT_AUX>(v0, outr, voff, ((g & 3) + 8 * (g >> 2)) * DIM * 4);
            u32x4 v1 = {act(c10[g]), act(c11[g]), act(c12[g]), act(c13[g])};
            store16_wt<OUT_AUX>(v1, outr, voff, (128 + (g & 3) + 8 * (g >> 2)) * DIM * 4);
        }
    } else if (wave == 8) {
        __builtin_amdgcn_s_setprio(3);
        unsigned* ctr = (unsigned*)(myws + CTR_OFF + wg * 128);
        unsigned* claims = (unsigned*)(myws + CTR_OFF + idx * 8 * 128 + 32);
        unsigned* dbg = (unsigned*)(myws + DBG_OFF + wg * 1024 + 128);
        if (idx < 16) push_loop<false>(P, P.x, rank, idx, ctr, claims, lane, dbg);
        else push_loop<true>(P, P.w, rank, idx - 16, ctr, claims, lane, dbg);
        TSTAMP(1);
    } else if (wave == 9) {
        __builtin_amdgcn_s_setprio(3);
        const unsigned long long t0 = __builtin_amdgcn_s_memrealtime();
        unsigned* ctr = (unsigned*)(myws + CTR_OFF + wg * 128 + 64);
        unsigned* tmo = (unsigned*)(myws + TMO_OFF);
        const unsigned prev = __hip_atomic_load(ctr, __ATOMIC_RELAXED, __HIP_MEMORY_SCOPE_AGENT);
        const float scale = P.sx[0] * P.sw[0];
        __builtin_amdgcn_s_barrier();
        const unsigned call = __builtin_amdgcn_readfirstlane(prev) + 1;
        __hip_atomic_store(ctr, call, __ATOMIC_RELAXED, __HIP_MEMORY_SCOPE_AGENT);
        const int p = (rank + (lane >> 1)) & 7, o = lane & 1;
        const unsigned* fl = lane < 16 ? (const unsigned*)(myws + FLAG_OFF + (size_t)(((o ? 16 + tj : ti) * NR + p) * 128)) : tmo;
        volatile LAS unsigned* sched = (volatile LAS unsigned*)(L8 + L_SCHED);
        volatile LAS float* lscale = (volatile LAS float*)(L8 + L_SCALE);
        *lscale = scale;
        const unsigned base = call * 16;
        const unsigned code0 = ((call & 1) * 64 + p * 8) + 1;
        int cur = 0, nsched = 0, known = 0;
        bool dead = false;
        const unsigned tx_ = SEND_SCHED.t[ti][15] >> 16, tw_ = SEND_SCHED.t[16 + tj][15] >> 16;
        const unsigned t_fast = (tx_ > tw_ ? tx_ : tw_) / 4 + POLL_LEAD;
        for (;;) {
            if (!dead) {
                unsigned val = 0;
                const bool act_ = lane == 16 || (lane < 16 && known <= cur && cur < NCH);
                if (act_) val = __hip_atomic_load(fl, __ATOMIC_RELAXED, __HIP_MEMORY_SCOPE_SYSTEM);
                if (__builtin_amdgcn_readlane(val, 16) == call) dead = true;
                if (act_) known = (int)(val - base);
            }
            if (dead) { *lscale = __builtin_nanf(""); known = 15; }
            const int other = __builtin_amdgcn_update_dpp(0, known, 0xB1, 0xf, 0xf, false);
            const int av = known < other ? known : other;
            for (;;) {
                const bool rdy = lane < 16 && av > cur && cur < NCH;
                const unsigned long long m = __ballot(rdy) & 0x5555ull;
                if (m == 0) break;
                if (rdy && o == 0) sched[nsched + __builtin_popcountll(m & ((1ull << lane) - 1))] = code0 + cur;
                cur += rdy ? 1 : 0;
                nsched += __builtin_popcountll(m);
            }
            if (nsched >= NSTEP) break;
            const unsigned long long now = __builtin_amdgcn_s_memrealtime() - t0;
            if (now > DEADLINE_TICKS) {
                __hip_atomic_store(tmo, call, __ATOMIC_RELAXED, __HIP_MEMORY_SCOPE_AGENT);
                dead = true;
            } else if ((unsigned)now >= t_fast) {
                __builtin_amdgcn_s_sleep(POLL_FAST);
            } else {
                const unsigned until = (unsigned)now + POLL_SLOW < t_fast ? (unsigned)now + POLL_SLOW : t_fast;
                while ((unsigned)(__builtin_amdgcn_s_memrealtime() - t0) < until) __builtin_amdgcn_s_sleep(4);
            }
        }
        TSTAMP(2);
    } else {
        __builtin_amdgcn_s_setprio(3);
        __builtin_amdgcn_s_barrier();
        const int o = wave - 10;
        volatile LAS unsigned* sched = (volatile LAS unsigned*)(L8 + L_SCHED);
        volatile LAS unsigned* freew = (volatile LAS unsigned*)(L8 + L_FREE);
        volatile LAS unsigned* fullw = (volatile LAS unsigned*)(L8 + L_FULL);
        const char* stage_x = myws + (size_t)ti * ITEM;
        const char* stage_w = myws + (size_t)(16 + tj) * ITEM;
        const unsigned v0 = lane * 16, v1 = v0 + 4096;
        auto dma8 = [&](const char* src, unsigned ldst) __attribute__((always_inline)) {
            unsigned keep;
#define DMA4(V) "global_load_lds_dwordx4 " V ", %[src] sc0 sc1\n\t" \
                "global_load_lds_dwordx4 " V ", %[src] offset:1024 sc0 sc1\n\t" \
                "global_load_lds_dwordx4 " V ", %[src] offset:2048 sc0 sc1\n\t" \
                "global_load_lds_dwordx4 " V ", %[src] offset:3072 sc0 sc1\n\t"
            asm volatile(
                "s_mov_b32 %[keep], m0\n\t"
                "s_mov_b32 m0, %[ld]\n\t"
                "s_nop 0\n\t"
                DMA4("%[v0]")
                "s_add_u32 m0, m0, 4096\n\t"
                "s_nop 0\n\t"
                DMA4("%[v1]")
                "s_mov_b32 m0, %[keep]\n\t"
                : [keep] "=&s"(keep) : [src] "s"(src), [ld] "s"(ldst), [v0] "v"(v0), [v1] "v"(v1) : "memory", "scc");
        };
        int issued = o, pub = o;
        while (pub < NSTEP) {
            bool did = false;
            if (issued < NSTEP && issued - pub < 4) {
                const unsigned codev = sched[issued];
                const int fr = (int)freew[lane & 7];
                const unsigned code = __builtin_amdgcn_readfirstlane(codev);
                if (code != 0u && __all(fr >= issued - 3)) {
                    const size_t off = (size_t)((code - 1) & 127u) * (NBLK * ITEM);
                    const char* sx_ = stage_x + off;
                    const char* sw_ = stage_w + off;
                    asm volatile("s_nop 4" : "+s"(sx_), "+s"(sw_));
                    const unsigned ldst = L_RING + (issued & 3) * 32768;
                    dma8(sx_, ldst); dma8(sw_, ldst + 16384); dma8(sx_ + 8192, ldst + 8192); dma8(sw_ + 8192, ldst + 24576);
                    issued += 2;
                    did = true;
                }
            }
            if (!did || issued - pub >= 4) {
                if (issued > pub) {
                    if (issued - pub == 2) wait_vm<0>(); else wait_vm<32>();
                    fullw[(pub & 3) * 8 + (lane & 7)] = pub + 1;
                    pub += 2;
                } else {
                    __builtin_amdgcn_s_sleep(1);
                }
            }
        }
        if (o == 0) TSTAMP(3);
    }
}

}

extern "C" void kernel_launch(int rank, int nranks, ncclComm_t comm,
                              hipStream_t stream,
                              void* const* d_in, const int* in_sizes, int n_in,
                              void* d_out, int out_size,
                              const int* in_dtypes, int out_dtype,
                              void* const* d_out_all,
                              void* const* d_ws_all, size_t ws_bytes) {
    if (nranks != NR || n_in < 4 || ws_bytes < WS_NEED || (unsigned)rank >= (unsigned)NR ||
        in_sizes[0] != DIM * KLOC || in_sizes[1] != DIM * KLOC || out_size != DIM * DIM) {
        fprintf(stderr, "kernel_launch: unexpected problem shape\n");
        return;
    }
    Params P;
    P.x = (const unsigned char*)d_in[0];
    P.w = (const unsigned char*)d_in[1];
    P.sx = (const float*)d_in[2];
    P.sw = (const float*)d_in[3];
    P.out = (float*)d_out;
    P.myws = (char*)d_ws_all[rank];
    {
        const int quad[NR] = {0, 1, 0, 0, 1, 1, 0, 1};
        int n = 0;
        for (int pass = 0; pass < 2; ++pass)
            for (int q = 0; q < NR; ++q) {
                const int p = (rank + q) % NR;
                if ((quad[p] == quad[rank]) == (pass == 0)) P.dws[n++] = (char*)d_ws_all[p];
            }
    }
    P.rank = rank;
    hipLaunchKernelGGL(fused_ag_gemm, dim3(256), dim3(768), 0, stream, P);
    HIP_CHECK(hipGetLastError());
}
```

```cpp
#include <hip/hip_runtime.h>
#include <rccl/rccl.h>
#include <cstdio>
#include <cstdint>
#include <type_traits>

#define HIP_CHECK(expr) do { hipError_t _e = (expr); if (_e != hipSuccess) { \
    fprintf(stderr, "HIP error %s at %s:%d\n", hipGetErrorString(_e), __FILE__, __LINE__); \
    return; } } while (0)

#define TAU16 440
#define TAU_AT 3700
#define TAU_ADD 30
#define LAGW 4
#define TOFF 80
#define S_FULL 31
#define S_LAST 7
#define NSP 2
#define S_HALF 13
#define POLL_MID 6
#define S_PREV 31
#define SKEW_L 24
#define SKEW_G 3
#define DCAP 2
#define DCAP2 8
#define FLAG_AT 3
#define FLAG_DEFER 500
#define CAP_SPAN 1200
#define OUT_AUX 17
#define PUSH_AUX 17
#define SRC_AUX 0
#define POLL_SLOW 200
#define POLL_FAST 2
#define POLL_LEAD 90

namespace {

typedef unsigned u32x4 __attribute__((ext_vector_type(4)));
typedef float f32x16 __attribute__((ext_vector_type(16)));
#define LAS __attribute__((address_space(3)))

constexpr int NR = 8;
constexpr int DIM = 4096;
constexpr int KLOC = DIM / NR;
constexpr int NCH = 8;
constexpr int NBLK = 32;
constexpr int NSTEP = NR * NCH;
constexpr unsigned ITEM = 16384;
constexpr unsigned HALF = 8192;
constexpr size_t STAGE_PAR = (size_t)NR * NCH * NBLK * ITEM;
constexpr size_t FLAG_OFF = 2 * STAGE_PAR;
constexpr size_t CTR_OFF = FLAG_OFF + 32768;
constexpr size_t TMO_OFF = CTR_OFF + 32768;
constexpr size_t DBG_OFF = TMO_OFF + 4096;
constexpr size_t WS_NEED = DBG_OFF + 257 * 1024;
constexpr unsigned long long DEADLINE_TICKS = 50000000ull;

constexpr int L_FULL = 0;
constexpr int L_FREE = 512;
constexpr int L_SCHED = 1024;
constexpr int L_SCALE = 1536;
constexpr int L_CTRL_BYTES = 2048;
constexpr int L_RING = 8192;
constexpr int L_TOTAL = L_RING + 4 * 32768;

struct Params {
    const unsigned char* x;
    const unsigned char* w;
    const float* sx;
    const float* sw;
    float* out;
    char* myws;
    char* dws[NR];
    int rank;
};

struct Sched { unsigned t[NBLK][16]; };
constexpr int grp_end(int b, int g) {
    const int u = 2 * (b & 15) + (b >> 4) + 1, c = g >> 1;
    const int base = c < NCH - NSP ? (c * 32 + u) * 16 : g * 256 + u * 8;
    const int lead = g + 1 + 2 * SKEW_G - 16 > 0 ? g + 1 + 2 * SKEW_G - 16 : 0;
    return base - lead * SKEW_L * (32 - u) / (2 * SKEW_G);
}
constexpr int store_key(int b, int g, int u) {
    const int c = g >> 1;
    if (c < NCH - NSP) {
        const int gap = (grp_end(b, g | 1) - grp_end(b, (g | 1) - 2)) / 16 - 1;
        int stride = c == NCH - 1 ? S_LAST : c >= NCH - NSP - 1 ? S_PREV : S_FULL;
        if (stride > gap) stride = gap;
        return grp_end(b, g | 1) - (15 - (g & 1) * 8 - u) * stride + 1024;
    }
    const int gap = (grp_end(b, g) - grp_end(b, g - 1)) / 8 - 1;
    int stride = g == 15 ? S_LAST : S_HALF;
    if (stride > gap) stride = gap;
    return grp_end(b, g) - (7 - u) * stride + 1024;
}
constexpr unsigned slot_time(unsigned n) { return TOFF * 4 + (n * TAU16 + (n > TAU_AT ? (n - TAU_AT) * TAU_ADD : 0)) / 64; }
constexpr Sched make_sched() {
    Sched r = {};
    unsigned short cnt[5184] = {};
    for (int b = 0; b < NBLK; ++b) for (int g = 0; g < 16; ++g) for (int u = 0; u < 8; ++u) ++cnt[store_key(b, g, u) + 1];
    for (int k = 1; k < 5184; ++k) cnt[k] += cnt[k - 1];
    for (int b = 0; b < NBLK; ++b)
        for (int g = 0; g < 16; ++g) {
            unsigned first = 0, last = 0;
            for (int u = 0; u < 8; ++u) { last = cnt[store_key(b, g, u)]++; if (u == 0) first = last; }
            r.t[b][g] = slot_time(first) | (slot_time(last) << 16);
        }
    return r;
}
__device__ constexpr Sched SEND_SCHED = make_sched();

__device__ __forceinline__ __amdgpu_buffer_rsrc_t make_rsrc(const void* p, unsigned bytes) {
    return __builtin_amdgcn_make_buffer_rsrc((void*)p, 0, (int)bytes, 0x00020000);
}

template <int AUX = 17>
__device__ __forceinline__ void store16_wt(u32x4 v, __amdgpu_buffer_rsrc_t r, int voff, int soff) {
    __builtin_amdgcn_raw_buffer_store_b128(v, r, voff, soff, AUX);
    __builtin_amdgcn_sched_barrier(0);
    asm volatile("s_nop 1" ::: "memory");
    __builtin_amdgcn_sched_barrier(0);
}

template <int I, int N, typename F> __device__ __forceinline__ void static_for(F&& f) {
    if constexpr (I < N) { f(std::integral_constant<int, I>{}); static_for<I + 1, N>(f); }
}

template <int N> __device__ __forceinline__ void wait_vm() {
    static_assert(N >= 0 && N <= 63, "");
    asm volatile("s_waitcnt vmcnt(%0)" :: "i"(N) : "memory");
}

#define CSTAMP(idx) do {} while (0)
#define TSTAMP(idx) do {} while (0)

template <bool IS_W>
__device__ __forceinline__ void push_loop(const Params& P, const unsigned char* srcp, int rank, int blk,
                                          unsigned* ctr, unsigned* claims, int lane, unsigned* dbg) {
    const unsigned t0 = (unsigned)__builtin_amdgcn_s_memrealtime();
    const __amdgpu_buffer_rsrc_t src = make_rsrc(srcp, DIM * KLOC);
    const int voff_ld = IS_W ? lane * 4 : (lane & 31) * KLOC + (lane >> 5) * 16;
    unsigned rawA[32], rawB[32];
    auto load_w = [&](int g, unsigned (&raw)[32]) __attribute__((always_inline)) {
        const int so = g * 32 * DIM + blk * 256;
        #pragma unroll
        for (int k = 0; k < 32; ++k) raw[k] = __builtin_amdgcn_raw_buffer_load_b32(src, voff_ld, so + k * DIM, SRC_AUX);
    };
    auto load_x = [&](int c) __attribute__((always_inline)) {
        const int so = blk * 256 * KLOC + c * 64;
        #pragma unroll
        for (int t = 0; t < 8; ++t) {
            u32x4 v = __builtin_amdgcn_raw_buffer_load_b128(src, voff_ld, so + t * 32 * KLOC, SRC_AUX);
            rawA[4 * t] = v.x; rawA[4 * t + 1] = v.y; rawA[4 * t + 2] = v.z; rawA[4 * t + 3] = v.w;
            v = __builtin_amdgcn_raw_buffer_load_b128(src, voff_ld, so + t * 32 * KLOC + 32, SRC_AUX);
            rawB[4 * t] = v.x; rawB[4 * t + 1] = v.y; rawB[4 * t + 2] = v.z; rawB[4 * t + 3] = v.w;
        }
    };
    auto claim_add = [&](int sl, unsigned d) __attribute__((always_inline)) {
        unsigned old;
        asm volatile("s_mov_b64 exec, 1\n\ts_nop 4\n\tglobal_atomic_add %0, %1, %2, %3 sc0\n\ts_mov_b64 exec, -1"
                     : "=&v"(old) : "v"(sl * 128), "v"(d), "s"(claims) : "memory");
        return old;
    };
    auto claim_wait = [&](unsigned& v) __attribute__((always_inline)) {
        asm volatile("s_waitcnt vmcnt(0)" : "+v"(v) :: "memory");
        return (unsigned)__builtin_amdgcn_readfirstlane(v);
    };
    unsigned xcc;
    asm volatile("s_getreg_b32 %0, hwreg(HW_REG_XCC_ID)" : "=s"(xcc));
    int e = (int)(xcc & 7u);
    const unsigned prev = __hip_atomic_load(ctr, __ATOMIC_RELAXED, __HIP_MEMORY_SCOPE_AGENT);
    unsigned got = claim_add(e, 1u);
    if constexpr (IS_W) load_w(0, rawA); else load_x(0);
    char* dws = P.dws[e];
    const unsigned* tsched = SEND_SCHED.t[(IS_W ? 16 : 0) | blk];
    { unsigned warm = tsched[0]; asm volatile("" :: "s"(warm), "s"(dws)); }
    __builtin_amdgcn_s_barrier();
    unsigned gs = claim_wait(got);
    const unsigned call = __builtin_amdgcn_readfirstlane(prev) + 1;
    #pragma nounroll
    for (int tries = 0; tries < 7 && gs != call - 1; ++tries) {
        got = claim_add(e, ~0u);
        claim_wait(got);
        e = (e + 1) & 7;
        got = claim_add(e, 1u);
        gs = claim_wait(got);
        dws = P.dws[e];
    }
    __hip_atomic_store(ctr, call, __ATOMIC_RELAXED, __HIP_MEMORY_SCOPE_AGENT);
    const bool paced = e != 0;
    const __amdgpu_buffer_rsrc_t dstr = make_rsrc(dws + (size_t)(call & 1) * STAGE_PAR + (size_t)rank * (NCH * NBLK * ITEM), NCH * NBLK * ITEM);
    const __amdgpu_buffer_rsrc_t flr = make_rsrc(dws + FLAG_OFF, 32768);
    const int voff_st = lane * 16;
    const int voff_fl = lane < 8 ? lane * 4 : 0x7ffffff0;
    const int b = blk + (IS_W ? 16 : 0);
    int lag = 0;
    unsigned pend = 0;
    auto send_grp = [&](int g, unsigned (&raw)[32], auto first, auto odd) __attribute__((always_inline)) {
        u32x4 o[8];
        if constexpr (IS_W) {
            #pragma unroll
            for (int r = 0; r < 8; ++r) {
                const unsigned t0_ = __builtin_amdgcn_perm(raw[4 * r + 1], raw[4 * r], 0x05010400u);
                const unsigned t1_ = __builtin_amdgcn_perm(raw[4 * r + 1], raw[4 * r], 0x07030602u);
                const unsigned t2_ = __builtin_amdgcn_perm(raw[4 * r + 3], raw[4 * r + 2], 0x05010400u);
                const unsigned t3_ = __builtin_amdgcn_perm(raw[4 * r + 3], raw[4 * r + 2], 0x07030602u);
                o[0 + (r >> 2)][r & 3] = __builtin_amdgcn_perm(t2_, t0_, 0x05040100u);
                o[2 + (r >> 2)][r & 3] = __builtin_amdgcn_perm(t2_, t0_, 0x07060302u);
                o[4 + (r >> 2)][r & 3] = __builtin_amdgcn_perm(t3_, t1_, 0x05040100u);
                o[6 + (r >> 2)][r & 3] = __builtin_amdgcn_perm(t3_, t1_, 0x07060302u);
            }
        } else {
            #pragma unroll
            for (int t = 0; t < 8; ++t) o[t] = u32x4{raw[4 * t], raw[4 * t + 1], raw[4 * t + 2], raw[4 * t + 3]};
        }
        #pragma unroll
        for (int u = 0; u < 8; ++u) asm volatile("" : "+v"(o[u]));
        const int so = (g * NBLK + b) * HALF;
        const unsigned ts = tsched[g];
        const int span = (ts >> 16) - (ts & 0xffffu);
        const bool capped = paced && span >= CAP_SPAN;
        const int ta = (ts & 0xffffu) * 7 + (capped ? 0 : lag);
        static_for<0, 8>([&](auto uc) __attribute__((always_inline)) {
            constexpr int u = decltype(uc)::value, L = IS_W ? 32 : (decltype(odd)::value ? 16 : 0), LU = decltype(first)::value ? 4 : 0;
            if (paced) {
                const int tgt = ta + span * u;
                for (;;) {
                    const int left = tgt - (int)((unsigned)__builtin_amdgcn_s_memrealtime() - t0) * 28;
                    if (left <= 0) break;
                    if (left > 40 * 28) __builtin_amdgcn_s_sleep(8); else if (left > 6 * 28) __builtin_amdgcn_s_sleep(1);
                }
            }
            if (capped) wait_vm<DCAP - 1 + ((u > LU && u <= LU + DCAP) ? L : 0)>();
            else if (DCAP2 < 8 && paced) wait_vm<DCAP2 - 1 + ((u > LU && u <= LU + DCAP2) ? L : 0)>();
            if (u == FLAG_AT && pend) {
                if (g < 14) wait_vm<FLAG_AT + L>(); else wait_vm<FLAG_AT>();
                __builtin_amdgcn_raw_buffer_store_b32(pend, flr, voff_fl, (b * NR + rank) * 128, 17);
                pend = 0;
            }
            if (capped && (u == 3 || u == 7)) {
                const int l_ = (int)((unsigned)__builtin_amdgcn_s_memrealtime() - t0) * 28 - (ta + span * u);
                lag += (l_ - lag) * LAGW / 16;
            }
            store16_wt<PUSH_AUX>(o[u], dstr, voff_st, so + u * 1024);
            if constexpr (IS_W) {
                if (decltype(first)::value && u == 0) load_w(1, rawB);
                if (u == LU && g < 14) load_w(g + 2, raw);
            } else if (decltype(odd)::value && u == 0 && g < 14) {
                load_x((g + 1) >> 1);
            }
        });
        if (decltype(odd)::value || g >= 16 - 2 * NSP) {
            const unsigned nx = g < 15 ? tsched[g + 1] : 0xffffffffu;
            if (FLAG_AT > 0 && paced && (int)((nx & 0xffffu) + ((nx >> 16) - (nx & 0xffffu)) * FLAG_AT / 7 - (ts >> 16)) < FLAG_DEFER) {
                pend = call * 32 + g + 1;
            } else {
                wait_vm<0>();
                __builtin_amdgcn_raw_buffer_store_b32(call * 32 + g + 1, flr, voff_fl, (b * NR + rank) * 128, 17);
            }
        }
    };
    send_grp(0, rawA, std::true_type{}, std::false_type{});
    send_grp(1, rawB, std::false_type{}, std::true_type{});
    #pragma nounroll
    for (int g = 2; g < 16; g += 2) {
        send_grp(g, rawA, std::false_type{}, std::false_type{});
        send_grp(g + 1, rawB, std::false_type{}, std::true_type{});
    }
    wait_vm<0>();
}

#define MFMA(c, a, b) "v_mfma_f32_32x32x64_f8f6f4 %[" #c "], " a ", " b ", %[" #c "] cbsz:0 blgp:1\n\t"
#define X0 "v[136:143]"
#define X1 "v[144:151]"
#define Y0 "v[152:159]"
#define Y1 "v[160:167]"
#define STR2(x) #x
#define STR(x) STR2(x)
#define RD(lo, hi, addr, off1, off2) \
    "ds_read_b128 " lo ", %[" #addr "] offset:" off1 "\n\t" \
    "ds_read_b128 " hi ", %[" #addr "] offset:" off2 "\n\t"
#define RD_X0(addr, o1, o2) RD("v[136:139]", "v[140:143]", addr, o1, o2)
#define RD_X1(addr, o1, o2) RD("v[144:147]", "v[148:151]", addr, o1, o2)
#define RD_Y0(addr, o1, o2) RD("v[152:155]", "v[156:159]", addr, o1, o2)
#define RD_Y1(addr, o1, o2) RD("v[160:163]", "v[164:167]", addr, o1, o2)
#define WAITL(n) "s_waitcnt lgkmcnt(" #n ")\n\t"
#define CHECK(FS) \
    "1:\n\t" \
    WAITL(0) \
    "v_readlane_b32 %[st], %[vf], %[sw]\n\t" \
    "s_nop 0\n\t" \
    "s_cmp_ge_u32 %[st], %[need]\n\t" \
    "s_cbranch_scc1 2f\n\t" \
    "s_sleep 1\n\t" \
    "ds_read_addtid_b32 %[vf] offset:" STR(FS) "*32\n\t" \
    "s_branch 1b\n\t" \
    "2:\n\t"
#define RELEASE(FS, NX) \
    WAITL(0) \
    "s_mov_b64 exec, %[mask]\n\t" \
    "ds_write_addtid_b32 %[vf] offset:512\n\t" \
    "s_mov_b64 exec, -1\n\t" \
    "ds_read_addtid_b32 %[vf] offset:" STR(NX) "*32\n\t"
#define STEP_FULL(SO, FS, NX) \
    CHECK(FS) \
    RD_X0(va, STR(SO), STR(SO) "+512") \
    RD_Y0(vb, STR(SO), STR(SO) "+1024") \
    RD_X1(va, STR(SO) "+4096", STR(SO) "+4608") \
    RD_Y1(vb, STR(SO) "+2048", STR(SO) "+3072") \
    WAITL(4) \
    MFMA(c00, X0, Y0) \
    WAITL(2) \
    MFMA(c10, X1, Y0) \
    RD_Y0(vb, STR(SO) "+4096", STR(SO) "+5120") \
    WAITL(2) \
    MFMA(c01, X0, Y1) \
    MFMA(c11, X1, Y1) \
    RD_Y1(vb, STR(SO) "+6144", STR(SO) "+7168") \
    WAITL(2) \
    MFMA(c02, X0, Y0) \
    MFMA(c12, X1, Y0) \
    RELEASE(FS, NX) \
    MFMA(c03, X0, Y1) \
    "s_add_u32 %[need], %[need], 1\n\t" \
    MFMA(c13, X1, Y1)
#define FLIP \
    "v_xor_b32 %[va], 0x10000, %[va]\n\t" \
    "v_xor_b32 %[vb], 0x10000, %[vb]\n\t"
#define FRAG_CLOBBERS \
    "v136", "v137", "v138", "v139", "v140", "v141", "v142", "v143", "v144", "v145", "v146", "v147", \
    "v148", "v149", "v150", "v151", "v152", "v153", "v154", "v155", "v156", "v157", "v158", "v159", \
    "v160", "v161", "v162", "v163", "v164", "v165", "v166", "v167"

__global__ void __launch_bounds__(768, 3) fused_ag_gemm(Params P) {
    __shared__ __attribute__((aligned(1024))) unsigned char lds[L_TOTAL];
    LAS unsigned char* L8 = (LAS unsigned char*)lds;
    const int wave = __builtin_amdgcn_readfirstlane(threadIdx.x >> 6);
    const int lane = threadIdx.x & 63;
    const int wg = blockIdx.x;
    const int e = wg & 7, idx = wg >> 3;
    const int ti = (e >> 1) + 4 * (idx >> 3);
    const int tj = (e & 1) + 2 * (idx & 7);
    char* const myws = P.myws;
    const int rank = P.rank;

    int wv_ = wave;
    asm volatile("" : "+s"(wv_));
    if (__builtin_expect(wv_ == 8, 1)) {
        __builtin_amdgcn_s_setprio(3);
        unsigned* ctr = (unsigned*)(myws + CTR_OFF + wg * 128);
        unsigned* claims = (unsigned*)(myws + CTR_OFF + idx * 8 * 128 + 32);
        unsigned* dbg = (unsigned*)(myws + DBG_OFF + wg * 1024 + 128);
        if (__builtin_expect(idx < 16, 1)) { push_loop<false>(P, P.x, rank, idx, ctr, claims, lane, dbg); TSTAMP(1); __builtin_amdgcn_endpgm(); }
        push_loop<true>(P, P.w, rank, idx - 16, ctr, claims, lane, dbg);
        TSTAMP(1);
        __builtin_amdgcn_endpgm();
    }
    if (wave == 9) {
        __builtin_amdgcn_s_setprio(3);
        const unsigned long long t0 = __builtin_amdgcn_s_memrealtime();
        unsigned* ctr = (unsigned*)(myws + CTR_OFF + wg * 128 + 64);
        unsigned* tmo = (unsigned*)(myws + TMO_OFF);
        const unsigned prev = __hip_atomic_load(ctr, __ATOMIC_RELAXED, __HIP_MEMORY_SCOPE_AGENT);
        const float scale = P.sx[0] * P.sw[0];
        __builtin_amdgcn_s_barrier();
        const unsigned call = __builtin_amdgcn_readfirstlane(prev) + 1;
        __hip_atomic_store(ctr, call, __ATOMIC_RELAXED, __HIP_MEMORY_SCOPE_AGENT);
        const int p = (rank + (lane >> 1)) & 7, o = lane & 1;
        const unsigned* fl = lane < 16 ? (const unsigned*)(myws + FLAG_OFF + (size_t)(((o ? 16 + tj : ti) * NR + p) * 128)) : tmo;
        volatile LAS unsigned* sched = (volatile LAS unsigned*)(L8 + L_SCHED);
        volatile LAS float* lscale = (volatile LAS float*)(L8 + L_SCALE);
        *lscale = scale;
        const unsigned base = call * 32;
        const unsigned code0 = ((call & 1) * 128 + p * 16) + 1;
        int cur = 0, nsched = 0, known = 0;
        bool dead = false;
        auto due = [&](int g) { const unsigned a = SEND_SCHED.t[ti][g] >> 16, b = SEND_SCHED.t[16 + tj][g] >> 16; return (a > b ? a : b) / 4 + POLL_LEAD; };
        const unsigned t_fast = due(15), t_mid = due(NSP ? 16 - 2 * NSP : 15);
        for (;;) {
            if (!dead) {
                unsigned val = 0;
                const bool act_ = lane == 16 || (lane < 16 && known <= cur && cur < 2 * NCH);
                if (act_) val = __hip_atomic_load(fl, __ATOMIC_RELAXED, __HIP_MEMORY_SCOPE_SYSTEM);
                if (__builtin_amdgcn_readlane(val, 16) == call) dead = true;
                if (act_) known = (int)(val - base);
            }
            if (dead) { *lscale = __builtin_nanf(""); known = 31; }
            const int other = __builtin_amdgcn_update_dpp(0, known, 0xB1, 0xf, 0xf, false);
            const int av = known < other ? known : other;
            for (;;) {
                const bool rdy = lane < 16 && av > cur && cur < 2 * NCH;
                const unsigned long long m = __ballot(rdy) & 0x5555ull;
                if (m == 0) break;
                if (rdy && o == 0) sched[nsched + __builtin_popcountll(m & ((1ull << lane) - 1))] = code0 + cur;
                cur += rdy ? 1 : 0;
                nsched += __builtin_popcountll(m);
            }
            if (nsched >= 2 * NSTEP) break;
            const unsigned long long now = __builtin_amdgcn_s_memrealtime() - t0;
            if (now > DEADLINE_TICKS) {
                __hip_atomic_store(tmo, call, __ATOMIC_RELAXED, __HIP_MEMORY_SCOPE_AGENT);
                dead = true;
            } else if ((unsigned)now >= t_fast) {
                __builtin_amdgcn_s_sleep(POLL_FAST);
            } else if ((unsigned)now >= t_mid) {
                __builtin_amdgcn_s_sleep(POLL_MID);
            } else {
                const unsigned until = (unsigned)now + POLL_SLOW < t_mid ? (unsigned)now + POLL_SLOW : t_mid;
                while ((unsigned)(__builtin_amdgcn_s_memrealtime() - t0) < until) __builtin_amdgcn_s_sleep(4);
            }
        }
        TSTAMP(2);
    } else if (wave >= 10) {
        __builtin_amdgcn_s_setprio(3);
        __builtin_amdgcn_s_barrier();
        const int o = wave - 10;
        volatile LAS unsigned* sched = (volatile LAS unsigned*)(L8 + L_SCHED);
        volatile LAS unsigned* freew = (volatile LAS unsigned*)(L8 + L_FREE);
        volatile LAS unsigned* fullw = (volatile LAS unsigned*)(L8 + L_FULL);
        const char* stage_x = myws + (size_t)ti * HALF;
        const char* stage_w = myws + (size_t)(16 + tj) * HALF;
        const unsigned v0 = lane * 16, v1 = v0 + 4096;
        auto dma8 = [&](const char* src, unsigned ldst) __attribute__((always_inline)) {
            unsigned keep;
#define DMA4(V) "global_load_lds_dwordx4 " V ", %[src] sc0 sc1\n\t" \
                "global_load_lds_dwordx4 " V ", %[src] offset:1024 sc0 sc1\n\t" \
                "global_load_lds_dwordx4 " V ", %[src] offset:2048 sc0 sc1\n\t" \
                "global_load_lds_dwordx4 " V ", %[src] offset:3072 sc0 sc1\n\t"
            asm volatile(
                "s_mov_b32 %[keep], m0\n\t"
                "s_mov_b32 m0, %[ld]\n\t"
                "s_nop 0\n\t"
                DMA4("%[v0]")
                "s_add_u32 m0, m0, 4096\n\t"
                "s_nop 0\n\t"
                DMA4("%[v1]")
                "s_mov_b32 m0, %[keep]\n\t"
                : [keep] "=&s"(keep) : [src] "s"(src), [ld] "s"(ldst), [v0] "v"(v0), [v1] "v"(v1) : "memory", "scc");
        };
        int issued = o, pub = o;
        while (pub < NSTEP) {
            bool did = false;
            if (issued < NSTEP && issued - pub < 4) {
                const unsigned codev = sched[2 * issued + (lane & 1)];
                const int fr = (int)freew[lane & 7];
                const unsigned code0 = __builtin_amdgcn_readlane(codev, 0), code1 = __builtin_amdgcn_readlane(codev, 1);
                if (code0 != 0u && code1 != 0u && __all(fr >= issued - 3)) {
                    size_t off0 = (size_t)((code0 - 1) & 255u) * (NBLK * HALF), off1 = (size_t)((code1 - 1) & 255u) * (NBLK * HALF);
                    asm volatile("s_nop 4" : "+s"(off0), "+s"(off1));
                    const unsigned ldst = L_RING + (issued & 3) * 32768;
                    dma8(stage_x + off0, ldst); dma8(stage_w + off0, ldst + 16384); dma8(stage_x + off1, ldst + 8192); dma8(stage_w + off1, ldst + 24576);
                    issued += 2;
                    did = true;
                }
            }
            if (!did || issued - pub >= 4) {
                if (issued > pub) {
                    if (issued - pub == 2) wait_vm<0>(); else wait_vm<32>();
                    fullw[(pub & 3) * 8 + (lane & 7)] = pub + 1;
                    pub += 2;
                } else {
                    __builtin_amdgcn_s_sleep(1);
                }
            }
        }
        if (o == 0) TSTAMP(3);
    } else {
        if (wave == 0) {
            #pragma unroll
            for (int i = 0; i < L_CTRL_BYTES / 256; ++i) ((LAS unsigned*)L8)[i * 64 + lane] = 0u;
            asm volatile("s_waitcnt lgkmcnt(0)" ::: "memory");
            TSTAMP(0);
            CSTAMP(9);
        }
        __builtin_amdgcn_s_barrier();
        if (wave >= 4) __builtin_amdgcn_s_setprio(1);
        const int wm = wave >> 1, wn = wave & 1;
        const int r = lane & 31, h = lane >> 5;
        unsigned va = L_RING + h * 8192 + wm * 1024 + r * 16;
        unsigned vb = L_RING + 16384 + h * 8192 + wn * 512 + r * 16;
        f32x16 c00 = {}, c01 = {}, c02 = {}, c03 = {}, c10 = {}, c11 = {}, c12 = {}, c13 = {};
        unsigned vf, st, need, keep;
        const unsigned long long mask = 1ull << wave;
        float* outp = P.out + (size_t)(ti * 256 + wm * 32) * DIM + tj * 256 + wn * 128;
        asm volatile("" : "+s"(outp));
        asm volatile(
            "s_mov_b32 %[keep], m0\n\t"
            "s_mov_b32 m0, 0\n\t"
            "s_mov_b32 %[need], 1\n\t"
            "ds_read_addtid_b32 %[vf] offset:0\n\t"
            "3:\n\t"
            STEP_FULL(0, 0, 1)
            STEP_FULL(32768, 1, 2)
            FLIP
            STEP_FULL(0, 2, 3)
            STEP_FULL(32768, 3, 0)
            FLIP
            "s_cmp_le_u32 %[need], %[nfull]\n\t"
            "s_cbranch_scc1 3b\n\t"
            WAITL(0)
            "s_mov_b32 m0, %[keep]\n\t"
            "s_nop 15\n\t"
            "s_nop 7\n\t"
            : [c00] "+v"(c00), [c01] "+v"(c01), [c02] "+v"(c02), [c03] "+v"(c03),
              [c10] "+v"(c10), [c11] "+v"(c11), [c12] "+v"(c12), [c13] "+v"(c13),
              [va] "+v"(va), [vb] "+v"(vb), [vf] "=&v"(vf), [st] "=&s"(st), [need] "=&s"(need), [keep] "=&s"(keep)
            : [sw] "s"(wave), [mask] "s"(mask), [nfull] "i"(NSTEP)
            : "memory", "scc", FRAG_CLOBBERS);
        if (wave == 0) { TSTAMP(8); CSTAMP(10); }
        const int lz = __builtin_amdgcn_mbcnt_hi(~0u, __builtin_amdgcn_mbcnt_lo(~0u, 0u));
        const __amdgpu_buffer_rsrc_t outr = make_rsrc(outp, 160u * DIM * 4u);
        const int voff = ((lz >> 5) * 4 * DIM + (lz & 31) * 4) * 4;
        const float sc = __builtin_bit_cast(float, __builtin_amdgcn_readfirstlane(*(volatile LAS unsigned*)(L8 + L_SCALE)));
        auto act = [&](float a) { return __builtin_bit_cast(unsigned, __builtin_elementwise_maximum(a * sc, 0.f)); };
        #pragma unroll
        for (int g = 0; g < 16; ++g) {
            u32x4 v0 = {act(c00[g]), act(c01[g]), act(c02[g]), act(c03[g])};
            store16_wt<OUT_AUX>(v0, outr, voff, ((g & 3) + 8 * (g >> 2)) * DIM * 4);
            u32x4 v1 = {act(c10[g]), act(c11[g]), act(c12[g]), act(c13[g])};
            store16_wt<OUT_AUX>(v1, outr, voff, (128 + (g & 3) + 8 * (g >> 2)) * DIM * 4);
        }
    }
}

}

extern "C" void kernel_launch(int rank, int nranks, ncclComm_t comm,
                              hipStream_t stream,
                              void* const* d_in, const int* in_sizes, int n_in,
                              void* d_out, int out_size,
                              const int* in_dtypes, int out_dtype,
                              void* const* d_out_all,
                              void* const* d_ws_all, size_t ws_bytes) {
    if (nranks != NR || n_in < 4 || ws_bytes < WS_NEED || (unsigned)rank >= (unsigned)NR ||
        in_sizes[0] != DIM * KLOC || in_sizes[1] != DIM * KLOC || out_size != DIM * DIM) {
        fprintf(stderr, "kernel_launch: unexpected problem shape\n");
        return;
    }
    Params P;
    P.x = (const unsigned char*)d_in[0];
    P.w = (const unsigned char*)d_in[1];
    P.sx = (const float*)d_in[2];
    P.sw = (const float*)d_in[3];
    P.out = (float*)d_out;
    P.myws = (char*)d_ws_all[rank];
    {
        const int quad[NR] = {0, 1, 0, 0, 1, 1, 0, 1};
        int n = 0;
        for (int pass = 0; pass < 2; ++pass)
            for (int q = 0; q < NR; ++q) {
                const int p = (rank + q) % NR;
                if ((quad[p] == quad[rank]) == (pass == 0)) P.dws[n++] = (char*)d_ws_all[p];
            }
    }
    P.rank = rank;
    hipLaunchKernelGGL(fused_ag_gemm, dim3(256), dim3(768), 0, stream, P);
    HIP_CHECK(hipGetLastError());
}
```

```cpp
#include <hip/hip_runtime.h>
#include <rccl/rccl.h>
#include <cstdio>
#include <cstdint>
#include <type_traits>

#define HIP_CHECK(expr) do { hipError_t _e = (expr); if (_e != hipSuccess) { \
    fprintf(stderr, "HIP error %s at %s:%d\n", hipGetErrorString(_e), __FILE__, __LINE__); \
    return; } } while (0)

#define TAU16 440
#define TAU_AT 3700
#define TAU_ADD 30
#define LAGW 4
#define ORDER 0
#define PW0 9
#define PW_M 12
#define TOFF 80
#define S_FULL 31
#define S_LAST 7
#define NSP 2
#define S_HALF 13
#define POLL_MID 6
#define S_PREV 31
#define SKEW_L 24
#define SKEW_G 3
#define DCAP 2
#define DCAP2 8
#define FLAG_AT 3
#define FLAG_DEFER 500
#define CAP_SPAN 1200
#define OUT_AUX 17
#define PUSH_AUX 17
#define SRC_AUX 0
#define POLL_SLOW 200
#define POLL_FAST 2
#define POLL_LEAD 90

namespace {

typedef unsigned u32x4 __attribute__((ext_vector_type(4)));
typedef float f32x16 __attribute__((ext_vector_type(16)));
#define LAS __attribute__((address_space(3)))

constexpr int NR = 8;
constexpr int DIM = 4096;
constexpr int KLOC = DIM / NR;
constexpr int NCH = 8;
constexpr int NBLK = 32;
constexpr int NSTEP = NR * NCH;
constexpr unsigned ITEM = 16384;
constexpr unsigned HALF = 8192;
constexpr size_t STAGE_PAR = (size_t)NR * NCH * NBLK * ITEM;
constexpr size_t FLAG_OFF = 2 * STAGE_PAR;
constexpr size_t CTR_OFF = FLAG_OFF + 32768;
constexpr size_t TMO_OFF = CTR_OFF + 32768;
constexpr size_t DBG_OFF = TMO_OFF + 4096;
constexpr size_t WS_NEED = DBG_OFF + 257 * 1024;
constexpr unsigned long long DEADLINE_TICKS = 50000000ull;

constexpr int L_FULL = 0;
constexpr int L_FREE = 512;
constexpr int L_SCHED = 1024;
constexpr int L_SCALE = 1536;
constexpr int L_CTRL_BYTES = 2048;
constexpr int L_RING = 8192;
constexpr int L_TOTAL = L_RING + 4 * 32768;

struct Params {
    const unsigned char* x;
    const unsigned char* w;
    const float* sx;
    const float* sw;
    float* out;
    char* myws;
    char* dws[NR];
    int rank;
};

struct Sched { unsigned t[NBLK][16]; };
constexpr int blk_pos(int b) {
    return 2 * (b & 15) + (b >> 4) + 1;
}
constexpr int grp_end(int b, int g) {
    const int u = blk_pos(b), c = g >> 1;
    const int base = c < NCH - NSP ? (c * 32 + u) * 16 : g * 256 + u * 8;
    const int lead = g + 1 + 2 * SKEW_G - 16 > 0 ? g + 1 + 2 * SKEW_G - 16 : 0;
    return base - lead * SKEW_L * (32 - u) / (2 * SKEW_G);
}
constexpr int store_key(int b, int g, int u) {
    const int c = g >> 1;
    if (c < NCH - NSP) {
        const int gap = (grp_end(b, g | 1) - grp_end(b, (g | 1) - 2)) / 16 - 1;
        int stride = c == NCH - 1 ? S_LAST : c >= NCH - NSP - 1 ? S_PREV : S_FULL;
        if (stride > gap) stride = gap;
        return grp_end(b, g | 1) - (15 - (g & 1) * 8 - u) * stride + 1024;
    }
    const int gap = (grp_end(b, g) - grp_end(b, g - 1)) / 8 - 1;
    int stride = g == 15 ? S_LAST : S_HALF;
    if (stride > gap) stride = gap;
    return grp_end(b, g) - (7 - u) * stride + 1024;
}
constexpr unsigned slot_time(unsigned n) { return TOFF * 4 + (n * TAU16 + (n > TAU_AT ? (n - TAU_AT) * TAU_ADD : 0)) / 64; }
constexpr Sched make_sched() {
    Sched r = {};
    unsigned short cnt[5184] = {};
    for (int b = 0; b < NBLK; ++b) for (int g = 0; g < 16; ++g) for (int u = 0; u < 8; ++u) ++cnt[store_key(b, g, u) + 1];
    for (int k = 1; k < 5184; ++k) cnt[k] += cnt[k - 1];
    for (int b = 0; b < NBLK; ++b)
        for (int g = 0; g < 16; ++g) {
            unsigned first = 0, last = 0;
            for (int u = 0; u < 8; ++u) { last = cnt[store_key(b, g, u)]++; if (u == 0) first = last; }
            r.t[b][g] = slot_time(first) | (slot_time(last) << 16);
        }
    return r;
}
__device__ constexpr Sched SEND_SCHED = make_sched();

__device__ __forceinline__ __amdgpu_buffer_rsrc_t make_rsrc(const void* p, unsigned bytes) {
    return __builtin_amdgcn_make_buffer_rsrc((void*)p, 0, (int)bytes, 0x00020000);
}

template <int AUX = 17>
__device__ __forceinline__ void store16_wt(u32x4 v, __amdgpu_buffer_rsrc_t r, int voff, int soff) {
    __builtin_amdgcn_raw_buffer_store_b128(v, r, voff, soff, AUX);
    __builtin_amdgcn_sched_barrier(0);
    asm volatile("s_nop 1" ::: "memory");
    __builtin_amdgcn_sched_barrier(0);
}

template <int I, int N, typename F> __device__ __forceinline__ void static_for(F&& f) {
    if constexpr (I < N) { f(std::integral_constant<int, I>{}); static_for<I + 1, N>(f); }
}

template <int N> __device__ __forceinline__ void wait_vm() {
    static_assert(N >= 0 && N <= 63, "");
    asm volatile("s_waitcnt vmcnt(%0)" :: "i"(N) : "memory");
}

#define CSTAMP(idx) do {} while (0)
#define TSTAMP(idx) do {} while (0)

template <bool IS_W>
__device__ __forceinline__ void push_loop(const Params& P, const unsigned char* srcp, int rank, int blk,
                                          unsigned* ctr, unsigned* claims, int lane, unsigned* dbg) {
    const unsigned t0 = (unsigned)__builtin_amdgcn_s_memrealtime();
    const __amdgpu_buffer_rsrc_t src = make_rsrc(srcp, DIM * KLOC);
    const int voff_ld = IS_W ? lane * 4 : (lane & 31) * KLOC + (lane >> 5) * 16;
    unsigned rawA[32], rawB[32];
    auto load_w = [&](int g, unsigned (&raw)[32]) __attribute__((always_inline)) {
        const int so = g * 32 * DIM + blk * 256;
        #pragma unroll
        for (int k = 0; k < 32; ++k) raw[k] = __builtin_amdgcn_raw_buffer_load_b32(src, voff_ld, so + k * DIM, SRC_AUX);
    };
    auto load_x = [&](int c) __attribute__((always_inline)) {
        const int so = blk * 256 * KLOC + c * 64;
        #pragma unroll
        for (int t = 0; t < 8; ++t) {
            u32x4 v = __builtin_amdgcn_raw_buffer_load_b128(src, voff_ld, so + t * 32 * KLOC, SRC_AUX);
            rawA[4 * t] = v.x; rawA[4 * t + 1] = v.y; rawA[4 * t + 2] = v.z; rawA[4 * t + 3] = v.w;
            v = __builtin_amdgcn_raw_buffer_load_b128(src, voff_ld, so + t * 32 * KLOC + 32, SRC_AUX);
            rawB[4 * t] = v.x; rawB[4 * t + 1] = v.y; rawB[4 * t + 2] = v.z; rawB[4 * t + 3] = v.w;
        }
    };
    auto claim_add = [&](int sl, unsigned d) __attribute__((always_inline)) {
        unsigned old;
        asm volatile("s_mov_b64 exec, 1\n\ts_nop 4\n\tglobal_atomic_add %0, %1, %2, %3 sc0\n\ts_mov_b64 exec, -1"
                     : "=&v"(old) : "v"(sl * 128), "v"(d), "s"(claims) : "memory");
        return old;
    };
    auto claim_wait = [&](unsigned& v) __attribute__((always_inline)) {
        asm volatile("s_waitcnt vmcnt(0)" : "+v"(v) :: "memory");
        return (unsigned)__builtin_amdgcn_readfirstlane(v);
    };
    unsigned xcc;
    asm volatile("s_getreg_b32 %0, hwreg(HW_REG_XCC_ID)" : "=s"(xcc));
    int e = (int)(xcc & 7u);
    const unsigned prev = __hip_atomic_load(ctr, __ATOMIC_RELAXED, __HIP_MEMORY_SCOPE_AGENT);
    unsigned got = claim_add(e, 1u);
    if constexpr (IS_W) load_w(0, rawA); else load_x(0);
    char* dws = P.dws[e];
    const unsigned* tsched = SEND_SCHED.t[(IS_W ? 16 : 0) | blk];
    { unsigned warm = tsched[0]; asm volatile("" :: "s"(warm), "s"(dws)); }
    __builtin_amdgcn_s_barrier();
    unsigned gs = claim_wait(got);
    const unsigned call = __builtin_amdgcn_readfirstlane(prev) + 1;
    #pragma nounroll
    for (int tries = 0; tries < 7 && gs != call - 1; ++tries) {
        got = claim_add(e, ~0u);
        claim_wait(got);
        e = (e + 1) & 7;
        got = claim_add(e, 1u);
        gs = claim_wait(got);
        dws = P.dws[e];
    }
    __hip_atomic_store(ctr, call, __ATOMIC_RELAXED, __HIP_MEMORY_SCOPE_AGENT);
    const bool paced = e != 0;
    const __amdgpu_buffer_rsrc_t dstr = make_rsrc(dws + (size_t)(call & 1) * STAGE_PAR + (size_t)rank * (NCH * NBLK * ITEM), NCH * NBLK * ITEM);
    const __amdgpu_buffer_rsrc_t flr = make_rsrc(dws + FLAG_OFF, 32768);
    const int voff_st = lane * 16;
    const int voff_fl = lane < 8 ? lane * 4 : 0x7ffffff0;
    const int b = blk + (IS_W ? 16 : 0);
    int lag = 0;
    unsigned pend = 0;
    auto send_grp = [&](int g, unsigned (&raw)[32], auto first, auto odd) __attribute__((always_inline)) {
        u32x4 o[8];
        if constexpr (IS_W) {
            #pragma unroll
            for (int r = 0; r < 8; ++r) {
                const unsigned t0_ = __builtin_amdgcn_perm(raw[4 * r + 1], raw[4 * r], 0x05010400u);
                const unsigned t1_ = __builtin_amdgcn_perm(raw[4 * r + 1], raw[4 * r], 0x07030602u);
                const unsigned t2_ = __builtin_amdgcn_perm(raw[4 * r + 3], raw[4 * r + 2], 0x05010400u);
                const unsigned t3_ = __builtin_amdgcn_perm(raw[4 * r + 3], raw[4 * r + 2], 0x07030602u);
                o[0 + (r >> 2)][r & 3] = __builtin_amdgcn_perm(t2_, t0_, 0x05040100u);
                o[2 + (r >> 2)][r & 3] = __builtin_amdgcn_perm(t2_, t0_, 0x07060302u);
                o[4 + (r >> 2)][r & 3] = __builtin_amdgcn_perm(t3_, t1_, 0x05040100u);
                o[6 + (r >> 2)][r & 3] = __builtin_amdgcn_perm(t3_, t1_, 0x07060302u);
            }
        } else {
            #pragma unroll
            for (int t = 0; t < 8; ++t) o[t] = u32x4{raw[4 * t], raw[4 * t + 1], raw[4 * t + 2], raw[4 * t + 3]};
        }
        #pragma unroll
        for (int u = 0; u < 8; ++u) asm volatile("" : "+v"(o[u]));
        const int so = (g * NBLK + b) * HALF;
        const unsigned ts = tsched[g];
        const int span = (ts >> 16) - (ts & 0xffffu);
        const bool capped = paced && span >= CAP_SPAN;
        const int ta = (ts & 0xffffu) * 7 + (capped ? 0 : lag);
        static_for<0, 8>([&](auto uc) __attribute__((always_inline)) {
            constexpr int u = decltype(uc)::value, L = IS_W ? 32 : (decltype(odd)::value ? 16 : 0), LU = decltype(first)::value ? 4 : 0;
            if (paced) {
                const int tgt = ta + span * u;
                for (;;) {
                    const int left = tgt - (int)((unsigned)__builtin_amdgcn_s_memrealtime() - t0) * 28;
                    if (left <= 0) break;
                    if (left > 40 * 28) __builtin_amdgcn_s_sleep(8); else if (left > 6 * 28) __builtin_amdgcn_s_sleep(1);
                }
            }
            if (capped) wait_vm<DCAP - 1 + ((u > LU && u <= LU + DCAP) ? L : 0)>();
            else if (DCAP2 < 8 && paced) wait_vm<DCAP2 - 1 + ((u > LU && u <= LU + DCAP2) ? L : 0)>();
            if (u == FLAG_AT && pend) {
                if (g < 14) wait_vm<FLAG_AT + L>(); else wait_vm<FLAG_AT>();
                __builtin_amdgcn_raw_buffer_store_b32(pend, flr, voff_fl, (b * NR + rank) * 128, 17);
                pend = 0;
            }
            if (capped && (u == 3 || u == 7)) {
                const int l_ = (int)((unsigned)__builtin_amdgcn_s_memrealtime() - t0) * 28 - (ta + span * u);
                lag += (l_ - lag) * LAGW / 16;
            }
            store16_wt<PUSH_AUX>(o[u], dstr, voff_st, so + u * 1024);
            if constexpr (IS_W) {
                if (decltype(first)::value && u == 0) load_w(1, rawB);
                if (u == LU && g < 14) load_w(g + 2, raw);
            } else if (decltype(odd)::value && u == 0 && g < 14) {
                load_x((g + 1) >> 1);
            }
        });
        if (decltype(odd)::value || g >= 16 - 2 * NSP) {
            const unsigned nx = g < 15 ? tsched[g + 1] : 0xffffffffu;
            if (FLAG_AT > 0 && paced && (int)((nx & 0xffffu) + ((nx >> 16) - (nx & 0xffffu)) * FLAG_AT / 7 - (ts >> 16)) < FLAG_DEFER) {
                pend = call * 32 + g + 1;
            } else {
                wait_vm<0>();
                __builtin_amdgcn_raw_buffer_store_b32(call * 32 + g + 1, flr, voff_fl, (b * NR + rank) * 128, 17);
            }
        }
    };
    send_grp(0, rawA, std::true_type{}, std::false_type{});
    send_grp(1, rawB, std::false_type{}, std::true_type{});
    #pragma nounroll
    for (int g = 2; g < 16; g += 2) {
        send_grp(g, rawA, std::false_type{}, std::false_type{});
        send_grp(g + 1, rawB, std::false_type{}, std::true_type{});
    }
    wait_vm<0>();
}

#define MFMA(c, a, b) "v_mfma_f32_32x32x64_f8f6f4 %[" #c "], " a ", " b ", %[" #c "] cbsz:0 blgp:1\n\t"
#define X0 "v[136:143]"
#define X1 "v[144:151]"
#define Y0 "v[152:159]"
#define Y1 "v[160:167]"
#define STR2(x) #x
#define STR(x) STR2(x)
#define RD(lo, hi, addr, off1, off2) \
    "ds_read_b128 " lo ", %[" #addr "] offset:" off1 "\n\t" \
    "ds_read_b128 " hi ", %[" #addr "] offset:" off2 "\n\t"
#define RD_X0(addr, o1, o2) RD("v[136:139]", "v[140:143]", addr, o1, o2)
#define RD_X1(addr, o1, o2) RD("v[144:147]", "v[148:151]", addr, o1, o2)
#define RD_Y0(addr, o1, o2) RD("v[152:155]", "v[156:159]", addr, o1, o2)
#define RD_Y1(addr, o1, o2) RD("v[160:163]", "v[164:167]", addr, o1, o2)
#define WAITL(n) "s_waitcnt lgkmcnt(" #n ")\n\t"
#define CHECK(FS) \
    "1:\n\t" \
    WAITL(0) \
    "v_readlane_b32 %[st], %[vf], %[sw]\n\t" \
    "s_nop 0\n\t" \
    "s_cmp_ge_u32 %[st], %[need]\n\t" \
    "s_cbranch_scc1 2f\n\t" \
    "s_sleep 1\n\t" \
    "ds_read_addtid_b32 %[vf] offset:" STR(FS) "*32\n\t" \
    "s_branch 1b\n\t" \
    "2:\n\t"
#define RELEASE(FS, NX) \
    WAITL(0) \
    "s_mov_b64 exec, %[mask]\n\t" \
    "ds_write_addtid_b32 %[vf] offset:512\n\t" \
    "s_mov_b64 exec, -1\n\t" \
    "ds_read_addtid_b32 %[vf] offset:" STR(NX) "*32\n\t"
#define PRETOUCH \
    "s_sub_u32 %[st], %[need], %[pw0]\n\t" \
    "s_cmp_lt_u32 %[st], 32\n\t" \
    "s_cbranch_scc0 6f\n\t" \
    "s_and_b32 %[t1], %[st], 12\n\t" \
    "s_add_u32 %[t1], %[t1], %[st]\n\t" \
    "s_and_b32 %[st], %[st], 16\n\t" \
    "s_mul_i32 %[st], %[st], 7\n\t" \
    "s_add_u32 %[t1], %[t1], %[st]\n\t" \
    "s_lshl_b32 %[t1], %[t1], 14\n\t" \
    "buffer_store_dwordx4 v[160:163], %[vo], %[srd], %[t1] offen " PW_SC "\n\t" \
    "6:\n\t"
#define PW_SC "sc0 sc1"
#define STEP_FULL(SO, FS, NX) \
    CHECK(FS) \
    RD_X0(va, STR(SO), STR(SO) "+512") \
    RD_Y0(vb, STR(SO), STR(SO) "+1024") \
    RD_X1(va, STR(SO) "+4096", STR(SO) "+4608") \
    RD_Y1(vb, STR(SO) "+2048", STR(SO) "+3072") \
    WAITL(4) \
    MFMA(c00, X0, Y0) \
    WAITL(2) \
    MFMA(c10, X1, Y0) \
    RD_Y0(vb, STR(SO) "+4096", STR(SO) "+5120") \
    WAITL(2) \
    MFMA(c01, X0, Y1) \
    MFMA(c11, X1, Y1) \
    RD_Y1(vb, STR(SO) "+6144", STR(SO) "+7168") \
    WAITL(2) \
    MFMA(c02, X0, Y0) \
    MFMA(c12, X1, Y0) \
    RELEASE(FS, NX) \
    MFMA(c03, X0, Y1) \
    PRETOUCH \
    "s_add_u32 %[need], %[need], 1\n\t" \
    MFMA(c13, X1, Y1)
#define FLIP \
    "v_xor_b32 %[va], 0x10000, %[va]\n\t" \
    "v_xor_b32 %[vb], 0x10000, %[vb]\n\t"
#define FRAG_CLOBBERS \
    "v136", "v137", "v138", "v139", "v140", "v141", "v142", "v143", "v144", "v145", "v146", "v147", \
    "v148", "v149", "v150", "v151", "v152", "v153", "v154", "v155", "v156", "v157", "v158", "v159", \
    "v160", "v161", "v162", "v163", "v164", "v165", "v166", "v167"

__global__ void __launch_bounds__(768, 3) fused_ag_gemm(Params P) {
    __shared__ __attribute__((aligned(1024))) unsigned char lds[L_TOTAL];
    LAS unsigned char* L8 = (LAS unsigned char*)lds;
    const int wave = __builtin_amdgcn_readfirstlane(threadIdx.x >> 6);
    const int lane = threadIdx.x & 63;
    const int wg = blockIdx.x;
    const int e = wg & 7, idx = wg >> 3;
    const int ti = (e >> 1) + 4 * (idx >> 3);
    const int tj = (e & 1) + 2 * (idx & 7);
    char* const myws = P.myws;
    const int rank = P.rank;

    int wv_ = wave;
    asm volatile("" : "+s"(wv_));
    if (__builtin_expect(wv_ == 8, 1)) {
        __builtin_amdgcn_s_setprio(3);
        unsigned* ctr = (unsigned*)(myws + CTR_OFF + wg * 128);
        unsigned* claims = (unsigned*)(myws + CTR_OFF + idx * 8 * 128 + 32);
        unsigned* dbg = (unsigned*)(myws + DBG_OFF + wg * 1024 + 128);
        if (__builtin_expect(idx < 16, 1)) { push_loop<false>(P, P.x, rank, idx, ctr, claims, lane, dbg); TSTAMP(1); __builtin_amdgcn_endpgm(); }
        push_loop<true>(P, P.w, rank, idx - 16, ctr, claims, lane, dbg);
        TSTAMP(1);
        __builtin_amdgcn_endpgm();
    }
    if (wave == 9) {
        __builtin_amdgcn_s_setprio(3);
        const unsigned long long t0 = __builtin_amdgcn_s_memrealtime();
        unsigned* ctr = (unsigned*)(myws + CTR_OFF + wg * 128 + 64);
        unsigned* tmo = (unsigned*)(myws + TMO_OFF);
        const unsigned prev = __hip_atomic_load(ctr, __ATOMIC_RELAXED, __HIP_MEMORY_SCOPE_AGENT);
        const float scale = P.sx[0] * P.sw[0];
        __builtin_amdgcn_s_barrier();
        const unsigned call = __builtin_amdgcn_readfirstlane(prev) + 1;
        __hip_atomic_store(ctr, call, __ATOMIC_RELAXED, __HIP_MEMORY_SCOPE_AGENT);
        const int p = (rank + (lane >> 1)) & 7, o = lane & 1;
        const unsigned* fl = lane < 16 ? (const unsigned*)(myws + FLAG_OFF + (size_t)(((o ? 16 + tj : ti) * NR + p) * 128)) : tmo;
        volatile LAS unsigned* sched = (volatile LAS unsigned*)(L8 + L_SCHED);
        volatile LAS float* lscale = (volatile LAS float*)(L8 + L_SCALE);
        *lscale = scale;
        const unsigned base = call * 32;
        const unsigned code0 = ((call & 1) * 128 + p * 16) + 1;
        int cur = 0, nsched = 0, known = 0;
        bool dead = false;
        auto due = [&](int g) { const unsigned a = SEND_SCHED.t[ti][g] >> 16, b = SEND_SCHED.t[16 + tj][g] >> 16; return (a > b ? a : b) / 4 + POLL_LEAD; };
        const unsigned t_fast = due(15), t_mid = due(NSP ? 16 - 2 * NSP : 15);
        for (;;) {
            if (!dead) {
                unsigned val = 0;
                const bool act_ = lane == 16 || (lane < 16 && known <= cur && cur < 2 * NCH);
                if (act_) val = __hip_atomic_load(fl, __ATOMIC_RELAXED, __HIP_MEMORY_SCOPE_SYSTEM);
                if (__builtin_amdgcn_readlane(val, 16) == call) dead = true;
                if (act_) known = (int)(val - base);
            }
            if (dead) { *lscale = __builtin_nanf(""); known = 31; }
            const int other = __builtin_amdgcn_update_dpp(0, known, 0xB1, 0xf, 0xf, false);
            const int av = known < other ? known : other;
            for (;;) {
                const bool rdy = lane < 16 && av > cur && cur < 2 * NCH;
                const unsigned long long m = __ballot(rdy) & 0x5555ull;
                if (m == 0) break;
                if (rdy && o == 0) sched[nsched + __builtin_popcountll(m & ((1ull << lane) - 1))] = code0 + cur;
                cur += rdy ? 1 : 0;
                nsched += __builtin_popcountll(m);
            }
            if (nsched >= 2 * NSTEP) break;
            const unsigned long long now = __builtin_amdgcn_s_memrealtime() - t0;
            if (now > DEADLINE_TICKS) {
                __hip_atomic_store(tmo, call, __ATOMIC_RELAXED, __HIP_MEMORY_SCOPE_AGENT);
                dead = true;
            } else if ((unsigned)now >= t_fast) {
                __builtin_amdgcn_s_sleep(POLL_FAST);
            } else if ((unsigned)now >= t_mid) {
                __builtin_amdgcn_s_sleep(POLL_MID);
            } else {
                const unsigned until = (unsigned)now + POLL_SLOW < t_mid ? (unsigned)now + POLL_SLOW : t_mid;
                while ((unsigned)(__builtin_amdgcn_s_memrealtime() - t0) < until) __builtin_amdgcn_s_sleep(4);
            }
        }
        TSTAMP(2);
    } else if (wave >= 10) {
        __builtin_amdgcn_s_setprio(3);
        __builtin_amdgcn_s_barrier();
        const int o = wave - 10;
        volatile LAS unsigned* sched = (volatile LAS unsigned*)(L8 + L_SCHED);
        volatile LAS unsigned* freew = (volatile LAS unsigned*)(L8 + L_FREE);
        volatile LAS unsigned* fullw = (volatile LAS unsigned*)(L8 + L_FULL);
        const char* stage_x = myws + (size_t)ti * HALF;
        const char* stage_w = myws + (size_t)(16 + tj) * HALF;
        const unsigned v0 = lane * 16, v1 = v0 + 4096;
        auto dma8 = [&](const char* src, unsigned ldst) __attribute__((always_inline)) {
            unsigned keep;
#define DMA4(V) "global_load_lds_dwordx4 " V ", %[src] sc0 sc1\n\t" \
                "global_load_lds_dwordx4 " V ", %[src] offset:1024 sc0 sc1\n\t" \
                "global_load_lds_dwordx4 " V ", %[src] offset:2048 sc0 sc1\n\t" \
                "global_load_lds_dwordx4 " V ", %[src] offset:3072 sc0 sc1\n\t"
            asm volatile(
                "s_mov_b32 %[keep], m0\n\t"
                "s_mov_b32 m0, %[ld]\n\t"
                "s_nop 0\n\t"
                DMA4("%[v0]")
                "s_add_u32 m0, m0, 4096\n\t"
                "s_nop 0\n\t"
                DMA4("%[v1]")
                "s_mov_b32 m0, %[keep]\n\t"
                : [keep] "=&s"(keep) : [src] "s"(src), [ld] "s"(ldst), [v0] "v"(v0), [v1] "v"(v1) : "memory", "scc");
        };
        int issued = o, pub = o;
        while (pub < NSTEP) {
            bool did = false;
            if (issued < NSTEP && issued - pub < 4) {
                const unsigned codev = sched[2 * issued + (lane & 1)];
                const int fr = (int)freew[lane & 7];
                const unsigned code0 = __builtin_amdgcn_readlane(codev, 0), code1 = __builtin_amdgcn_readlane(codev, 1);
                if (code0 != 0u && code1 != 0u && __all(fr >= issued - 3)) {
                    size_t off0 = (size_t)((code0 - 1) & 255u) * (NBLK * HALF), off1 = (size_t)((code1 - 1) & 255u) * (NBLK * HALF);
                    asm volatile("s_nop 4" : "+s"(off0), "+s"(off1));
                    const unsigned ldst = L_RING + (issued & 3) * 32768;
                    dma8(stage_x + off0, ldst); dma8(stage_w + off0, ldst + 16384); dma8(stage_x + off1, ldst + 8192); dma8(stage_w + off1, ldst + 24576);
                    issued += 2;
                    did = true;
                }
            }
            if (!did || issued - pub >= 4) {
                if (issued > pub) {
                    if (issued - pub == 2) wait_vm<0>(); else wait_vm<32>();
                    fullw[(pub & 3) * 8 + (lane & 7)] = pub + 1;
                    pub += 2;
                } else {
                    __builtin_amdgcn_s_sleep(1);
                }
            }
        }
        if (o == 0) TSTAMP(3);
    } else {
        if (wave == 0) {
            #pragma unroll
            for (int i = 0; i < L_CTRL_BYTES / 256; ++i) ((LAS unsigned*)L8)[i * 64 + lane] = 0u;
            asm volatile("s_waitcnt lgkmcnt(0)" ::: "memory");
            TSTAMP(0);
            CSTAMP(9);
        }
        __builtin_amdgcn_s_barrier();
        if (wave >= 4) __builtin_amdgcn_s_setprio(1);
        const int wm = wave >> 1, wn = wave & 1;
        const int r = lane & 31, h = lane >> 5;
        unsigned va = L_RING + h * 8192 + wm * 1024 + r * 16;
        unsigned vb = L_RING + 16384 + h * 8192 + wn * 512 + r * 16;
        f32x16 c00 = {}, c01 = {}, c02 = {}, c03 = {}, c10 = {}, c11 = {}, c12 = {}, c13 = {};
        unsigned vf, st, need, keep;
        const unsigned long long mask = 1ull << wave;
        float* outp = P.out + (size_t)(ti * 256 + wm * 32) * DIM + tj * 256 + wn * 128;
        asm volatile("" : "+s"(outp));
        unsigned t1;
        const unsigned pw0 = (ti > tj ? ti : tj) >= PW_M ? PW0 : 1000u;
        const u32x4 srd = {(unsigned)(size_t)outp, (unsigned)((size_t)outp >> 32) & 0xffffu, 160u * DIM * 4u, 0x00020000u};
        const unsigned vo = (h * 4 * DIM + r * 4) * 4;
        asm volatile(
            "s_mov_b32 %[keep], m0\n\t"
            "s_mov_b32 m0, 0\n\t"
            "s_mov_b32 %[need], 1\n\t"
            "ds_read_addtid_b32 %[vf] offset:0\n\t"
            "3:\n\t"
            STEP_FULL(0, 0, 1)
            STEP_FULL(32768, 1, 2)
            FLIP
            STEP_FULL(0, 2, 3)
            STEP_FULL(32768, 3, 0)
            FLIP
            "s_cmp_le_u32 %[need], %[nfull]\n\t"
            "s_cbranch_scc1 3b\n\t"
            "s_waitcnt vmcnt(0) lgkmcnt(0)\n\t"
            "s_mov_b32 m0, %[keep]\n\t"
            "s_nop 15\n\t"
            "s_nop 7\n\t"
            : [c00] "+v"(c00), [c01] "+v"(c01), [c02] "+v"(c02), [c03] "+v"(c03),
              [c10] "+v"(c10), [c11] "+v"(c11), [c12] "+v"(c12), [c13] "+v"(c13),
              [va] "+v"(va), [vb] "+v"(vb), [vf] "=&v"(vf), [st] "=&s"(st), [need] "=&s"(need), [keep] "=&s"(keep), [t1] "=&s"(t1)
            : [sw] "s"(wave), [mask] "s"(mask), [nfull] "i"(NSTEP), [srd] "s"(srd), [vo] "v"(vo), [pw0] "s"(pw0)
            : "memory", "scc", FRAG_CLOBBERS);
        if (wave == 0) { TSTAMP(8); CSTAMP(10); }
        const int lz = __builtin_amdgcn_mbcnt_hi(~0u, __builtin_amdgcn_mbcnt_lo(~0u, 0u));
        const __amdgpu_buffer_rsrc_t outr = make_rsrc(outp, 160u * DIM * 4u);
        const int voff = ((lz >> 5) * 4 * DIM + (lz & 31) * 4) * 4;
        const float sc = __builtin_bit_cast(float, __builtin_amdgcn_readfirstlane(*(volatile LAS unsigned*)(L8 + L_SCALE)));
        auto act = [&](float a) { return __builtin_bit_cast(unsigned, __builtin_elementwise_maximum(a * sc, 0.f)); };
        #pragma unroll
        for (int g = 0; g < 16; ++g) {
            u32x4 v0 = {act(c00[g]), act(c01[g]), act(c02[g]), act(c03[g])};
            store16_wt<OUT_AUX>(v0, outr, voff, ((g & 3) + 8 * (g >> 2)) * DIM * 4);
            u32x4 v1 = {act(c10[g]), act(c11[g]), act(c12[g]), act(c13[g])};
            store16_wt<OUT_AUX>(v1, outr, voff, (128 + (g & 3) + 8 * (g >> 2)) * DIM * 4);
        }
    }
}

}

extern "C" void kernel_launch(int rank, int nranks, ncclComm_t comm,
                              hipStream_t stream,
                              void* const* d_in, const int* in_sizes, int n_in,
                              void* d_out, int out_size,
                              const int* in_dtypes, int out_dtype,
                              void* const* d_out_all,
                              void* const* d_ws_all, size_t ws_bytes) {
    if (nranks != NR || n_in < 4 || ws_bytes < WS_NEED || (unsigned)rank >= (unsigned)NR ||
        in_sizes[0] != DIM * KLOC || in_sizes[1] != DIM * KLOC || out_size != DIM * DIM) {
        fprintf(stderr, "kernel_launch: unexpected problem shape\n");
        return;
    }
    Params P;
    P.x = (const unsigned char*)d_in[0];
    P.w = (const unsigned char*)d_in[1];
    P.sx = (const float*)d_in[2];
    P.sw = (const float*)d_in[3];
    P.out = (float*)d_out;
    P.myws = (char*)d_ws_all[rank];
    {
        const int quad[NR] = {0, 1, 0, 0, 1, 1, 0, 1};
        int n = 0;
        for (int pass = 0; pass < 2; ++pass)
            for (int q = 0; q < NR; ++q) {
                const int p = (rank + q) % NR;
                if ((quad[p] == quad[rank]) == (pass == 0)) P.dws[n++] = (char*)d_ws_all[p];
            }
    }
    P.rank = rank;
    hipLaunchKernelGGL(fused_ag_gemm, dim3(256), dim3(768), 0, stream, P);
    HIP_CHECK(hipGetLastError());
}
```

```cpp
#include <hip/hip_runtime.h>
#include <rccl/rccl.h>
#include <cstdio>
#include <cstdint>
#include <type_traits>

#define HIP_CHECK(expr) do { hipError_t _e = (expr); if (_e != hipSuccess) { \
    fprintf(stderr, "HIP error %s at %s:%d\n", hipGetErrorString(_e), __FILE__, __LINE__); \
    return; } } while (0)

#define TAU16 434
#define TAU_AT 3700
#define TAU_ADD 30
#define LAGW 4
#define ORDER 0
#define PW0 9
#define PW_M 12
#define PG 2
#define PULL_K 4
#define PULL_DEFER 800
#define AV_SLEEP 2
#define REBASE 50
#define PU 3
#define PUSH_DT 53
#define AV_LEAD 1
#define TOFF 240
#define S_FULL 31
#define S_LAST 7
#define NSP 2
#define S_HALF 13
#define POLL_MID 6
#define S_PREV 31
#define SKEW_L 24
#define SKEW_G 3
#define DCAP 2
#define DCAP2 8
#define FLAG_AT 3
#define FLAG_DEFER 500
#define CAP_SPAN 1200
#define OUT_AUX 17
#define PUSH_AUX 17
#define SRC_AUX 0
#define POLL_SLOW 200
#define POLL_FAST 2
#define POLL_LEAD 90

namespace {

typedef unsigned u32x4 __attribute__((ext_vector_type(4)));
typedef float f32x16 __attribute__((ext_vector_type(16)));
#define LAS __attribute__((address_space(3)))
#define GAS __attribute__((address_space(1)))

constexpr int NR = 8;
constexpr int DIM = 4096;
constexpr int KLOC = DIM / NR;
constexpr int NCH = 8;
constexpr int NBLK = 32;
constexpr int NSTEP = NR * NCH;
constexpr unsigned ITEM = 16384;
constexpr unsigned HALF = 8192;
constexpr size_t STAGE_PAR = (size_t)NR * NCH * NBLK * ITEM;
constexpr size_t FLAG_OFF = 2 * STAGE_PAR;
constexpr size_t AVAIL_OFF = FLAG_OFF + 32768;
constexpr size_t CTR_OFF = AVAIL_OFF + 32768;
constexpr size_t TMO_OFF = CTR_OFF + 32768;
constexpr size_t DBG_OFF = TMO_OFF + 4096;
constexpr size_t WS_NEED = DBG_OFF + 257 * 1024;
constexpr unsigned long long DEADLINE_TICKS = 50000000ull;

constexpr int L_FULL = 0;
constexpr int L_FREE = 512;
constexpr int L_SCHED = 1024;
constexpr int L_SCALE = 1536;
constexpr int L_CTRL_BYTES = 2048;
constexpr int L_RING = 8192;
constexpr int L_TOTAL = L_RING + 4 * 32768;

struct Params {
    const unsigned char* x;
    const unsigned char* w;
    const float* sx;
    const float* sw;
    float* out;
    char* myws;
    char* dws[NR];
    int rank;
    int drank[NR];
    int erot[NR];
    int rot[NR];
};

struct Sched { unsigned t[NBLK][16]; };
constexpr int blk_pos(int b) {
    return 2 * (b & 15) + (b >> 4) + 1;
}
constexpr int grp_end(int b, int g) {
    const int u = blk_pos(b), c = g >> 1;
    const int base = c < NCH - NSP ? (c * 32 + u) * 16 : g * 256 + u * 8;
    const int lead = g + 1 + 2 * SKEW_G - 16 > 0 ? g + 1 + 2 * SKEW_G - 16 : 0;
    return base - lead * SKEW_L * (32 - u) / (2 * SKEW_G);
}
constexpr int store_key(int b, int g, int u) {
    const int c = g >> 1;
    if (c < NCH - NSP) {
        const int gap = (grp_end(b, g | 1) - grp_end(b, (g | 1) - 2)) / 16 - 1;
        int stride = c == NCH - 1 ? S_LAST : c >= NCH - NSP - 1 ? S_PREV : S_FULL;
        if (stride > gap) stride = gap;
        return grp_end(b, g | 1) - (15 - (g & 1) * 8 - u) * stride + 1024;
    }
    const int gap = (grp_end(b, g) - grp_end(b, g - 1)) / 8 - 1;
    int stride = g == 15 ? S_LAST : S_HALF;
    if (stride > gap) stride = gap;
    return grp_end(b, g) - (7 - u) * stride + 1024;
}
constexpr unsigned slot_time(unsigned n) { return TOFF * 4 + (n * TAU16 + (n > TAU_AT ? (n - TAU_AT) * TAU_ADD : 0)) / 64; }
constexpr Sched make_sched() {
    Sched r = {};
    unsigned short cnt[5184] = {};
    for (int b = 0; b < NBLK; ++b) for (int g = 0; g < 16; ++g) for (int u = g ? 0 : PU; u < 8; ++u) ++cnt[store_key(b, g, u) + 1];
    for (int k = 1; k < 5184; ++k) cnt[k] += cnt[k - 1];
    for (int b = 0; b < NBLK; ++b)
        for (int g = 0; g < 16; ++g) {
            unsigned first = 0, last = 0;
            for (int u = g ? 0 : PU; u < 8; ++u) { last = cnt[store_key(b, g, u)]++; if (u == (g ? 0 : PU)) first = last; }
            r.t[b][g] = slot_time(first) | (slot_time(last) << 16);
        }
    return r;
}
__device__ constexpr Sched SEND_SCHED = make_sched();

__device__ __forceinline__ __amdgpu_buffer_rsrc_t make_rsrc(const void* p, unsigned bytes) {
    return __builtin_amdgcn_make_buffer_rsrc((void*)p, 0, (int)bytes, 0x00020000);
}

template <int AUX = 17>
__device__ __forceinline__ void store16_wt(u32x4 v, __amdgpu_buffer_rsrc_t r, int voff, int soff) {
    __builtin_amdgcn_raw_buffer_store_b128(v, r, voff, soff, AUX);
    __builtin_amdgcn_sched_barrier(0);
    asm volatile("s_nop 1" ::: "memory");
    __builtin_amdgcn_sched_barrier(0);
}

template <int I, int N, typename F> __device__ __forceinline__ void static_for(F&& f) {
    if constexpr (I < N) { f(std::integral_constant<int, I>{}); static_for<I + 1, N>(f); }
}

template <int N> __device__ __forceinline__ void wait_vm() {
    static_assert(N >= 0 && N <= 63, "");
    asm volatile("s_waitcnt vmcnt(%0)" :: "i"(N) : "memory");
}

#define CSTAMP(idx) do {} while (0)
#define TSTAMP(idx) do {} while (0)

template <bool IS_W, typename F, typename G>
__device__ __forceinline__ void load_item(const unsigned char* srcp, int blk, int c, int lane, u32x4 (&o)[16], F&& issued, G&& first) {
    const __amdgpu_buffer_rsrc_t src = make_rsrc(srcp, DIM * KLOC);
    if constexpr (IS_W) {
        unsigned raw[64];
        const int so = c * 64 * DIM + blk * 256;
        #pragma unroll
        for (int k = 0; k < 64; ++k) raw[k] = __builtin_amdgcn_raw_buffer_load_b32(src, lane * 4, so + k * DIM, 0);
        issued();
        #pragma unroll
        for (int r = 0; r < 16; ++r) {
            if (r == 4) {
                __builtin_amdgcn_sched_barrier(0);
                first();
                __builtin_amdgcn_sched_barrier(0);
            }
            const unsigned t0_ = __builtin_amdgcn_perm(raw[4 * r + 1], raw[4 * r], 0x05010400u);
            const unsigned t1_ = __builtin_amdgcn_perm(raw[4 * r + 1], raw[4 * r], 0x07030602u);
            const unsigned t2_ = __builtin_amdgcn_perm(raw[4 * r + 3], raw[4 * r + 2], 0x05010400u);
            const unsigned t3_ = __builtin_amdgcn_perm(raw[4 * r + 3], raw[4 * r + 2], 0x07030602u);
            const int q = (r >> 3) * 8 + ((r >> 2) & 1);
            o[q + 0][r & 3] = __builtin_amdgcn_perm(t2_, t0_, 0x05040100u);
            o[q + 2][r & 3] = __builtin_amdgcn_perm(t2_, t0_, 0x07060302u);
            o[q + 4][r & 3] = __builtin_amdgcn_perm(t3_, t1_, 0x05040100u);
            o[q + 6][r & 3] = __builtin_amdgcn_perm(t3_, t1_, 0x07060302u);
        }
    } else {
        const int so = blk * 256 * KLOC + c * 64, vo = (lane & 31) * KLOC + (lane >> 5) * 16;
        #pragma unroll
        for (int t = 0; t < 8; ++t) {
            o[t] = __builtin_amdgcn_raw_buffer_load_b128(src, vo, so + t * 32 * KLOC, 0);
            o[8 + t] = __builtin_amdgcn_raw_buffer_load_b128(src, vo, so + t * 32 * KLOC + 32, 0);
        }
        issued();
        first();
    }
}

__device__ __forceinline__ void xfer_wave(const Params& P, int wg, int lane, unsigned* dbg) {
    unsigned t0 = (unsigned)__builtin_amdgcn_s_memrealtime();
    const unsigned tstart = t0;
    char* const myws = P.myws;
    const int rank = P.rank, b = wg >> 3;
    unsigned* ctr = (unsigned*)(myws + CTR_OFF + wg * 128);
    unsigned* claims = (unsigned*)(myws + CTR_OFF + b * 8 * 128 + 32);
    auto claim_add = [&](int sl, unsigned d) __attribute__((always_inline)) {
        unsigned old;
        asm volatile("s_mov_b64 exec, 1\n\ts_nop 4\n\tglobal_atomic_add %0, %1, %2, %3 sc0\n\ts_mov_b64 exec, -1"
                     : "=&v"(old) : "v"(sl * 128), "v"(d), "s"(claims) : "memory");
        return old;
    };
    auto claim_wait = [&](unsigned& v) __attribute__((always_inline)) {
        asm volatile("s_waitcnt vmcnt(0)" : "+v"(v) :: "memory");
        return (unsigned)__builtin_amdgcn_readfirstlane(v);
    };
    auto pace = [&](int tgt) __attribute__((always_inline)) {
        for (;;) {
            const int left = tgt - (int)((unsigned)__builtin_amdgcn_s_memrealtime() - t0) * 28;
            if (left <= 0) break;
            if (left > 40 * 28) __builtin_amdgcn_s_sleep(8); else if (left > 6 * 28) __builtin_amdgcn_s_sleep(1);
        }
    };
    unsigned xcc;
    asm volatile("s_getreg_b32 %0, hwreg(HW_REG_XCC_ID)" : "=s"(xcc));
    int e = (int)(xcc & 7u);
    const unsigned prev = __hip_atomic_load(ctr, __ATOMIC_RELAXED, __HIP_MEMORY_SCOPE_AGENT);
    unsigned got = claim_add(e, 1u);
    unsigned* const availp = (unsigned*)(myws + AVAIL_OFF + b * 128);
    const int voff = lane * 16;
    unsigned call = 0, pv = 0;
    const GAS unsigned* chk = nullptr;
    char* peer = nullptr;
    int pr = 0, rot = 0;
    auto stage = [&](auto firstpass) __attribute__((always_inline)) {
        u32x4 it[16];
        const int so = (e * 2 * NBLK + b) * HALF;
        int tr = 0;
        auto issued = [&]() __attribute__((always_inline)) {
            __builtin_amdgcn_sched_barrier(0);
            peer = P.dws[e]; pr = P.drank[e]; rot = P.erot[e];
            asm volatile("" : "+s"(peer), "+s"(pr), "+s"(rot));
            const unsigned long long a_ = lane < 9 ? (unsigned long long)peer + AVAIL_OFF + b * 128 + (lane < 8 ? (lane + rot) & 7 : 8 + rot) * 4
                                                   : (unsigned long long)myws + TMO_OFF;
            chk = (const GAS unsigned*)a_;
            if (decltype(firstpass)::value) __builtin_amdgcn_s_barrier();
        };
        auto first = [&]() __attribute__((always_inline)) {
            call = __builtin_amdgcn_readfirstlane(prev) + 1;
            const size_t mywin = (size_t)(call & 1) * STAGE_PAR + (size_t)rank * (NCH * NBLK * ITEM);
            store16_wt(it[0], make_rsrc(peer + mywin, NCH * NBLK * ITEM), voff, so);
            tr = (int)((unsigned)__builtin_amdgcn_s_memrealtime() - t0) * 28;
        };
        if (b < 16) load_item<false>(P.x, b, e, lane, it, issued, first); else load_item<true>(P.w, b - 16, e, lane, it, issued, first);
        const size_t mywin = (size_t)(call & 1) * STAGE_PAR + (size_t)rank * (NCH * NBLK * ITEM);
        const __amdgpu_buffer_rsrc_t own = make_rsrc(myws + mywin, NCH * NBLK * ITEM);
        const __amdgpu_buffer_rsrc_t far = make_rsrc(peer + mywin, NCH * NBLK * ITEM);
        #pragma unroll
        for (int u = 0; u < 16; ++u) store16_wt(it[u], own, voff, so + (u >> 3) * (NBLK * HALF) + (u & 7) * 1024);
        #pragma unroll
        for (int u = 1; u < PU; ++u) {
            pace(tr + u * PUSH_DT * 28);
            if (u == PU - 1 && lane < 10) pv = __hip_atomic_load(chk, __ATOMIC_RELAXED, __HIP_MEMORY_SCOPE_SYSTEM);
            store16_wt(it[u], far, voff, so + u * 1024);
            if (u == PU - 1 - AV_LEAD) {
                asm volatile("s_waitcnt vmcnt(%1)" : "+v"(got) : "i"(PU - 1 - AV_LEAD) : "memory");
                __hip_atomic_store(availp + e, call, __ATOMIC_RELAXED, __HIP_MEMORY_SCOPE_SYSTEM);
            }
        }
    };
    stage(std::true_type{});
    unsigned gs = __builtin_amdgcn_readfirstlane(got);
    if (gs != call - 1) {
        #pragma nounroll
        for (int tries = 0; tries < 7 && gs != call - 1; ++tries) {
            got = claim_add(e, ~0u);
            claim_wait(got);
            e = (e + 1) & 7;
            got = claim_add(e, 1u);
            gs = claim_wait(got);
        }
        stage(std::false_type{});
    }
    const size_t win = (size_t)(call & 1) * STAGE_PAR + (size_t)pr * (NCH * NBLK * ITEM);
    const __amdgpu_buffer_rsrc_t rsrc = make_rsrc(peer + win, NCH * NBLK * ITEM);
    const __amdgpu_buffer_rsrc_t ldst = make_rsrc(myws + win, NCH * NBLK * ITEM);
    const __amdgpu_buffer_rsrc_t lfl = make_rsrc(myws + FLAG_OFF, 32768);
    const int voff_fl = lane < 8 ? lane * 4 : 0x7ffffff0;
    const int fo = (b * NR + pr) * 128;
    const unsigned* tsched = SEND_SCHED.t[b];
    const unsigned base = call * 32;
    int have = __builtin_ctzll(~__ballot(lane < 9 && pv >= call));
    auto await = [&](int need_) __attribute__((always_inline)) {
        while (have < need_) {
            unsigned v = 0;
            if (lane < 10) v = __hip_atomic_load(chk, __ATOMIC_RELAXED, __HIP_MEMORY_SCOPE_SYSTEM);
            have = __builtin_ctzll(~__ballot(lane < 9 && v >= call));
            if (have >= need_) break;
            if (__builtin_amdgcn_readlane(v, 9) == call) return false;
            if ((unsigned)__builtin_amdgcn_s_memrealtime() - tstart > (unsigned)DEADLINE_TICKS) {
                __hip_atomic_store((unsigned*)(myws + TMO_OFF), call, __ATOMIC_RELAXED, __HIP_MEMORY_SCOPE_AGENT);
                return false;
            }
            __builtin_amdgcn_s_sleep(AV_SLEEP);
        }
        return true;
    };
    if (e == 0) {
        __hip_atomic_store(ctr, call, __ATOMIC_RELAXED, __HIP_MEMORY_SCOPE_AGENT);
        if (await(NCH)) __builtin_amdgcn_raw_buffer_store_b32(base + 16, lfl, voff_fl, fo, 17);
        return;
    }
    auto pushed_done = [&]() __attribute__((always_inline)) {
        __hip_atomic_store(availp + 8 + e, call, __ATOMIC_RELAXED, __HIP_MEMORY_SCOPE_SYSTEM);
        __hip_atomic_store(ctr, call, __ATOMIC_RELAXED, __HIP_MEMORY_SCOPE_AGENT);
    };
    if (!await(1)) { wait_vm<0>(); pushed_done(); return; }
    {
        const unsigned now = (unsigned)__builtin_amdgcn_s_memrealtime();
        if ((int)(now - t0) > TOFF + REBASE) t0 = now - (TOFF + REBASE);
    }
    {
        const int so = (rot * 2 * NBLK + b) * HALF;
        const unsigned ts = tsched[0];
        const int ta = (ts & 0xffffu) * 7, span = (int)((ts >> 16) - (ts & 0xffffu)) * 7 / (7 - PU);
        u32x4 o[8];
        static_for<PU, 8>([&](auto uc) __attribute__((always_inline)) {
            constexpr int u = decltype(uc)::value;
            pace(ta + span * (u - PU));
            o[u] = __builtin_amdgcn_raw_buffer_load_b128(rsrc, voff, so + u * 1024, 17);
            __builtin_amdgcn_sched_barrier(0);
            if constexpr (u >= PU + 2) store16_wt(o[u - 2], ldst, voff, so + (u - 2) * 1024);
        });
        store16_wt(o[6], ldst, voff, so + 6 * 1024);
        store16_wt(o[7], ldst, voff, so + 7 * 1024);
        pushed_done();
    }
    int lag = 0;
    unsigned pend = 0;
    #pragma nounroll
    for (int g = 1; g < 16; ++g) {
        const int so = (((g + 2 * rot) & 15) * NBLK + b) * HALF;
        const unsigned ts = tsched[g];
        const int span = (ts >> 16) - (ts & 0xffffu);
        u32x4 o[8];
        if (span >= CAP_SPAN) {
            const int ta = (ts & 0xffffu) * 7;
            static_for<0, 8>([&](auto uc) __attribute__((always_inline)) {
                constexpr int u = decltype(uc)::value;
                pace(ta + span * u);
                if (u == 2 && pend) {
                    wait_vm<3>();
                    __builtin_amdgcn_raw_buffer_store_b32(pend, lfl, voff_fl, fo, 17);
                    pend = 0;
                }
                if (u == 3 || u == 7) {
                    const int l_ = (int)((unsigned)__builtin_amdgcn_s_memrealtime() - t0) * 28 - (ta + span * u);
                    lag += (l_ - lag) * LAGW / 16;
                }
                o[u] = __builtin_amdgcn_raw_buffer_load_b128(rsrc, voff, so + u * 1024, 17);
                __builtin_amdgcn_sched_barrier(0);
                if constexpr (u > 0) store16_wt(o[u - 1], ldst, voff, so + (u - 1) * 1024);
            });
            store16_wt(o[7], ldst, voff, so + 7 * 1024);
        } else {
            const int ta = (ts & 0xffffu) * 7 + lag;
            static_for<0, 8>([&](auto uc) __attribute__((always_inline)) {
                constexpr int u = decltype(uc)::value;
                pace(ta + span * u);
                if (u == FLAG_AT && pend) {
                    wait_vm<FLAG_AT>();
                    __builtin_amdgcn_raw_buffer_store_b32(pend, lfl, voff_fl, fo, 17);
                    pend = 0;
                }
                o[u] = __builtin_amdgcn_raw_buffer_load_b128(rsrc, voff, so + u * 1024, 17);
                __builtin_amdgcn_sched_barrier(0);
                if constexpr (u >= PULL_K) store16_wt(o[u - PULL_K], ldst, voff, so + (u - PULL_K) * 1024);
            });
            #pragma unroll
            for (int u = 8 - PULL_K; u < 8; ++u) store16_wt(o[u], ldst, voff, so + u * 1024);
        }
        if ((g & 1) || g >= 16 - 2 * NSP) {
            if (!await(9)) break;
            const unsigned nx = g < 15 ? tsched[g + 1] : 0xffffffffu;
            if ((int)((nx & 0xffffu) + ((nx >> 16) - (nx & 0xffffu)) * 2 / 7 - (ts >> 16)) < PULL_DEFER) {
                pend = base + g + 1;
            } else {
                wait_vm<0>();
                __builtin_amdgcn_raw_buffer_store_b32(base + g + 1, lfl, voff_fl, fo, 17);
            }
        }
    }
}
static_assert(FLAG_AT <= PULL_K && PU >= 2 + AV_LEAD && PU <= 6, "");

#define MFMA(c, a, b) "v_mfma_f32_32x32x64_f8f6f4 %[" #c "], " a ", " b ", %[" #c "] cbsz:0 blgp:1\n\t"
#define X0 "v[136:143]"
#define X1 "v[144:151]"
#define Y0 "v[152:159]"
#define Y1 "v[160:167]"
#define STR2(x) #x
#define STR(x) STR2(x)
#define RD(lo, hi, addr, off1, off2) \
    "ds_read_b128 " lo ", %[" #addr "] offset:" off1 "\n\t" \
    "ds_read_b128 " hi ", %[" #addr "] offset:" off2 "\n\t"
#define RD_X0(addr, o1, o2) RD("v[136:139]", "v[140:143]", addr, o1, o2)
#define RD_X1(addr, o1, o2) RD("v[144:147]", "v[148:151]", addr, o1, o2)
#define RD_Y0(addr, o1, o2) RD("v[152:155]", "v[156:159]", addr, o1, o2)
#define RD_Y1(addr, o1, o2) RD("v[160:163]", "v[164:167]", addr, o1, o2)
#define WAITL(n) "s_waitcnt lgkmcnt(" #n ")\n\t"
#define CHECK(FS) \
    "1:\n\t" \
    WAITL(0) \
    "v_readlane_b32 %[st], %[vf], %[sw]\n\t" \
    "s_nop 0\n\t" \
    "s_cmp_ge_u32 %[st], %[need]\n\t" \
    "s_cbranch_scc1 2f\n\t" \
    "s_sleep 1\n\t" \
    "ds_read_addtid_b32 %[vf] offset:" STR(FS) "*32\n\t" \
    "s_branch 1b\n\t" \
    "2:\n\t"
#define RELEASE(FS, NX) \
    WAITL(0) \
    "s_mov_b64 exec, %[mask]\n\t" \
    "ds_write_addtid_b32 %[vf] offset:512\n\t" \
    "s_mov_b64 exec, -1\n\t" \
    "ds_read_addtid_b32 %[vf] offset:" STR(NX) "*32\n\t"
#define PRETOUCH \
    "s_sub_u32 %[st], %[need], %[pw0]\n\t" \
    "s_cmp_lt_u32 %[st], 32\n\t" \
    "s_cbranch_scc0 6f\n\t" \
    "s_and_b32 %[t1], %[st], 12\n\t" \
    "s_add_u32 %[t1], %[t1], %[st]\n\t" \
    "s_and_b32 %[st], %[st], 16\n\t" \
    "s_mul_i32 %[st], %[st], 7\n\t" \
    "s_add_u32 %[t1], %[t1], %[st]\n\t" \
    "s_lshl_b32 %[t1], %[t1], 14\n\t" \
    "buffer_store_dwordx4 v[160:163], %[vo], %[srd], %[t1] offen " PW_SC "\n\t" \
    "6:\n\t"
#define PW_SC "sc0 sc1"
#define STEP_FULL(SO, FS, NX) \
    CHECK(FS) \
    RD_X0(va, STR(SO), STR(SO) "+512") \
    RD_Y0(vb, STR(SO), STR(SO) "+1024") \
    RD_X1(va, STR(SO) "+4096", STR(SO) "+4608") \
    RD_Y1(vb, STR(SO) "+2048", STR(SO) "+3072") \
    WAITL(4) \
    MFMA(c00, X0, Y0) \
    WAITL(2) \
    MFMA(c10, X1, Y0) \
    RD_Y0(vb, STR(SO) "+4096", STR(SO) "+5120") \
    WAITL(2) \
    MFMA(c01, X0, Y1) \
    MFMA(c11, X1, Y1) \
    RD_Y1(vb, STR(SO) "+6144", STR(SO) "+7168") \
    WAITL(2) \
    MFMA(c02, X0, Y0) \
    MFMA(c12, X1, Y0) \
    RELEASE(FS, NX) \
    MFMA(c03, X0, Y1) \
    PRETOUCH \
    "s_add_u32 %[need], %[need], 1\n\t" \
    MFMA(c13, X1, Y1)
#define FLIP \
    "v_xor_b32 %[va], 0x10000, %[va]\n\t" \
    "v_xor_b32 %[vb], 0x10000, %[vb]\n\t"
#define FRAG_CLOBBERS \
    "v136", "v137", "v138", "v139", "v140", "v141", "v142", "v143", "v144", "v145", "v146", "v147", \
    "v148", "v149", "v150", "v151", "v152", "v153", "v154", "v155", "v156", "v157", "v158", "v159", \
    "v160", "v161", "v162", "v163", "v164", "v165", "v166", "v167"

__global__ void __launch_bounds__(768, 3) fused_ag_gemm(Params P) {
    __shared__ __attribute__((aligned(1024))) unsigned char lds[L_TOTAL];
    LAS unsigned char* L8 = (LAS unsigned char*)lds;
    const int wave = __builtin_amdgcn_readfirstlane(threadIdx.x >> 6);
    const int lane = threadIdx.x & 63;
    const int wg = blockIdx.x;
    const int e = wg & 7, idx = wg >> 3;
    const int ti = (e >> 1) + 4 * (idx >> 3);
    const int tj = (e & 1) + 2 * (idx & 7);
    char* const myws = P.myws;
    const int rank = P.rank;

    int wv_ = wave;
    asm volatile("" : "+s"(wv_));
    if (__builtin_expect(wv_ == 8, 1)) {
        __builtin_amdgcn_s_setprio(3);
        xfer_wave(P, wg, lane, (unsigned*)(myws + DBG_OFF + wg * 1024 + 128));
        wait_vm<0>();
        TSTAMP(1);
        __builtin_amdgcn_endpgm();
    }
    if (wave == 9) {
        __builtin_amdgcn_s_setprio(3);
        const unsigned long long t0 = __builtin_amdgcn_s_memrealtime();
        unsigned* ctr = (unsigned*)(myws + CTR_OFF + wg * 128 + 64);
        unsigned* tmo = (unsigned*)(myws + TMO_OFF);
        const unsigned prev = __hip_atomic_load(ctr, __ATOMIC_RELAXED, __HIP_MEMORY_SCOPE_AGENT);
        const float scale = P.sx[0] * P.sw[0];
        __builtin_amdgcn_s_barrier();
        const unsigned call = __builtin_amdgcn_readfirstlane(prev) + 1;
        __hip_atomic_store(ctr, call, __ATOMIC_RELAXED, __HIP_MEMORY_SCOPE_AGENT);
        const int p = (rank + (lane >> 1)) & 7, o = lane & 1;
        const unsigned* fl = lane < 16 ? (const unsigned*)(myws + FLAG_OFF + (size_t)(((o ? 16 + tj : ti) * NR + p) * 128)) : tmo;
        volatile LAS unsigned* sched = (volatile LAS unsigned*)(L8 + L_SCHED);
        volatile LAS float* lscale = (volatile LAS float*)(L8 + L_SCALE);
        *lscale = scale;
        const int rot2 = 2 * P.rot[p];
        const unsigned base = call * 32;
        const unsigned code0 = ((call & 1) * 128 + p * 16) + 1;
        int cur = 0, nsched = 0, known = 0;
        bool dead = false;
        auto due = [&](int g) { const unsigned a = SEND_SCHED.t[ti][g] >> 16, b = SEND_SCHED.t[16 + tj][g] >> 16; return (a > b ? a : b) / 4 + POLL_LEAD; };
        const unsigned t_fast = due(15), t_mid = due(NSP ? 16 - 2 * NSP : 15);
        for (;;) {
            if (!dead) {
                unsigned val = 0;
                const bool act_ = lane == 16 || (lane < 16 && known <= cur && cur < 2 * NCH);
                if (act_) val = __hip_atomic_load(fl, __ATOMIC_RELAXED, __HIP_MEMORY_SCOPE_SYSTEM);
                if (__builtin_amdgcn_readlane(val, 16) == call) dead = true;
                if (act_) known = (int)(val - base);
            }
            if (dead) { *lscale = __builtin_nanf(""); known = 31; }
            const int other = __builtin_amdgcn_update_dpp(0, known, 0xB1, 0xf, 0xf, false);
            const int av = known < other ? known : other;
            for (;;) {
                const bool rdy = lane < 16 && av > cur && cur < 2 * NCH;
                const unsigned long long m = __ballot(rdy) & 0x5555ull;
                if (m == 0) break;
                if (rdy && o == 0) sched[nsched + __builtin_popcountll(m & ((1ull << lane) - 1))] = code0 + ((cur + rot2) & 15);
                cur += rdy ? 1 : 0;
                nsched += __builtin_popcountll(m);
            }
            if (nsched >= 2 * NSTEP) break;
            const unsigned long long now = __builtin_amdgcn_s_memrealtime() - t0;
            if (now > DEADLINE_TICKS) {
                __hip_atomic_store(tmo, call, __ATOMIC_RELAXED, __HIP_MEMORY_SCOPE_AGENT);
                dead = true;
            } else if ((unsigned)now >= t_fast) {
                __builtin_amdgcn_s_sleep(POLL_FAST);
            } else if ((unsigned)now >= t_mid) {
                __builtin_amdgcn_s_sleep(POLL_MID);
            } else {
                const unsigned until = (unsigned)now + POLL_SLOW < t_mid ? (unsigned)now + POLL_SLOW : t_mid;
                while ((unsigned)(__builtin_amdgcn_s_memrealtime() - t0) < until) __builtin_amdgcn_s_sleep(4);
            }
        }
        TSTAMP(2);
    } else if (wave >= 10) {
        __builtin_amdgcn_s_setprio(3);
        __builtin_amdgcn_s_barrier();
        const int o = wave - 10;
        volatile LAS unsigned* sched = (volatile LAS unsigned*)(L8 + L_SCHED);
        volatile LAS unsigned* freew = (volatile LAS unsigned*)(L8 + L_FREE);
        volatile LAS unsigned* fullw = (volatile LAS unsigned*)(L8 + L_FULL);
        const char* stage_x = myws + (size_t)ti * HALF;
        const char* stage_w = myws + (size_t)(16 + tj) * HALF;
        const unsigned v0 = lane * 16, v1 = v0 + 4096;
        auto dma8 = [&](const char* src, unsigned ldst) __attribute__((always_inline)) {
            unsigned keep;
#define DMA4(V) "global_load_lds_dwordx4 " V ", %[src] sc0 sc1\n\t" \
                "global_load_lds_dwordx4 " V ", %[src] offset:1024 sc0 sc1\n\t" \
                "global_load_lds_dwordx4 " V ", %[src] offset:2048 sc0 sc1\n\t" \
                "global_load_lds_dwordx4 " V ", %[src] offset:3072 sc0 sc1\n\t"
            asm volatile(
                "s_mov_b32 %[keep], m0\n\t"
                "s_mov_b32 m0, %[ld]\n\t"
                "s_nop 0\n\t"
                DMA4("%[v0]")
                "s_add_u32 m0, m0, 4096\n\t"
                "s_nop 0\n\t"
                DMA4("%[v1]")
                "s_mov_b32 m0, %[keep]\n\t"
                : [keep] "=&s"(keep) : [src] "s"(src), [ld] "s"(ldst), [v0] "v"(v0), [v1] "v"(v1) : "memory", "scc");
        };
        int issued = o, pub = o;
        while (pub < NSTEP) {
            bool did = false;
            if (issued < NSTEP && issued - pub < 4) {
                const unsigned codev = sched[2 * issued + (lane & 1)];
                const int fr = (int)freew[lane & 7];
                const unsigned code0 = __builtin_amdgcn_readlane(codev, 0), code1 = __builtin_amdgcn_readlane(codev, 1);
                if (code0 != 0u && code1 != 0u && __all(fr >= issued - 3)) {
                    size_t off0 = (size_t)((code0 - 1) & 255u) * (NBLK * HALF), off1 = (size_t)((code1 - 1) & 255u) * (NBLK * HALF);
                    asm volatile("s_nop 4" : "+s"(off0), "+s"(off1));
                    const unsigned ldst = L_RING + (issued & 3) * 32768;
                    dma8(stage_x + off0, ldst); dma8(stage_w + off0, ldst + 16384); dma8(stage_x + off1, ldst + 8192); dma8(stage_w + off1, ldst + 24576);
                    issued += 2;
                    did = true;
                }
            }
            if (!did || issued - pub >= 4) {
                if (issued > pub) {
                    if (issued - pub == 2) wait_vm<0>(); else wait_vm<32>();
                    fullw[(pub & 3) * 8 + (lane & 7)] = pub + 1;
                    pub += 2;
                } else {
                    __builtin_amdgcn_s_sleep(1);
                }
            }
        }
        if (o == 0) TSTAMP(3);
    } else {
        if (wave == 0) {
            #pragma unroll
            for (int i = 0; i < L_CTRL_BYTES / 256; ++i) ((LAS unsigned*)L8)[i * 64 + lane] = 0u;
            asm volatile("s_waitcnt lgkmcnt(0)" ::: "memory");
            TSTAMP(0);
            CSTAMP(9);
        }
        __builtin_amdgcn_s_barrier();
        if (wave >= 4) __builtin_amdgcn_s_setprio(1);
        const int wm = wave >> 1, wn = wave & 1;
        const int r = lane & 31, h = lane >> 5;
        unsigned va = L_RING + h * 8192 + wm * 1024 + r * 16;
        unsigned vb = L_RING + 16384 + h * 8192 + wn * 512 + r * 16;
        f32x16 c00 = {}, c01 = {}, c02 = {}, c03 = {}, c10 = {}, c11 = {}, c12 = {}, c13 = {};
        unsigned vf, st, need, keep;
        const unsigned long long mask = 1ull << wave;
        float* outp = P.out + (size_t)(ti * 256 + wm * 32) * DIM + tj * 256 + wn * 128;
        asm volatile("" : "+s"(outp));
        unsigned t1;
        const unsigned pw0 = (ti > tj ? ti : tj) >= PW_M ? PW0 : 1000u;
        const u32x4 srd = {(unsigned)(size_t)outp, (unsigned)((size_t)outp >> 32) & 0xffffu, 160u * DIM * 4u, 0x00020000u};
        const unsigned vo = (h * 4 * DIM + r * 4) * 4;
        asm volatile(
            "s_mov_b32 %[keep], m0\n\t"
            "s_mov_b32 m0, 0\n\t"
            "s_mov_b32 %[need], 1\n\t"
            "ds_read_addtid_b32 %[vf] offset:0\n\t"
            "3:\n\t"
            STEP_FULL(0, 0, 1)
            STEP_FULL(32768, 1, 2)
            FLIP
            STEP_FULL(0, 2, 3)
            STEP_FULL(32768, 3, 0)
            FLIP
            "s_cmp_le_u32 %[need], %[nfull]\n\t"
            "s_cbranch_scc1 3b\n\t"
            "s_waitcnt vmcnt(0) lgkmcnt(0)\n\t"
            "s_mov_b32 m0, %[keep]\n\t"
            "s_nop 15\n\t"
            "s_nop 7\n\t"
            : [c00] "+v"(c00), [c01] "+v"(c01), [c02] "+v"(c02), [c03] "+v"(c03),
              [c10] "+v"(c10), [c11] "+v"(c11), [c12] "+v"(c12), [c13] "+v"(c13),
              [va] "+v"(va), [vb] "+v"(vb), [vf] "=&v"(vf), [st] "=&s"(st), [need] "=&s"(need), [keep] "=&s"(keep), [t1] "=&s"(t1)
            : [sw] "s"(wave), [mask] "s"(mask), [nfull] "i"(NSTEP), [srd] "s"(srd), [vo] "v"(vo), [pw0] "s"(pw0)
            : "memory", "scc", FRAG_CLOBBERS);
        if (wave == 0) { TSTAMP(8); CSTAMP(10); }
        const int lz = __builtin_amdgcn_mbcnt_hi(~0u, __builtin_amdgcn_mbcnt_lo(~0u, 0u));
        const __amdgpu_buffer_rsrc_t outr = make_rsrc(outp, 160u * DIM * 4u);
        const int voff = ((lz >> 5) * 4 * DIM + (lz & 31) * 4) * 4;
        const float sc = __builtin_bit_cast(float, __builtin_amdgcn_readfirstlane(*(volatile LAS unsigned*)(L8 + L_SCALE)));
        auto act = [&](float a) { return __builtin_bit_cast(unsigned, __builtin_elementwise_maximum(a * sc, 0.f)); };
        #pragma unroll
        for (int g = 0; g < 16; ++g) {
            u32x4 v0 = {act(c00[g]), act(c01[g]), act(c02[g]), act(c03[g])};
            store16_wt<OUT_AUX>(v0, outr, voff, ((g & 3) + 8 * (g >> 2)) * DIM * 4);
            u32x4 v1 = {act(c10[g]), act(c11[g]), act(c12[g]), act(c13[g])};
            store16_wt<OUT_AUX>(v1, outr, voff, (128 + (g & 3) + 8 * (g >> 2)) * DIM * 4);
        }
    }
}

}

extern "C" void kernel_launch(int rank, int nranks, ncclComm_t comm,
                              hipStream_t stream,
                              void* const* d_in, const int* in_sizes, int n_in,
                              void* d_out, int out_size,
                              const int* in_dtypes, int out_dtype,
                              void* const* d_out_all,
                              void* const* d_ws_all, size_t ws_bytes) {
    if (nranks != NR || n_in < 4 || ws_bytes < WS_NEED || (unsigned)rank >= (unsigned)NR ||
        in_sizes[0] != DIM * KLOC || in_sizes[1] != DIM * KLOC || out_size != DIM * DIM) {
        fprintf(stderr, "kernel_launch: unexpected problem shape\n");
        return;
    }
    Params P;
    P.x = (const unsigned char*)d_in[0];
    P.w = (const unsigned char*)d_in[1];
    P.sx = (const float*)d_in[2];
    P.sw = (const float*)d_in[3];
    P.out = (float*)d_out;
    P.myws = (char*)d_ws_all[rank];
    {
        const int quad[NR] = {0, 1, 0, 0, 1, 1, 0, 1};
        for (int r = 0; r < NR; ++r) {
            int n = 0;
            for (int pass = 0; pass < 2; ++pass)
                for (int q = 0; q < NR; ++q) {
                    const int p = (r + q) % NR;
                    if ((quad[p] == quad[r]) != (pass == 0)) continue;
                    if (r == rank) { P.dws[n] = (char*)d_ws_all[p]; P.drank[n] = p; }
                    if (p == rank) P.rot[r] = n;
                    ++n;
                }
        }
    }
    for (int e = 0; e < NR; ++e) P.erot[e] = P.rot[P.drank[e]];
    P.rank = rank;
    hipLaunchKernelGGL(fused_ag_gemm, dim3(256), dim3(768), 0, stream, P);
    HIP_CHECK(hipGetLastError());
}
```

```cpp
#include <hip/hip_runtime.h>
#include <rccl/rccl.h>
#include <cstdio>
#include <cstdint>
#include <type_traits>

#define HIP_CHECK(expr) do { hipError_t _e = (expr); if (_e != hipSuccess) { \
    fprintf(stderr, "HIP error %s at %s:%d\n", hipGetErrorString(_e), __FILE__, __LINE__); \
    return; } } while (0)

namespace {

typedef unsigned u32x4 __attribute__((ext_vector_type(4)));
typedef float f32x16 __attribute__((ext_vector_type(16)));
#define LAS __attribute__((address_space(3)))

constexpr int NR = 8;
constexpr int DIM = 4096;
constexpr int KLOC = DIM / NR;
constexpr int NCH = 8;
constexpr int NBLK = 32;
constexpr int NSTEP = NR * NCH;
constexpr unsigned ITEM = 16384;
constexpr unsigned HALF = 8192;
constexpr size_t STAGE_PAR = (size_t)NR * NCH * NBLK * ITEM;
constexpr size_t FLAG_OFF = 2 * STAGE_PAR;
constexpr size_t CTR_OFF = FLAG_OFF + 32768;
constexpr size_t TMO_OFF = CTR_OFF + 32768;
constexpr size_t WS_NEED = TMO_OFF + 128;
constexpr unsigned long long DEADLINE_TICKS = 150000000ull;

constexpr int TAU16 = 440;
constexpr int TAU_AT = 3700, TAU_ADD = 30;
constexpr int TOFF = 50;
constexpr int S_FULL = 31;
constexpr int S_HALF = 13, S_LAST = 7;
constexpr int NSP = 2;
constexpr int SKEW_L = 24;
constexpr int SKEW_G = 3;
constexpr int DCAP = 2;
constexpr int CAP_SPAN = 1200;
constexpr int FLAG_AT = 3, FLAG_DEFER = 500;
constexpr int POLL_SLOW = 200;
constexpr int POLL_MID = 20, POLL_FAST = 1;
constexpr int POLL_LEAD = 90;
constexpr int PW0 = 9, PW_M = 12;

constexpr int L_FULL = 0;
constexpr int L_FREE = 512;
constexpr int L_SCHED = 1024;
constexpr int L_SCALE = 1536;
constexpr int L_CTRL_BYTES = 2048;
constexpr int L_RING = 8192;
constexpr int L_TOTAL = L_RING + 4 * 32768;

struct Params {
    const unsigned char* x;
    const unsigned char* w;
    const float* sx;
    const float* sw;
    float* out;
    char* myws;
    char* dws[NR];
    int rank;
};

struct Sched { unsigned t[NBLK][16]; };
constexpr int blk_pos(int b) { return 2 * (b & 15) + (b >> 4) + 1; }
constexpr int grp_end(int b, int g) {
    const int u = blk_pos(b), c = g >> 1;
    const int base = c < NCH - NSP ? (c * 32 + u) * 16 : g * 256 + u * 8;
    const int lead = g + 1 + 2 * SKEW_G - 16 > 0 ? g + 1 + 2 * SKEW_G - 16 : 0;
    return base - lead * SKEW_L * (32 - u) / (2 * SKEW_G);
}
constexpr int store_key(int b, int g, int u) {
    const int c = g >> 1;
    if (c < NCH - NSP) {
        const int gap = (grp_end(b, g | 1) - grp_end(b, (g | 1) - 2)) / 16 - 1;
        int stride = S_FULL;
        if (stride > gap) stride = gap;
        return grp_end(b, g | 1) - (15 - (g & 1) * 8 - u) * stride + 1024;
    }
    const int gap = (grp_end(b, g) - grp_end(b, g - 1)) / 8 - 1;
    int stride = g == 15 ? S_LAST : S_HALF;
    if (stride > gap) stride = gap;
    return grp_end(b, g) - (7 - u) * stride + 1024;
}
constexpr unsigned slot_time(unsigned n) { return TOFF * 4 + (n * TAU16 + (n > TAU_AT ? (n - TAU_AT) * TAU_ADD : 0)) / 64; }
constexpr Sched make_sched() {
    Sched r = {};
    unsigned short cnt[5184] = {};
    for (int b = 0; b < NBLK; ++b) for (int g = 0; g < 16; ++g) for (int u = 0; u < 8; ++u) ++cnt[store_key(b, g, u) + 1];
    for (int k = 1; k < 5184; ++k) cnt[k] += cnt[k - 1];
    for (int b = 0; b < NBLK; ++b)
        for (int g = 0; g < 16; ++g) {
            unsigned first = 0, last = 0;
            for (int u = 0; u < 8; ++u) { last = cnt[store_key(b, g, u)]++; if (u == 0) first = last; }
            r.t[b][g] = slot_time(first) | (slot_time(last) << 16);
        }
    return r;
}
__device__ constexpr Sched SEND_SCHED = make_sched();

__device__ __forceinline__ __amdgpu_buffer_rsrc_t make_rsrc(const void* p, unsigned bytes) {
    return __builtin_amdgcn_make_buffer_rsrc((void*)p, 0, (int)bytes, 0x00020000);
}

__device__ __forceinline__ void store16_wt(u32x4 v, __amdgpu_buffer_rsrc_t r, int voff, int soff) {
    __builtin_amdgcn_raw_buffer_store_b128(v, r, voff, soff, 17);
    __builtin_amdgcn_sched_barrier(0);
    asm volatile("s_nop 1" ::: "memory");
    __builtin_amdgcn_sched_barrier(0);
}

template <int I, int N, typename F> __device__ __forceinline__ void static_for(F&& f) {
    if constexpr (I < N) { f(std::integral_constant<int, I>{}); static_for<I + 1, N>(f); }
}

template <int N> __device__ __forceinline__ void wait_vm() {
    static_assert(N >= 0 && N <= 63, "");
    asm volatile("s_waitcnt vmcnt(%0)" :: "i"(N) : "memory");
}

template <bool IS_W>
__device__ __forceinline__ void push_loop(const Params& P, const unsigned char* srcp, int rank, int blk,
                                          unsigned* ctr, unsigned* claims, int lane) {
    const unsigned t0 = (unsigned)__builtin_amdgcn_s_memrealtime();
    const __amdgpu_buffer_rsrc_t src = make_rsrc(srcp, DIM * KLOC);
    const int voff_ld = IS_W ? lane * 4 : (lane & 31) * KLOC + (lane >> 5) * 16;
    unsigned rawA[32], rawB[32];
    auto load_w = [&](int g, unsigned (&raw)[32]) __attribute__((always_inline)) {
        const int so = g * 32 * DIM + blk * 256;
        #pragma unroll
        for (int k = 0; k < 32; ++k) raw[k] = __builtin_amdgcn_raw_buffer_load_b32(src, voff_ld, so + k * DIM, 0);
    };
    auto load_x = [&](int c) __attribute__((always_inline)) {
        const int so = blk * 256 * KLOC + c * 64;
        #pragma unroll
        for (int t = 0; t < 8; ++t) {
            u32x4 v = __builtin_amdgcn_raw_buffer_load_b128(src, voff_ld, so + t * 32 * KLOC, 0);
            rawA[4 * t] = v.x; rawA[4 * t + 1] = v.y; rawA[4 * t + 2] = v.z; rawA[4 * t + 3] = v.w;
            v = __builtin_amdgcn_raw_buffer_load_b128(src, voff_ld, so + t * 32 * KLOC + 32, 0);
            rawB[4 * t] = v.x; rawB[4 * t + 1] = v.y; rawB[4 * t + 2] = v.z; rawB[4 * t + 3] = v.w;
        }
    };
    auto claim_add = [&](int sl, unsigned d) __attribute__((always_inline)) {
        unsigned old;
        asm volatile("s_mov_b64 exec, 1\n\ts_nop 4\n\tglobal_atomic_add %0, %1, %2, %3 sc0\n\ts_mov_b64 exec, -1"
                     : "=&v"(old) : "v"(sl * 128), "v"(d), "s"(claims) : "memory");
        return old;
    };
    auto claim_wait = [&](unsigned& v) __attribute__((always_inline)) {
        asm volatile("s_waitcnt vmcnt(0)" : "+v"(v) :: "memory");
        return (unsigned)__builtin_amdgcn_readfirstlane(v);
    };
    unsigned xcc;
    asm volatile("s_getreg_b32 %0, hwreg(HW_REG_XCC_ID)" : "=s"(xcc));
    int e = (int)(xcc & 7u);
    const unsigned prev = __hip_atomic_load(ctr, __ATOMIC_RELAXED, __HIP_MEMORY_SCOPE_AGENT);
    unsigned got = claim_add(e, 1u);
    if constexpr (IS_W) load_w(0, rawA); else load_x(0);
    char* dws = P.dws[e];
    const unsigned* tsched = SEND_SCHED.t[(IS_W ? 16 : 0) | blk];
    { unsigned warm = tsched[0]; asm volatile("" :: "s"(warm), "s"(dws)); }
    __builtin_amdgcn_s_barrier();
    asm volatile("s_waitcnt vmcnt(%1)" : "+v"(got) : "i"(IS_W ? 32 : 16) : "memory");
    unsigned gs = __builtin_amdgcn_readfirstlane(got);
    const unsigned call = __builtin_amdgcn_readfirstlane(prev) + 1;
    #pragma nounroll
    for (int tries = 0; tries < 7 && gs != call - 1; ++tries) {
        got = claim_add(e, ~0u);
        claim_wait(got);
        e = (e + 1) & 7;
        got = claim_add(e, 1u);
        gs = claim_wait(got);
        dws = P.dws[e];
    }
    const bool paced = e != 0;
    const __amdgpu_buffer_rsrc_t dstr = make_rsrc(dws + (size_t)(call & 1) * STAGE_PAR + (size_t)rank * (NCH * NBLK * ITEM), NCH * NBLK * ITEM);
    const __amdgpu_buffer_rsrc_t flr = make_rsrc(dws + FLAG_OFF, 32768);
    const int voff_st = lane * 16;
    const int voff_fl = lane < 8 ? lane * 4 : 0x7ffffff0;
    const int b = blk + (IS_W ? 16 : 0);
    int lag = 0;
    unsigned pend = 0;
    auto send_grp = [&](int g, unsigned (&raw)[32], auto first, auto odd) __attribute__((always_inline)) {
        u32x4 o[8];
        auto xpose = [&](int r0, int r1) __attribute__((always_inline)) {
            #pragma unroll
            for (int r = r0; r < r1; ++r) {
                const unsigned t0_ = __builtin_amdgcn_perm(raw[4 * r + 1], raw[4 * r], 0x05010400u);
                const unsigned t1_ = __builtin_amdgcn_perm(raw[4 * r + 1], raw[4 * r], 0x07030602u);
                const unsigned t2_ = __builtin_amdgcn_perm(raw[4 * r + 3], raw[4 * r + 2], 0x05010400u);
                const unsigned t3_ = __builtin_amdgcn_perm(raw[4 * r + 3], raw[4 * r + 2], 0x07030602u);
                o[0 + (r >> 2)][r & 3] = __builtin_amdgcn_perm(t2_, t0_, 0x05040100u);
                o[2 + (r >> 2)][r & 3] = __builtin_amdgcn_perm(t2_, t0_, 0x07060302u);
                o[4 + (r >> 2)][r & 3] = __builtin_amdgcn_perm(t3_, t1_, 0x05040100u);
                o[6 + (r >> 2)][r & 3] = __builtin_amdgcn_perm(t3_, t1_, 0x07060302u);
            }
        };
        if constexpr (IS_W) {
            xpose(0, decltype(first)::value ? 4 : 8);
        } else {
            #pragma unroll
            for (int t = 0; t < 8; ++t) o[t] = u32x4{raw[4 * t], raw[4 * t + 1], raw[4 * t + 2], raw[4 * t + 3]};
        }
        if constexpr (!decltype(first)::value) {
            #pragma unroll
            for (int u = 0; u < 8; ++u) asm volatile("" : "+v"(o[u]));
        }
        const int so = (g * NBLK + b) * HALF;
        const unsigned ts = tsched[g];
        const int span = (ts >> 16) - (ts & 0xffffu);
        const bool capped = paced && span >= CAP_SPAN;
        const int ta = (ts & 0xffffu) * 7 + (capped ? 0 : lag);
        static_for<0, 8>([&](auto uc) __attribute__((always_inline)) {
            constexpr int u = decltype(uc)::value, L = IS_W ? 32 : (decltype(odd)::value ? 16 : 0), LU = decltype(first)::value ? 4 : 0;
            if (paced) {
                const int tgt = ta + span * u;
                for (;;) {
                    const int left = tgt - (int)((unsigned)__builtin_amdgcn_s_memrealtime() - t0) * 28;
                    if (left <= 0 || left > (1 << 24)) break;
                    if (left > 40 * 28) __builtin_amdgcn_s_sleep(8); else if (left > 6 * 28) __builtin_amdgcn_s_sleep(1);
                }
            }
            if (decltype(first)::value && u < 2) {}
            else if (capped) wait_vm<DCAP - 1 + ((u > LU && u <= LU + DCAP) ? L : 0)>();
            if (u == FLAG_AT && pend) {
                wait_vm<FLAG_AT>();
                __builtin_amdgcn_raw_buffer_store_b32(pend, flr, voff_fl, (b * NR + rank) * 128, 17);
                pend = 0;
            }
            if (capped && (u == 3 || u == 7)) {
                const int l_ = (int)((unsigned)__builtin_amdgcn_s_memrealtime() - t0) * 28 - (ta + span * u);
                lag += (l_ - lag) / 4;
            }
            store16_wt(o[u], dstr, voff_st, so + u * 1024);
            if constexpr (IS_W) {
                if (decltype(first)::value && u == 0) { xpose(4, 8); __builtin_amdgcn_sched_barrier(0); load_w(1, rawB); }
                if (u == LU && g < 14) load_w(g + 2, raw);
            } else if (decltype(odd)::value && u == 0 && g < 14) {
                load_x((g + 1) >> 1);
            }
        });
        if (decltype(odd)::value || g >= 16 - 2 * NSP) {
            const unsigned nx = g < 15 ? tsched[g + 1] : 0xffffffffu;
            if (FLAG_AT > 0 && paced && (int)((nx & 0xffffu) + ((nx >> 16) - (nx & 0xffffu)) * FLAG_AT / 7 - (ts >> 16)) < FLAG_DEFER) {
                pend = call * 32 + g + 1;
            } else {
                wait_vm<0>();
                __builtin_amdgcn_raw_buffer_store_b32(call * 32 + g + 1, flr, voff_fl, (b * NR + rank) * 128, 17);
            }
        }
    };
    send_grp(0, rawA, std::true_type{}, std::false_type{});
    send_grp(1, rawB, std::false_type{}, std::true_type{});
    #pragma nounroll
    for (int g = 2; g < 16; g += 2) {
        send_grp(g, rawA, std::false_type{}, std::false_type{});
        send_grp(g + 1, rawB, std::false_type{}, std::true_type{});
    }
    __hip_atomic_store(ctr, call, __ATOMIC_RELAXED, __HIP_MEMORY_SCOPE_AGENT);
    wait_vm<0>();
}

#define MFMA(c, a, b) "v_mfma_f32_32x32x64_f8f6f4 %[" #c "], " a ", " b ", %[" #c "] cbsz:0 blgp:1\n\t"
#define X0 "v[136:143]"
#define X1 "v[144:151]"
#define Y0 "v[152:159]"
#define Y1 "v[160:167]"
#define STR2(x) #x
#define STR(x) STR2(x)
#define RD(lo, hi, addr, off1, off2) \
    "ds_read_b128 " lo ", %[" #addr "] offset:" off1 "\n\t" \
    "ds_read_b128 " hi ", %[" #addr "] offset:" off2 "\n\t"
#define RD_X0(addr, o1, o2) RD("v[136:139]", "v[140:143]", addr, o1, o2)
#define RD_X1(addr, o1, o2) RD("v[144:147]", "v[148:151]", addr, o1, o2)
#define RD_Y0(addr, o1, o2) RD("v[152:155]", "v[156:159]", addr, o1, o2)
#define RD_Y1(addr, o1, o2) RD("v[160:163]", "v[164:167]", addr, o1, o2)
#define WAITL(n) "s_waitcnt lgkmcnt(" #n ")\n\t"
#define CHECK(FS) \
    "1:\n\t" \
    WAITL(0) \
    "v_readlane_b32 %[st], %[vf], %[sw]\n\t" \
    "s_nop 0\n\t" \
    "s_cmp_ge_u32 %[st], %[need]\n\t" \
    "s_cbranch_scc1 2f\n\t" \
    "s_sleep 1\n\t" \
    "ds_read_addtid_b32 %[vf] offset:" STR(FS) "*32\n\t" \
    "s_branch 1b\n\t" \
    "2:\n\t"
#define RELEASE(FS, NX) \
    WAITL(0) \
    "s_mov_b64 exec, %[mask]\n\t" \
    "ds_write_addtid_b32 %[vf] offset:512\n\t" \
    "s_mov_b64 exec, -1\n\t" \
    "ds_read_addtid_b32 %[vf] offset:" STR(NX) "*32\n\t"
#define PRETOUCH \
    "s_sub_u32 %[st], %[need], %[pw0]\n\t" \
    "s_cmp_lt_u32 %[st], 32\n\t" \
    "s_cbranch_scc0 6f\n\t" \
    "s_and_b32 %[t1], %[st], 12\n\t" \
    "s_add_u32 %[t1], %[t1], %[st]\n\t" \
    "s_and_b32 %[st], %[st], 16\n\t" \
    "s_mul_i32 %[st], %[st], 7\n\t" \
    "s_add_u32 %[t1], %[t1], %[st]\n\t" \
    "s_lshl_b32 %[t1], %[t1], 14\n\t" \
    "buffer_store_dwordx4 v[160:163], %[vo], %[srd], %[t1] offen sc0 sc1\n\t" \
    "6:\n\t"
#define STEP_FULL(SO, FS, NX) \
    CHECK(FS) \
    RD_X0(va, STR(SO), STR(SO) "+512") \
    RD_Y0(vb, STR(SO), STR(SO) "+1024") \
    RD_X1(va, STR(SO) "+4096", STR(SO) "+4608") \
    RD_Y1(vb, STR(SO) "+2048", STR(SO) "+3072") \
    WAITL(4) \
    MFMA(c00, X0, Y0) \
    WAITL(2) \
    MFMA(c10, X1, Y0) \
    RD_Y0(vb, STR(SO) "+4096", STR(SO) "+5120") \
    WAITL(2) \
    MFMA(c01, X0, Y1) \
    MFMA(c11, X1, Y1) \
    RD_Y1(vb, STR(SO) "+6144", STR(SO) "+7168") \
    WAITL(2) \
    MFMA(c02, X0, Y0) \
    MFMA(c12, X1, Y0) \
    RELEASE(FS, NX) \
    MFMA(c03, X0, Y1) \
    PRETOUCH \
    "s_add_u32 %[need], %[need], 1\n\t" \
    MFMA(c13, X1, Y1)
#define FLIP \
    "v_xor_b32 %[va], 0x10000, %[va]\n\t" \
    "v_xor_b32 %[vb], 0x10000, %[vb]\n\t"
#define FRAG_CLOBBERS \
    "v136", "v137", "v138", "v139", "v140", "v141", "v142", "v143", "v144", "v145", "v146", "v147", \
    "v148", "v149", "v150", "v151", "v152", "v153", "v154", "v155", "v156", "v157", "v158", "v159", \
    "v160", "v161", "v162", "v163", "v164", "v165", "v166", "v167"

__global__ void __launch_bounds__(768, 3) fused_ag_gemm(Params P) {
    __shared__ __attribute__((aligned(1024))) unsigned char lds[L_TOTAL];
    LAS unsigned char* L8 = (LAS unsigned char*)lds;
    const int wave = __builtin_amdgcn_readfirstlane(threadIdx.x >> 6);
    const int lane = threadIdx.x & 63;
    const int wg = blockIdx.x;
    const int e = wg & 7, idx = wg >> 3;
    const int ti = (e >> 1) + 4 * (idx >> 3);
    const int tj = (e & 1) + 2 * (idx & 7);
    char* const myws = P.myws;
    const int rank = P.rank;

    int wv_ = wave;
    asm volatile("" : "+s"(wv_));
    if (__builtin_expect(wv_ == 8, 1)) {
        __builtin_amdgcn_s_setprio(3);
        unsigned* ctr = (unsigned*)(myws + CTR_OFF + wg * 128);
        unsigned* claims = (unsigned*)(myws + CTR_OFF + idx * 8 * 128 + 32);
        if (__builtin_expect(idx < 16, 1)) { push_loop<false>(P, P.x, rank, idx, ctr, claims, lane); __builtin_amdgcn_endpgm(); }
        push_loop<true>(P, P.w, rank, idx - 16, ctr, claims, lane);
        __builtin_amdgcn_endpgm();
    }
    if (wave == 9) {
        __builtin_amdgcn_s_setprio(3);
        const unsigned long long t0 = __builtin_amdgcn_s_memrealtime();
        unsigned* ctr = (unsigned*)(myws + CTR_OFF + wg * 128 + 64);
        unsigned* tmo = (unsigned*)(myws + TMO_OFF);
        const unsigned prev = __hip_atomic_load(ctr, __ATOMIC_RELAXED, __HIP_MEMORY_SCOPE_AGENT);
        const float scale = P.sx[0] * P.sw[0];
        __builtin_amdgcn_s_barrier();
        const unsigned call = __builtin_amdgcn_readfirstlane(prev) + 1;
        __hip_atomic_store(ctr, call, __ATOMIC_RELAXED, __HIP_MEMORY_SCOPE_AGENT);
        const int p = (rank + (lane >> 1)) & 7, o = lane & 1;
        const unsigned* fl = lane < 16 ? (const unsigned*)(myws + FLAG_OFF + (size_t)(((o ? 16 + tj : ti) * NR + p) * 128)) : tmo;
        volatile LAS unsigned* sched = (volatile LAS unsigned*)(L8 + L_SCHED);
        volatile LAS float* lscale = (volatile LAS float*)(L8 + L_SCALE);
        *lscale = scale;
        const unsigned base = call * 32;
        const unsigned code0 = ((call & 1) * 128 + p * 16) + 1;
        int cur = 0, nsched = 0, known = 0;
        bool dead = false;
        auto due = [&](int g) { const unsigned a = SEND_SCHED.t[ti][g] >> 16, b = SEND_SCHED.t[16 + tj][g] >> 16; return (a > b ? a : b) / 4 + POLL_LEAD; };
        const unsigned t_fast = due(15), t_mid = due(NSP ? 16 - 2 * NSP : 15);
        for (;;) {
            if (!dead) {
                unsigned val = 0;
                const bool act_ = lane == 16 || (lane < 16 && known <= cur && cur < 2 * NCH);
                if (act_) val = __hip_atomic_load(fl, __ATOMIC_RELAXED, __HIP_MEMORY_SCOPE_SYSTEM);
                if (__builtin_amdgcn_readlane(val, 16) == call) dead = true;
                if (act_) known = (int)(val - base);
            }
            if (dead) { *lscale = __builtin_nanf(""); known = 31; }
            const int other = __builtin_amdgcn_update_dpp(0, known, 0xB1, 0xf, 0xf, false);
            const int av = known < other ? known : other;
            for (;;) {
                const bool rdy = lane < 16 && av > cur && cur < 2 * NCH;
                const unsigned long long m = __ballot(rdy) & 0x5555ull;
                if (m == 0) break;
                if (rdy && o == 0) sched[nsched + __builtin_popcountll(m & ((1ull << lane) - 1))] = code0 + cur;
                cur += rdy ? 1 : 0;
                nsched += __builtin_popcountll(m);
            }
            if (nsched >= 2 * NSTEP) break;
            const unsigned long long now = __builtin_amdgcn_s_memrealtime() - t0;
            if (now > DEADLINE_TICKS) {
                __hip_atomic_store(tmo, call, __ATOMIC_RELAXED, __HIP_MEMORY_SCOPE_AGENT);
                dead = true;
            } else if ((unsigned)now >= t_fast) {
                __builtin_amdgcn_s_sleep(POLL_FAST);
            } else if ((unsigned)now >= t_mid) {
                __builtin_amdgcn_s_sleep(POLL_MID);
            } else {
                const unsigned until = (unsigned)now + POLL_SLOW < t_mid ? (unsigned)now + POLL_SLOW : t_mid;
                while ((unsigned)(__builtin_amdgcn_s_memrealtime() - t0) < until) __builtin_amdgcn_s_sleep(4);
            }
        }
    } else if (wave >= 10) {
        __builtin_amdgcn_s_setprio(3);
        __builtin_amdgcn_s_barrier();
        const int o = wave - 10;
        volatile LAS unsigned* sched = (volatile LAS unsigned*)(L8 + L_SCHED);
        volatile LAS unsigned* freew = (volatile LAS unsigned*)(L8 + L_FREE);
        volatile LAS unsigned* fullw = (volatile LAS unsigned*)(L8 + L_FULL);
        const char* stage_x = myws + (size_t)ti * HALF;
        const char* stage_w = myws + (size_t)(16 + tj) * HALF;
        const unsigned v0 = lane * 16, v1 = v0 + 4096;
        auto dma8 = [&](const char* src, unsigned ldst) __attribute__((always_inline)) {
            unsigned keep;
#define DMA4(V) "global_load_lds_dwordx4 " V ", %[src] sc0 sc1\n\t" \
                "global_load_lds_dwordx4 " V ", %[src] offset:1024 sc0 sc1\n\t" \
                "global_load_lds_dwordx4 " V ", %[src] offset:2048 sc0 sc1\n\t" \
                "global_load_lds_dwordx4 " V ", %[src] offset:3072 sc0 sc1\n\t"
            asm volatile(
                "s_mov_b32 %[keep], m0\n\t"
                "s_mov_b32 m0, %[ld]\n\t"
                "s_nop 0\n\t"
                DMA4("%[v0]")
                "s_add_u32 m0, m0, 4096\n\t"
                "s_nop 0\n\t"
                DMA4("%[v1]")
                "s_mov_b32 m0, %[keep]\n\t"
                : [keep] "=&s"(keep) : [src] "s"(src), [ld] "s"(ldst), [v0] "v"(v0), [v1] "v"(v1) : "memory", "scc");
        };
        int issued = o, pub = o;
        while (pub < NSTEP) {
            bool did = false;
            if (issued < NSTEP && issued - pub < 4) {
                const unsigned codev = sched[2 * issued + (lane & 1)];
                const int fr = (int)freew[lane & 7];
                const unsigned code0 = __builtin_amdgcn_readlane(codev, 0), code1 = __builtin_amdgcn_readlane(codev, 1);
                if (code0 != 0u && code1 != 0u && __all(fr >= issued - 3)) {
                    size_t off0 = (size_t)((code0 - 1) & 255u) * (NBLK * HALF), off1 = (size_t)((code1 - 1) & 255u) * (NBLK * HALF);
                    asm volatile("s_nop 4" : "+s"(off0), "+s"(off1));
                    const unsigned ldst = L_RING + (issued & 3) * 32768;
                    dma8(stage_x + off0, ldst); dma8(stage_w + off0, ldst + 16384); dma8(stage_x + off1, ldst + 8192); dma8(stage_w + off1, ldst + 24576);
                    issued += 2;
                    did = true;
                }
            }
            if (!did || issued - pub >= 4) {
                if (issued > pub) {
                    if (issued - pub == 2) wait_vm<0>(); else wait_vm<32>();
                    fullw[(pub & 3) * 8 + (lane & 7)] = pub + 1;
                    pub += 2;
                } else {
                    __builtin_amdgcn_s_sleep(1);
                }
            }
        }
    } else {
        if (wave == 0) {
            #pragma unroll
            for (int i = 0; i < L_CTRL_BYTES / 256; ++i) ((LAS unsigned*)L8)[i * 64 + lane] = 0u;
            asm volatile("s_waitcnt lgkmcnt(0)" ::: "memory");
        }
        __builtin_amdgcn_s_barrier();
        if (wave >= 4) __builtin_amdgcn_s_setprio(1);
        const int wm = wave >> 1, wn = wave & 1;
        const int r = lane & 31, h = lane >> 5;
        unsigned va = L_RING + h * 8192 + wm * 1024 + r * 16;
        unsigned vb = L_RING + 16384 + h * 8192 + wn * 512 + r * 16;
        f32x16 c00 = {}, c01 = {}, c02 = {}, c03 = {}, c10 = {}, c11 = {}, c12 = {}, c13 = {};
        unsigned vf, st, need, keep;
        const unsigned long long mask = 1ull << wave;
        float* outp = P.out + (size_t)(ti * 256 + wm * 32) * DIM + tj * 256 + wn * 128;
        asm volatile("" : "+s"(outp));
        unsigned t1;
        const unsigned pw0 = (ti > tj ? ti : tj) >= PW_M ? PW0 : 1000u;
        const u32x4 srd = {(unsigned)(size_t)outp, (unsigned)((size_t)outp >> 32) & 0xffffu, 160u * DIM * 4u, 0x00020000u};
        const unsigned vo = (h * 4 * DIM + r * 4) * 4;
        asm volatile(
            "s_mov_b32 %[keep], m0\n\t"
            "s_mov_b32 m0, 0\n\t"
            "s_mov_b32 %[need], 1\n\t"
            "ds_read_addtid_b32 %[vf] offset:0\n\t"
            "3:\n\t"
            STEP_FULL(0, 0, 1)
            STEP_FULL(32768, 1, 2)
            FLIP
            STEP_FULL(0, 2, 3)
            STEP_FULL(32768, 3, 0)
            FLIP
            "s_cmp_le_u32 %[need], %[nfull]\n\t"
            "s_cbranch_scc1 3b\n\t"
            "s_waitcnt vmcnt(0) lgkmcnt(0)\n\t"
            "s_mov_b32 m0, %[keep]\n\t"
            "s_nop 15\n\t"
            "s_nop 7\n\t"
            : [c00] "+v"(c00), [c01] "+v"(c01), [c02] "+v"(c02), [c03] "+v"(c03),
              [c10] "+v"(c10), [c11] "+v"(c11), [c12] "+v"(c12), [c13] "+v"(c13),
              [va] "+v"(va), [vb] "+v"(vb), [vf] "=&v"(vf), [st] "=&s"(st), [need] "=&s"(need), [keep] "=&s"(keep), [t1] "=&s"(t1)
            : [sw] "s"(wave), [mask] "s"(mask), [nfull] "i"(NSTEP), [srd] "s"(srd), [vo] "v"(vo), [pw0] "s"(pw0)
            : "memory", "scc", FRAG_CLOBBERS);
        const int lz = __builtin_amdgcn_mbcnt_hi(~0u, __builtin_amdgcn_mbcnt_lo(~0u, 0u));
        const __amdgpu_buffer_rsrc_t outr = make_rsrc(outp, 160u * DIM * 4u);
        const int voff = ((lz >> 5) * 4 * DIM + (lz & 31) * 4) * 4;
        const float sc = __builtin_bit_cast(float, __builtin_amdgcn_readfirstlane(*(volatile LAS unsigned*)(L8 + L_SCALE)));
        auto act = [&](float a) { return __builtin_bit_cast(unsigned, __builtin_elementwise_maximum(a * sc, 0.f)); };
        #pragma unroll
        for (int g = 0; g < 16; ++g) {
            u32x4 v0 = {act(c00[g]), act(c01[g]), act(c02[g]), act(c03[g])};
            store16_wt(v0, outr, voff, ((g & 3) + 8 * (g >> 2)) * DIM * 4);
            u32x4 v1 = {act(c10[g]), act(c11[g]), act(c12[g]), act(c13[g])};
            store16_wt(v1, outr, voff, (128 + (g & 3) + 8 * (g >> 2)) * DIM * 4);
        }
    }
}

}

extern "C" void kernel_launch(int rank, int nranks, ncclComm_t comm,
                              hipStream_t stream,
                              void* const* d_in, const int* in_sizes, int n_in,
                              void* d_out, int out_size,
                              const int* in_dtypes, int out_dtype,
                              void* const* d_out_all,
                              void* const* d_ws_all, size_t ws_bytes) {
    if (nranks != NR || n_in < 4 || ws_bytes < WS_NEED || (unsigned)rank >= (unsigned)NR ||
        in_sizes[0] != DIM * KLOC || in_sizes[1] != DIM * KLOC || out_size != DIM * DIM) {
        fprintf(stderr, "kernel_launch: unexpected problem shape\n");
        return;
    }
    Params P;
    P.x = (const unsigned char*)d_in[0];
    P.w = (const unsigned char*)d_in[1];
    P.sx = (const float*)d_in[2];
    P.sw = (const float*)d_in[3];
    P.out = (float*)d_out;
    P.myws = (char*)d_ws_all[rank];
    {
        const int quad[NR] = {0, 1, 0, 0, 1, 1, 0, 1};
        int n = 0;
        for (int pass = 0; pass < 2; ++pass)
            for (int q = 0; q < NR; ++q) {
                const int p = (rank + q) % NR;
                if ((quad[p] == quad[rank]) == (pass == 0)) P.dws[n++] = (char*)d_ws_all[p];
            }
    }
    P.rank = rank;
    hipLaunchKernelGGL(fused_ag_gemm, dim3(256), dim3(768), 0, stream, P);
    HIP_CHECK(hipGetLastError());
}
```
